```python
import numpy as np

kernel_name = "gpu_ag_gemm_m4096_k4096_n4096_w8a8_dequant"


ABSMAX_RTOL = 0.001

COMPUTE_DTYPE = "int8"
EPILOGUE = "dequant"
IN_QUANT = "int8"
OUT_QUANT = None
IN_DTYPES = {"x": "int8", "w_mat": "int8",
             "scale_x": "float32", "scale_w": "float32"}
WORLD_SIZE = 8

def setup_inputs():

    x = np.random.randint(-127, 128, size=(4096, 4096)).astype(np.int8)
    w_mat = np.random.randint(-127, 128, size=(4096, 4096)).astype(np.int8)
    scale_x = (0.02 * (1.0 + np.random.rand(1))).astype(np.float32)
    scale_w = (0.02 * (1.0 + np.random.rand(1))).astype(np.float32)
    return {"x": x, "w_mat": w_mat, "scale_x": scale_x, "scale_w": scale_w}

def reference(x, w_mat, scale_x, scale_w):
    acc = x.astype(np.float64) @ w_mat.astype(np.float64)
    y = (acc.astype(np.float32) * (scale_x * scale_w)).astype(np.float32)
    return y
```

```cpp
#include <hip/hip_runtime.h>
#include <rccl/rccl.h>
#include <cstdio>
#include <cstdint>

#define HIP_CHECK(expr) do { hipError_t _e = (expr); if (_e != hipSuccess) { \
    fprintf(stderr, "HIP error %s at %s:%d\n", hipGetErrorString(_e), __FILE__, __LINE__); \
    return; } } while (0)

#define GAS __attribute__((address_space(1)))
#define LAS __attribute__((address_space(3)))
typedef unsigned u32x4 __attribute__((ext_vector_type(4)));
typedef int i32x4 __attribute__((ext_vector_type(4)));
typedef float f32x4 __attribute__((ext_vector_type(4)));
typedef GAS unsigned gu32;

constexpr int NR = 8;
constexpr int MS = 512;
constexpr int KD = 4096;
constexpr int NS = 512;
constexpr int GRID = 256;
constexpr int NWAVES = 9;
constexpr int UNIT = 4096;
constexpr size_t SHARD = (size_t)MS * KD;
constexpr size_t OFF_DATA = 0;
constexpr size_t OFF_FLAG = 2 * NR * SHARD;
constexpr size_t OFF_CNT = OFF_FLAG + NR * 32 * 128;
constexpr size_t OFF_TMO = OFF_CNT + GRID * 128;
constexpr size_t OFF_DBG = OFF_TMO + 4096;
constexpr size_t WS_NEED = OFF_DBG + GRID * NWAVES * 64 * 8;
constexpr unsigned long long DEADLINE_TICKS = 50000000ull;

#define OUT_AUX 16

struct Params {
    const signed char* x;
    const signed char* w;
    const float* sx;
    const float* sw;
    float* out;
    char* ws[NR];
    int rank;
    int pad;
};

__device__ __forceinline__ unsigned long long rt() { return __builtin_amdgcn_s_memrealtime(); }

#define STAMP(i) do { } while (0)

__device__ __forceinline__ void store_wt(char* p, u32x4 v) {
    asm volatile("global_store_dwordx4 %0, %1, off sc0 sc1\n\ts_nop 1" :: "v"(p), "v"(v) : "memory");
}

template <int N>
__device__ __forceinline__ void process(const int (&kc)[4], __amdgpu_buffer_rsrc_t rsrc, unsigned voff,
                                        unsigned sbase, unsigned ustride, unsigned jstride,
                                        const LAS char* lb0, const LAS char* lb1, i32x4& acc0, i32x4& acc1,
                                        const gu32* flagp, unsigned& fv) {
    u32x4 a[N][4];
#pragma unroll
    for (int u = 0; u < N; ++u)
#pragma unroll
        for (int j = 0; j < 4; ++j)
            a[u][j] = __builtin_amdgcn_raw_buffer_load_b128(rsrc, voff, sbase + kc[u] * ustride + j * jstride, 17);
    fv = __hip_atomic_load(flagp, __ATOMIC_RELAXED, __HIP_MEMORY_SCOPE_SYSTEM);
#pragma unroll
    for (int u = 0; u < N; ++u)
#pragma unroll
        for (int j = 0; j < 4; ++j) {
            const int o = (kc[u] * 4 + j) * 2048;
            i32x4 b0 = *(const LAS i32x4*)(lb0 + o);
            i32x4 b1 = *(const LAS i32x4*)(lb1 + o);
            i32x4 av = __builtin_bit_cast(i32x4, a[u][j]);
            acc0 = __builtin_amdgcn_mfma_i32_16x16x64_i8(b0, av, acc0, 0, 0, 0);
            acc1 = __builtin_amdgcn_mfma_i32_16x16x64_i8(b1, av, acc1, 0, 0, 0);
        }
}

__global__ void __launch_bounds__(NWAVES * 64, 1) ag_gemm_kernel(Params P) {
    __shared__ __attribute__((aligned(16))) char smem[KD * 32 + 16];
    const int b = blockIdx.x;
    const int lane = threadIdx.x & 63;
    const int wave = __builtin_amdgcn_readfirstlane(threadIdx.x >> 6);
    const unsigned rank = (unsigned)P.rank;
    char* myws = P.ws[rank & 7];
    LAS unsigned* arrive = (LAS unsigned*)(smem + KD * 32);
    gu32* cntp = (gu32*)(myws + OFF_CNT + (size_t)b * 128);
    gu32* tmo = (gu32*)(myws + OFF_TMO);
    STAMP(0);
    const unsigned cntv = __hip_atomic_load(cntp, __ATOMIC_RELAXED, __HIP_MEMORY_SCOPE_AGENT);

    if (wave == 0) {
        __builtin_amdgcn_s_setprio(3);
        const int kc = b >> 4, rbl = b & 15;
        u32x4 d[2][4];
#pragma unroll
        for (int g = 0; g < 2; ++g)
#pragma unroll
            for (int j = 0; j < 4; ++j)
                d[g][j] = *(const GAS u32x4*)((const GAS signed char*)P.x + (size_t)((g * 16 + rbl) * 16 + (lane & 15)) * KD
                                              + kc * 256 + j * 64 + (lane >> 4) * 16);
        if (lane == 0) *arrive = 0;
        asm volatile("s_waitcnt lgkmcnt(0)" ::: "memory");
        __builtin_amdgcn_s_barrier();
        const unsigned call = __builtin_amdgcn_readfirstlane(cntv) + 1;
        if (lane == 0) __hip_atomic_fetch_add(arrive, 1u, __ATOMIC_RELAXED, __HIP_MEMORY_SCOPE_WORKGROUP);
        STAMP(1);
        const size_t doff = OFF_DATA + (size_t)(call & 1) * NR * SHARD + (size_t)rank * SHARD;
#pragma unroll
        for (int g = 0; g < 2; ++g) {
            const int rb = g * 16 + rbl;
            for (int r = 0; r < NR - 1; ++r) {
                const unsigned dst = (rank + 1 + (unsigned)(b + r) % (NR - 1)) & 7;
                char* pw = P.ws[dst];
                char* p = pw + doff + (size_t)(rb * 16 + kc) * UNIT + lane * 16;
#pragma unroll
                for (int j = 0; j < 4; ++j) store_wt(p + j * 1024, d[g][j]);
                asm volatile("s_waitcnt vmcnt(0)" ::: "memory");
                if (lane == 0)
                    __hip_atomic_store((gu32*)(pw + OFF_FLAG + (size_t)(rank * 32 + rb) * 128 + kc * 4), call,
                                       __ATOMIC_RELAXED, __HIP_MEMORY_SCOPE_SYSTEM);
                STAMP(2 + g * 7 + r);
            }
        }
        return;
    }

    const int t = threadIdx.x - 64;
    const int xcd = b & 7, ti = b >> 3;
    const int rs = xcd * 2 + (ti >> 4);
    const int cb = ti & 15;
    {
        const int ch = t & 1, kg = t >> 1;
        const GAS signed char* wp = (const GAS signed char*)P.w + (size_t)kg * 16 * NS + cb * 32 + ch * 16;
        u32x4 r[16];
#pragma unroll
        for (int i = 0; i < 16; ++i) r[i] = *(const GAS u32x4*)(wp + (size_t)i * NS);
        __builtin_amdgcn_s_barrier();
        const int ks = kg >> 2, kq = kg & 3;
        const int s = (kq & 1) | (ch << 1) | ((kq >> 1) << 2);
        LAS char* base = (LAS char*)smem + (ks * 2 + ch) * 1024 + kq * 256;
#pragma unroll
        for (int cq = 0; cq < 4; ++cq) {
            u32x4 o[4];
#pragma unroll
            for (int q = 0; q < 4; ++q) {
                const unsigned r0 = r[4 * q][cq], r1 = r[4 * q + 1][cq], r2 = r[4 * q + 2][cq], r3 = r[4 * q + 3][cq];
                const unsigned lo01 = __builtin_amdgcn_perm(r1, r0, 0x05010400);
                const unsigned hi01 = __builtin_amdgcn_perm(r1, r0, 0x07030602);
                const unsigned lo23 = __builtin_amdgcn_perm(r3, r2, 0x05010400);
                const unsigned hi23 = __builtin_amdgcn_perm(r3, r2, 0x07030602);
                o[0][q] = __builtin_amdgcn_perm(lo23, lo01, 0x05040100);
                o[1][q] = __builtin_amdgcn_perm(lo23, lo01, 0x07060302);
                o[2][q] = __builtin_amdgcn_perm(hi23, hi01, 0x05040100);
                o[3][q] = __builtin_amdgcn_perm(hi23, hi01, 0x07060302);
            }
#pragma unroll
            for (int e = 0; e < 4; ++e) *(LAS u32x4*)(base + (((cq * 4 + e) ^ s) * 16)) = o[e];
        }
    }
    const unsigned call = __builtin_amdgcn_readfirstlane(cntv) + 1;
    const float scale = P.sx[0] * P.sw[0];
    asm volatile("s_waitcnt lgkmcnt(0)" ::: "memory");
    if (lane == 0) __hip_atomic_fetch_add(arrive, 1u, __ATOMIC_RELEASE, __HIP_MEMORY_SCOPE_WORKGROUP);
    while (__builtin_amdgcn_readfirstlane(__hip_atomic_load(arrive, __ATOMIC_ACQUIRE, __HIP_MEMORY_SCOPE_WORKGROUP)) < NWAVES)
        __builtin_amdgcn_s_sleep(1);
    if (threadIdx.x == 64) __hip_atomic_store(cntp, call, __ATOMIC_RELAXED, __HIP_MEMORY_SCOPE_AGENT);
    STAMP(1);

    const unsigned src = (unsigned)(wave - 1);
    const bool own = (src == rank);
    const int c = lane & 15, kq = lane >> 4;
    const LAS char* lb0 = (const LAS char*)smem + kq * 256 + ((c ^ ((kq & 1) | ((kq >> 1) << 2))) * 16);
    const LAS char* lb1 = (const LAS char*)smem + 1024 + kq * 256 + ((c ^ ((kq & 1) | 2 | ((kq >> 1) << 2))) * 16);
    void* abase = own ? (void*)P.x : (void*)(myws + OFF_DATA + (size_t)(call & 1) * NR * SHARD + (size_t)src * SHARD);
    const __amdgpu_buffer_rsrc_t rsrc = __builtin_amdgcn_make_buffer_rsrc(abase, 0, (int)SHARD, 0x00020000);
    const __amdgpu_buffer_rsrc_t orsrc = __builtin_amdgcn_make_buffer_rsrc(P.out, 0, NR * MS * NS * 4, 0x00020000);
    const unsigned voff = own ? (unsigned)(c * KD + kq * 16) : (unsigned)(lane * 16);
    const unsigned ustride = own ? 256u : (unsigned)UNIT;
    const unsigned jstride = own ? 64u : 1024u;
    bool dead = false;

    for (int g = 0; g < 2; ++g) {
        const int rb = g * 16 + rs;
        const unsigned sbase = (unsigned)rb * 16u * KD;
        const gu32* flagp = (const gu32*)(myws + OFF_FLAG + (size_t)(src * 32 + rb) * 128 + (lane & 15) * 4);
        i32x4 acc0 = {0, 0, 0, 0}, acc1 = {0, 0, 0, 0};
        unsigned done = 0;
        unsigned fv = own ? call : __hip_atomic_load(flagp, __ATOMIC_RELAXED, __HIP_MEMORY_SCOPE_SYSTEM);
        unsigned long long t0 = rt();
        unsigned spins = 0;
        while (done != 0xFFFFu && !dead) {
            unsigned todo = own ? 0xFFFFu : ((unsigned)__ballot(fv >= call) & 0xFFFFu);
            todo &= ~done;
            asm volatile("" ::: "memory");
            if (todo == 0) {
                __builtin_amdgcn_s_sleep(1);
                if ((++spins & 63u) == 0u) {
                    if (__builtin_amdgcn_readfirstlane(__hip_atomic_load(tmo, __ATOMIC_RELAXED, __HIP_MEMORY_SCOPE_AGENT)) == call
                        || rt() - t0 > DEADLINE_TICKS) {
                        if (lane == 0) __hip_atomic_store(tmo, call, __ATOMIC_RELAXED, __HIP_MEMORY_SCOPE_AGENT);
                        dead = true;
                    }
                }
                fv = __hip_atomic_load(flagp, __ATOMIC_RELAXED, __HIP_MEMORY_SCOPE_SYSTEM);
                continue;
            }
            const int n = __builtin_popcount(todo);
            int kc[4];
            unsigned m = todo;
            kc[0] = __builtin_ctz(m); m &= m - 1;
            kc[1] = m ? __builtin_ctz(m) : 0; m &= m - 1;
            kc[2] = m ? __builtin_ctz(m) : 0; m &= m - 1;
            kc[3] = m ? __builtin_ctz(m) : 0; m &= m - 1;
            if (n >= 4) {
                process<4>(kc, rsrc, voff, sbase, ustride, jstride, lb0, lb1, acc0, acc1, flagp, fv);
            } else if (n == 3) {
                process<3>(kc, rsrc, voff, sbase, ustride, jstride, lb0, lb1, acc0, acc1, flagp, fv);
            } else if (n == 2) {
                process<2>(kc, rsrc, voff, sbase, ustride, jstride, lb0, lb1, acc0, acc1, flagp, fv);
            } else {
                process<1>(kc, rsrc, voff, sbase, ustride, jstride, lb0, lb1, acc0, acc1, flagp, fv);
            }
            done |= todo & ~m;
        }
        STAMP(2 + 2 * g);
        if (__builtin_amdgcn_readfirstlane(__hip_atomic_load(tmo, __ATOMIC_RELAXED, __HIP_MEMORY_SCOPE_AGENT)) == call) dead = true;
        f32x4 y0, y1;
#pragma unroll
        for (int e = 0; e < 4; ++e) {
            y0[e] = dead ? __builtin_nanf("") : (float)acc0[e] * scale;
            y1[e] = dead ? __builtin_nanf("") : (float)acc1[e] * scale;
        }
        const unsigned ooff = ((src * MS + rb * 16 + c) * NS + cb * 32 + kq * 4) * 4;
        __builtin_amdgcn_raw_buffer_store_b128(__builtin_bit_cast(u32x4, y0), orsrc, ooff, 0, OUT_AUX);
        __builtin_amdgcn_raw_buffer_store_b128(__builtin_bit_cast(u32x4, y1), orsrc, ooff + 64, 0, OUT_AUX);
        STAMP(3 + 2 * g);
    }
}

extern "C" void kernel_launch(int rank, int nranks, ncclComm_t comm,
                              hipStream_t stream,
                              void* const* d_in, const int* in_sizes, int n_in,
                              void* d_out, int out_size,
                              const int* in_dtypes, int out_dtype,
                              void* const* d_out_all,
                              void* const* d_ws_all, size_t ws_bytes) {
    if (nranks != NR || (unsigned)rank >= (unsigned)NR || n_in != 4 || in_sizes[0] != MS * KD || in_sizes[1] != KD * NS
        || out_size != NR * MS * NS || ws_bytes < WS_NEED) {
        fprintf(stderr, "kernel_launch: unexpected shapes\n");
        return;
    }
    Params P;
    P.x = (const signed char*)d_in[0];
    P.w = (const signed char*)d_in[1];
    P.sx = (const float*)d_in[2];
    P.sw = (const float*)d_in[3];
    P.out = (float*)d_out;
    for (int i = 0; i < NR; ++i) P.ws[i] = (char*)d_ws_all[i];
    P.rank = rank;
    P.pad = 0;
    ag_gemm_kernel<<<GRID, NWAVES * 64, 0, stream>>>(P);
    HIP_CHECK(hipGetLastError());
}
```

```cpp
#include <hip/hip_runtime.h>
#include <rccl/rccl.h>
#include <cstdio>
#include <cstdint>

#define HIP_CHECK(expr) do { hipError_t _e = (expr); if (_e != hipSuccess) { \
    fprintf(stderr, "HIP error %s at %s:%d\n", hipGetErrorString(_e), __FILE__, __LINE__); \
    return; } } while (0)

#define GAS __attribute__((address_space(1)))
#define LAS __attribute__((address_space(3)))
typedef unsigned u32x4 __attribute__((ext_vector_type(4)));
typedef int i32x4 __attribute__((ext_vector_type(4)));
typedef float f32x4 __attribute__((ext_vector_type(4)));
typedef GAS unsigned gu32;
typedef GAS unsigned long long gu64;
typedef __amdgpu_buffer_rsrc_t rsrc_t;

#define NGB 6
#define PW 3
#define POLL_SLEEP 1
#define OWN_AUX 17
constexpr int NR = 8;
constexpr int MS = 512;
constexpr int KD = 4096;
constexpr int NS = 512;
constexpr int NAB = 32 - NGB;
constexpr int GRID = 256;
constexpr int NWAVES = 8;
constexpr int NPUSH = 56;
constexpr int RS = 12;
constexpr int NLIST = 128 + NGB;
constexpr int MAXT = (NLIST + RS - 1) / RS;
constexpr int TSTR = 65;
constexpr unsigned SHARD = MS * KD;
constexpr unsigned OUTB = NR * MS * NS * 4;
constexpr unsigned PAR_BYTES = (NR + 1) * SHARD;
constexpr unsigned OFF_W = NR * SHARD;
constexpr unsigned OFF_Y = 2 * PAR_BYTES;
constexpr unsigned OFF_XF = OFF_Y + 8 * 256 * 4096;
constexpr unsigned OFF_WF = OFF_XF + NR * 32 * 8 * 128;
constexpr unsigned OFF_DF = OFF_WF + 8192;
constexpr unsigned OFF_Q = OFF_DF + 4096;
constexpr unsigned OFF_QT = OFF_Q + 4096;
constexpr unsigned OFF_CNT = OFF_QT + 4096;
constexpr unsigned OFF_TMO = OFF_CNT + GRID * 128;
constexpr unsigned OFF_DBG = OFF_TMO + 4096;
constexpr unsigned WS_NEED = OFF_DBG + GRID * NWAVES * 64 * 8;
constexpr int NQ = 2 * (128 - NGB - RS);
constexpr int NSTAT = 128 + 16 * NGB;
constexpr int NPW = 8 * PW;
static_assert(8 * PW + 2 * RS <= 64 && NQ <= 256, "flag / queue layout");
constexpr unsigned long long DEADLINE_TICKS = 50000000ull;
constexpr int SYS = 17;

struct Params {
    const signed char* x;
    const signed char* w;
    const float* sx;
    const float* sw;
    char* ws[NR];
    char* out[NR];
    int rank;
    int pad;
};

__device__ __forceinline__ unsigned long long rt() { return __builtin_amdgcn_s_memrealtime(); }
__device__ __forceinline__ rsrc_t mk(const void* p, unsigned bytes) {
    return __builtin_amdgcn_make_buffer_rsrc((void*)p, 0, (int)bytes, 0x00020000);
}
__device__ __forceinline__ unsigned rfl(unsigned v) { return __builtin_amdgcn_readfirstlane(v); }

#define STAMP(i) do { } while (0)

__device__ __forceinline__ void transpose16x4(const unsigned (&r)[16], u32x4 (&o)[4]) {
#pragma unroll
    for (int q = 0; q < 4; ++q) {
        const unsigned r0 = r[4 * q], r1 = r[4 * q + 1], r2 = r[4 * q + 2], r3 = r[4 * q + 3];
        const unsigned lo01 = __builtin_amdgcn_perm(r1, r0, 0x05010400);
        const unsigned hi01 = __builtin_amdgcn_perm(r1, r0, 0x07030602);
        const unsigned lo23 = __builtin_amdgcn_perm(r3, r2, 0x05010400);
        const unsigned hi23 = __builtin_amdgcn_perm(r3, r2, 0x07030602);
        o[0][q] = __builtin_amdgcn_perm(lo23, lo01, 0x05040100);
        o[1][q] = __builtin_amdgcn_perm(lo23, lo01, 0x07060302);
        o[2][q] = __builtin_amdgcn_perm(hi23, hi01, 0x05040100);
        o[3][q] = __builtin_amdgcn_perm(hi23, hi01, 0x07060302);
    }
}

__device__ __forceinline__ bool expired(unsigned& spins, unsigned long long t0, gu32* tmo, unsigned call, int lane) {
    __builtin_amdgcn_s_sleep(POLL_SLEEP);
    spins = rfl(spins + 1u);
    if ((spins & 63u) != 0u) return false;
    const bool late = rfl(__hip_atomic_load(tmo, __ATOMIC_RELAXED, __HIP_MEMORY_SCOPE_AGENT)) == call
                      || rfl(rt() - t0 > DEADLINE_TICKS ? 1u : 0u) != 0u;
    if (late) __hip_atomic_store(tmo, call, __ATOMIC_RELAXED, __HIP_MEMORY_SCOPE_AGENT);
    return late;
}

__global__ void __launch_bounds__(NWAVES * 64, 2) ag_gemm_kernel(Params P) {
    __shared__ __attribute__((aligned(16))) int smem[MAXT * 16 * TSTR + 32];
    const int b = blockIdx.x;
    const int lane = threadIdx.x & 63;
    const int wave = rfl(threadIdx.x >> 6);
    const unsigned rank = (unsigned)P.rank & 7u;
    const unsigned myidx = rank & 3u, mycls = rank & 4u, otcls = mycls ^ 4u;
    char* myws = P.ws[rank];
    gu32* cntp = (gu32*)(myws + OFF_CNT + b * 128 + wave * 4);
    gu32* tmo = (gu32*)(myws + OFF_TMO);
    STAMP(0);
    const unsigned cntv = __hip_atomic_load(cntp, __ATOMIC_RELAXED, __HIP_MEMORY_SCOPE_AGENT);
    const rsrc_t xr = mk(P.x, SHARD);
    const rsrc_t wr = mk(P.w, SHARD);
    const int c = lane & 15, kq = lane >> 4;

    if (b < NPUSH) {
        if (wave >= PW) return;
        const unsigned di = (unsigned)b % 7u, cu = (unsigned)b / 7u;
        const bool cross = di >= 3u;
        const unsigned dst = (cross ? (otcls | (di - 3u)) : (mycls | ((myidx + 1u + di) & 3u))) & 7u;
        char* pw = P.ws[dst];
        const rsrc_t dr = mk(pw, OFF_Y);
        const rsrc_t dor = mk(P.out[dst], OUTB);
        const rsrc_t yr = mk(myws + OFF_Y + (dst & 3u) * (512u * 4096u), 512u * 4096u);
        const gu32* qp = (const gu32*)(myws + OFF_Q + (dst & 3u) * 1024u);
        const unsigned xv = (unsigned)(c * KD + kq * 16);
        const unsigned quart = ((myidx + dst) & 3u) * 128u;
        const unsigned wv = (unsigned)(kq * 16 * NS + c * 4) + quart;
        const int nstat = cross ? NSTAT : 16 * NAB;
        const int nitems = cross ? NSTAT + NQ : 16 * NAB;
        u32x4 d[4], e[4];
        auto fetch = [&](int n, unsigned ent) {
            if (n >= nstat) {
#pragma unroll
                for (int j = 0; j < 4; ++j) e[j] = __builtin_amdgcn_raw_buffer_load_b128(yr, lane * 16 + j * 1024, (ent & 511u) * 4096u, SYS);
            } else if (cross && n < 128) {
                const unsigned so = (unsigned)(n & 63) * 64u * NS + (unsigned)(n >> 6) * 64u;
#pragma unroll
                for (int i = 0; i < 16; ++i) e[i >> 2][i & 3] = __builtin_amdgcn_raw_buffer_load_b32(wr, wv + i * NS, so, 0);
            } else {
                const int m = cross ? n - 128 : n + 16 * NGB;
                const unsigned so = (unsigned)(m >> 4) * 16u * KD + (unsigned)(m & 15) * 256u;
#pragma unroll
                for (int j = 0; j < 4; ++j) e[j] = __builtin_amdgcn_raw_buffer_load_b128(xr, xv + j * 64, so, 0);
            }
        };
        auto settle = [&](int n) {
            if (cross && n < 128) {
                unsigned r[16];
#pragma unroll
                for (int i = 0; i < 16; ++i) r[i] = e[i >> 2][i & 3];
                transpose16x4(r, d);
            } else {
#pragma unroll
                for (int j = 0; j < 4; ++j) d[j] = e[j];
            }
        };
        auto qslot = [&](int n) { return qp + ((unsigned)(n - NSTAT) & 255u); };
        int n = wave * 8 + (int)cu, n1 = n + NPW;
        fetch(n, 0);
        const unsigned call = rfl(cntv) + 1;
        const unsigned ctag = call & 0x7fffffu;
        const unsigned pbase = (call & 1u) * PAR_BYTES;
        gu32* tick = (gu32*)(myws + OFF_QT + (8u + di) * 128u);
        const unsigned tbase = (call - 1u) * (unsigned)(nitems - NPW) - 2u * NPW;
        const unsigned long long t0 = rt();
        unsigned spins = 0;
        unsigned ent = 0, ent1 = 0;
        bool dead = false;
        settle(n);
        STAMP(1);
        while (n < nitems) {
            const bool pre = n1 < nstat || (n1 < nitems && (ent1 >> 9) == ctag);
            if (pre) fetch(n1, ent1);
            unsigned tk;
            asm volatile("s_mov_b64 exec, 1\n\tglobal_atomic_add %0, %1, %2, off sc0\n\ts_mov_b64 exec, -1"
                         : "=&v"(tk) : "v"(tick), "v"(n1 < nitems ? 1u : 0u) : "memory");
            __builtin_amdgcn_sched_barrier(0);
            unsigned fo = 0;
            if (n >= nstat) {
                const unsigned oo = ((ent & 255u) * 16u * NS + quart + ((ent >> 8) & 1u) * 64u) * 4u + (unsigned)(kq * NS * 4 + c * 16);
                if (!dead) {
#pragma unroll
                    for (int j = 0; j < 4; ++j) __builtin_amdgcn_raw_buffer_store_b128(d[j], dor, oo + j * 4 * NS * 4, 0, SYS);
                }
            } else {
                unsigned so;
                if (cross && n < 128) {
                    const unsigned u = myidx * 128u + (unsigned)n;
                    so = pbase + OFF_W + u * 4096u;
                    fo = OFF_WF + (u >> 3) * 128u + (u & 7u) * 4u;
                } else {
                    const unsigned m = (unsigned)(cross ? n - 128 : n + 16 * NGB);
                    so = pbase + rank * SHARD + m * 4096u;
                    fo = OFF_XF + (rank * 256u + (m >> 1)) * 128u + (m & 1u) * 4u;
                }
#pragma unroll
                for (int j = 0; j < 4; ++j) __builtin_amdgcn_raw_buffer_store_b128(d[j], dr, so + lane * 16 + j * 1024, 0, SYS);
            }
            unsigned entv = 0;
            if (n1 + 63 >= nstat && n1 < nitems && cross)
                asm volatile("global_atomic_add %0, %1, %2, off sc0" : "+v"(entv) : "v"(qslot(n1 + lane)), "v"(0u) : "memory");
            asm volatile("s_waitcnt vmcnt(0)" : "+v"(e[0]), "+v"(e[1]), "+v"(e[2]), "+v"(e[3]), "+v"(entv), "+v"(tk) :: "memory");
            if (n < nstat && lane == 0) __hip_atomic_store((gu32*)(pw + fo), call, __ATOMIC_RELAXED, __HIP_MEMORY_SCOPE_SYSTEM);
            if (!pre && n1 < nitems) {
                ent1 = rfl(entv);
                while ((ent1 >> 9) != ctag && !dead) {
                    dead = expired(spins, t0, tmo, call, lane);
                    unsigned ev;
                    asm volatile("s_mov_b64 exec, 1\n\tglobal_atomic_add %0, %1, %2, off sc0\n\ts_mov_b64 exec, -1\n\ts_waitcnt vmcnt(0)"
                                 : "=&v"(ev) : "v"(qslot(n1)), "v"(0u) : "memory");
                    ent1 = rfl(ev);
                }
                asm volatile("" ::: "memory");
                fetch(n1, ent1);
                asm volatile("" : "+v"(e[0]), "+v"(e[1]), "+v"(e[2]), "+v"(e[3]));
            }
            settle(n1);
            ent = ent1;
            n = n1;
            if (n1 < nitems) n1 = (int)((rfl(tk) - tbase) & 0x7fffffffu);
            ent1 = (unsigned)(n1 - n) < 64u ? __builtin_amdgcn_readlane(entv, (n1 - n) & 63) : 0u;
        }
        if (lane == 0) {
            if (cross && !dead)
                __hip_atomic_store((gu32*)(pw + OFF_DF + (myidx * 64u + cu * PW + (unsigned)wave) * 4u), call,
                                   __ATOMIC_RELAXED, __HIP_MEMORY_SCOPE_SYSTEM);
            __hip_atomic_store(cntp, call, __ATOMIC_RELAXED, __HIP_MEMORY_SCOPE_AGENT);
        }
        return;
    }
    if (b < 64) {
        if (b != NPUSH || wave != 0) return;
        const unsigned call = rfl(cntv) + 1;
        if (lane == 0) __hip_atomic_store(cntp, call, __ATOMIC_RELAXED, __HIP_MEMORY_SCOPE_AGENT);
        const gu32* fp = (const gu32*)(myws + OFF_DF + lane * 4);
        const bool need = lane < 8 * PW + 2 * RS;
        const unsigned long long t0 = rt();
        unsigned spins = 0;
        for (;;) {
            bool ok = true;
#pragma unroll
            for (int i = 0; i < 4; ++i) ok &= __hip_atomic_load(fp + i * 64, __ATOMIC_RELAXED, __HIP_MEMORY_SCOPE_SYSTEM) >= call;
            ok |= !need;
            if (__all(ok)) break;
            if (expired(spins, t0, tmo, call, lane)) {
                if (lane == 0) __hip_atomic_store((gu32*)(P.out[rank] + (size_t)rank * MS * NS * 4), 0x7fc00000u,
                                                  __ATOMIC_RELAXED, __HIP_MEMORY_SCOPE_SYSTEM);
                break;
            }
        }
        STAMP(1);
        return;
    }

    const int xcd = b & 7, slot = (b - 64) >> 3;
    const int rs = 3 * (xcd >> 1) + (slot >> 3);
    const bool remote = (slot & 4) != 0;
    const unsigned dj = (unsigned)slot & 3u;
    const unsigned cblk = (unsigned)xcd & 1u;
    const unsigned owner = remote ? (otcls | dj) : rank;
    const unsigned col0 = remote ? ((myidx + dj) & 3u) * 128u + cblk * 64u : (cblk * 4u + dj) * 64u;
    LAS int* tcnt = (LAS int*)smem + MAXT * 16 * TSTR;

    u32x4 wf[8][4];
    if (!remote) {
        const unsigned wv = (unsigned)(kq * 16 * NS + c * 4);
#pragma unroll
        for (int s = 0; s < 8; ++s) {
            unsigned r[16];
            const unsigned so = (unsigned)(wave * 8 + s) * 64u * NS + col0;
#pragma unroll
            for (int i = 0; i < 16; ++i) r[i] = __builtin_amdgcn_raw_buffer_load_b32(wr, wv + i * NS, so, 0);
            transpose16x4(r, wf[s]);
        }
    }
    for (int i = threadIdx.x; i < MAXT * 16 * TSTR + 32; i += NWAVES * 64) smem[i] = 0;
    __syncthreads();
    const unsigned call = rfl(cntv) + 1;
    if (lane == 0) __hip_atomic_store(cntp, call, __ATOMIC_RELAXED, __HIP_MEMORY_SCOPE_AGENT);
    const float scale = P.sx[0] * P.sw[0];
    const unsigned pbase = (call & 1u) * PAR_BYTES;
    const rsrc_t sr = mk(myws + pbase, PAR_BYTES);
    const rsrc_t orr = mk(P.out[owner & 7u], OUTB);
    const rsrc_t yr = mk(myws + OFF_Y + dj * (512u * 4096u), 512u * 4096u);
    const int tlast = (3 * NAB - 1 - rs) / RS;
    const unsigned long long t0 = rt();
    unsigned spins = 0;
    bool dead = false;

    unsigned tsrc, trb;
    bool tvalid;
    {
        const unsigned idx = (unsigned)rs + RS * (unsigned)lane;
        if (idx < 3u * NAB) { tsrc = mycls | ((myidx + 1u + idx % 3u) & 3u); trb = NGB + idx / 3u; tvalid = true; }
        else if (idx < 4u * NAB) { tsrc = rank; trb = NGB + idx - 3u * NAB; tvalid = true; }
        else if (idx < 128u) { const unsigned u = idx - 4u * NAB; tsrc = otcls | (u & 3u); trb = u >> 2; tvalid = tsrc != owner; }
        else { tsrc = rank; trb = (idx - 128u) & 31u; tvalid = !remote && idx < NLIST; }
    }
    const unsigned tdesc = tsrc * 32u + trb;
    const unsigned long long validmask = __ballot(tvalid);
    const unsigned long long ownmask = __ballot(tvalid && tsrc == rank);
    const gu64* flagp = (tvalid && tsrc != rank) ? (const gu64*)(myws + OFF_XF + (tdesc * 8u + (unsigned)wave) * 128u)
                                                 : (const gu64*)tmo;
    const unsigned vown = (unsigned)(c * KD + kq * 16), vrem = (unsigned)(lane * 16);

    unsigned pend = (unsigned)validmask;
    auto refill = [&](unsigned ready, u32x4 (&a)[8]) {
        const unsigned todo = ready & pend;
        const int t = todo ? __builtin_ctz(todo) : -1;
        pend &= ~(todo & (0u - todo));
        const unsigned desc = __builtin_amdgcn_readlane(tdesc, t & 63);
        const bool own = (desc >> 5) == rank;
        const rsrc_t r = mk(own ? (const char*)P.x : myws + pbase, todo ? (own ? SHARD : OFF_W) : 0u);
        const unsigned so = own ? (desc & 31u) * 65536u + (unsigned)wave * 512u : desc * 65536u + (unsigned)wave * 8192u;
        const unsigned step = own ? 64u : 1024u;
        const unsigned vo = own ? vown : vrem;
#pragma unroll
        for (int s = 0; s < 8; ++s) a[s] = __builtin_amdgcn_raw_buffer_load_b128(r, vo, so + s * step, SYS);
        return t;
    };
    auto arrive = [&](int t) {
        asm volatile("s_waitcnt lgkmcnt(0)" ::: "memory");
        if (lane == 0 && __hip_atomic_fetch_add(tcnt + t, 1, __ATOMIC_RELAXED, __HIP_MEMORY_SCOPE_WORKGROUP) == NWAVES - 1)
            __hip_atomic_fetch_or(tcnt + 16, 1 << t, __ATOMIC_RELAXED, __HIP_MEMORY_SCOPE_WORKGROUP);
    };
    auto store_tile = [&](int t) {
        const unsigned desc = __builtin_amdgcn_readlane(tdesc, t);
        const bool bad = __hip_atomic_load(tcnt + 17, __ATOMIC_RELAXED, __HIP_MEMORY_SCOPE_WORKGROUP) != 0;
        const float sc = bad ? __builtin_nanf("") : scale;
        const LAS int* tp = (const LAS int*)smem + t * 16 * TSTR + kq * TSTR + c * 4;
        f32x4 y[4];
#pragma unroll
        for (int i = 0; i < 4; ++i)
#pragma unroll
            for (int e = 0; e < 4; ++e) y[i][e] = (float)tp[i * 4 * TSTR + e] * sc;
        if (!remote || t == tlast) {
            const unsigned oo = (desc * 16u * NS + col0) * 4u + (unsigned)(kq * NS * 4 + c * 16);
#pragma unroll
            for (int i = 0; i < 4; ++i) {
                if (OWN_AUX == SYS || remote)
                    __builtin_amdgcn_raw_buffer_store_b128(__builtin_bit_cast(u32x4, y[i]), orr, oo + i * 4 * NS * 4, 0, SYS);
                else
                    __builtin_amdgcn_raw_buffer_store_b128(__builtin_bit_cast(u32x4, y[i]), orr, oo + i * 4 * NS * 4, 0, OWN_AUX);
            }
            if (remote) {
                asm volatile("s_waitcnt vmcnt(0)" ::: "memory");
                if (lane == 0)
                    __hip_atomic_store((gu32*)(P.ws[owner & 7u] + OFF_DF + (myidx * 64u + 8u * PW + cblk * RS + (unsigned)rs) * 4u), call,
                                       __ATOMIC_RELAXED, __HIP_MEMORY_SCOPE_SYSTEM);
            }
        } else {
            unsigned ticket = 0;
            asm volatile("s_mov_b64 exec, 1\n\tglobal_atomic_add %0, %1, %2, off sc0\n\ts_mov_b64 exec, -1"
                         : "=&v"(ticket) : "v"(myws + OFF_QT + dj * 128u), "v"(1u) : "memory");
            const unsigned id = cblk * 256u + desc;
#pragma unroll
            for (int i = 0; i < 4; ++i)
                __builtin_amdgcn_raw_buffer_store_b128(__builtin_bit_cast(u32x4, y[i]), yr, id * 4096u + i * 1024 + lane * 16, 0, SYS);
            asm volatile("s_waitcnt vmcnt(0)" : "+v"(ticket) :: "memory");
            if (lane == 0)
                asm volatile("global_atomic_swap %0, %1, off" :: "v"(myws + OFF_Q + dj * 1024u + ((ticket - (call - 1u) * NQ) & 255u) * 4u),
                             "v"(((call & 0x7fffffu) << 9) | id) : "memory");
        }
        STAMP(8 + t);
    };
    u32x4 a0[8], a1[8];
    int t0i = refill((unsigned)ownmask, a0);
    int t1i = refill((unsigned)ownmask, a1);
    if (remote) {
        const unsigned u0 = (dj * 2u + cblk) * 64u + (unsigned)wave * 8u;
        const gu32* fp = (const gu32*)(myws + OFF_WF + u0 * 16u + (lane & 7) * 4u);
        unsigned have = 0;
#pragma unroll
        for (int s = 0; s < 8; ++s) {
            while (((have >> s) & 1u) == 0u && !dead) {
                have = (unsigned)__ballot(__hip_atomic_load(fp, __ATOMIC_RELAXED, __HIP_MEMORY_SCOPE_SYSTEM) >= call);
                if (((have >> s) & 1u) == 0u) dead = expired(spins, t0, tmo, call, lane);
            }
            asm volatile("" ::: "memory");
#pragma unroll
            for (int e = 0; e < 4; ++e)
                wf[s][e] = __builtin_amdgcn_raw_buffer_load_b128(sr, lane * 16 + e * 1024, OFF_W + (u0 + s) * 4096u, SYS);
        }
#pragma unroll
        for (int s = 0; s < 8; s += 2)
            asm volatile("" : "+v"(wf[s][0]), "+v"(wf[s][1]), "+v"(wf[s][2]), "+v"(wf[s][3]),
                              "+v"(wf[s + 1][0]), "+v"(wf[s + 1][1]), "+v"(wf[s + 1][2]), "+v"(wf[s + 1][3]));
    }
    STAMP(1);

    unsigned long long fv = __hip_atomic_load(flagp, __ATOMIC_RELAXED, __HIP_MEMORY_SCOPE_SYSTEM);
    asm volatile("" : "+v"(fv));
    bool done = false;
    auto half = [&](u32x4 (&a)[8], int& t, int other) {
        i32x4 acc[4];
        const int tc = t;
        if (tc >= 0) {
#pragma unroll
            for (int e = 0; e < 4; ++e) acc[e] = (i32x4){0, 0, 0, 0};
#pragma unroll
            for (int s = 0; s < 8; ++s)
#pragma unroll
                for (int e = 0; e < 4; ++e)
                    acc[e] = __builtin_amdgcn_mfma_i32_16x16x64_i8(__builtin_bit_cast(i32x4, wf[s][e]), __builtin_bit_cast(i32x4, a[s]), acc[e], 0, 0, 0);
        }
        const unsigned rdy = (unsigned)__ballot((unsigned)fv >= call && (unsigned)(fv >> 32) >= call) | (unsigned)ownmask;
        if (__builtin_amdgcn_readlane((unsigned)fv, 63) == call) dead = true;
        asm volatile("" ::: "memory");
        fv = __hip_atomic_load(flagp, __ATOMIC_RELAXED, __HIP_MEMORY_SCOPE_SYSTEM);
        t = refill(dead ? 0u : rdy, a);
        __builtin_amdgcn_sched_barrier(0);
        if (tc >= 0) {
            LAS int* tp = (LAS int*)smem + tc * 16 * TSTR + c * TSTR + kq * 16;
#pragma unroll
            for (int r = 0; r < 4; ++r)
#pragma unroll
                for (int e = 0; e < 4; ++e)
                    __hip_atomic_fetch_add(tp + r * 4 + e, acc[e][r], __ATOMIC_RELAXED, __HIP_MEMORY_SCOPE_WORKGROUP);
            arrive(tc);
            STAMP(25 + 3 * tc);
        }
        if (t >= 0 || other >= 0) return;
        for (;;) {
            if (dead && pend != 0u) {
                if (lane == 0) __hip_atomic_store(tcnt + 17, 1, __ATOMIC_RELAXED, __HIP_MEMORY_SCOPE_WORKGROUP);
                for (; pend != 0u; pend &= pend - 1u) arrive(__builtin_ctz(pend));
            }
            const unsigned fin = rfl(__hip_atomic_load(tcnt + 16, __ATOMIC_RELAXED, __HIP_MEMORY_SCOPE_WORKGROUP));
            const unsigned clm = rfl(__hip_atomic_load(tcnt + 18, __ATOMIC_RELAXED, __HIP_MEMORY_SCOPE_WORKGROUP));
            if ((fin & ~clm) != 0u) {
                const int ts = __builtin_ctz(fin & ~clm);
                int old = 0;
                if (lane == 0) old = __hip_atomic_fetch_or(tcnt + 18, 1 << ts, __ATOMIC_RELAXED, __HIP_MEMORY_SCOPE_WORKGROUP);
                if (((rfl(old) >> ts) & 1u) != 0u) continue;
                asm volatile("" : "+v"(fv) :: "memory");
                store_tile(ts);
            } else if (pend == 0u) {
                if (clm == (unsigned)validmask) { done = true; return; }
                __builtin_amdgcn_s_sleep(1);
                continue;
            }
            if (((unsigned)__ballot((unsigned)fv >= call && (unsigned)(fv >> 32) >= call) & pend) != 0u) return;
            if (pend == 0u) continue;
            if (expired(spins, t0, tmo, call, lane)) dead = true;
            fv = __hip_atomic_load(flagp, __ATOMIC_RELAXED, __HIP_MEMORY_SCOPE_SYSTEM);
        }
    };
    do {
        half(a0, t0i, t1i);
        half(a1, t1i, t0i);
    } while (!done);
    STAMP(2);
    STAMP(3);
}

extern "C" void kernel_launch(int rank, int nranks, ncclComm_t comm,
                              hipStream_t stream,
                              void* const* d_in, const int* in_sizes, int n_in,
                              void* d_out, int out_size,
                              const int* in_dtypes, int out_dtype,
                              void* const* d_out_all,
                              void* const* d_ws_all, size_t ws_bytes) {
    if (nranks != NR || (unsigned)rank >= (unsigned)NR || n_in != 4 || in_sizes[0] != MS * KD || in_sizes[1] != KD * NS
        || out_size != NR * MS * NS || ws_bytes < WS_NEED) {
        fprintf(stderr, "kernel_launch: unexpected shapes\n");
        return;
    }
    Params P;
    P.x = (const signed char*)d_in[0];
    P.w = (const signed char*)d_in[1];
    P.sx = (const float*)d_in[2];
    P.sw = (const float*)d_in[3];
    for (int i = 0; i < NR; ++i) { P.ws[i] = (char*)d_ws_all[i]; P.out[i] = (char*)d_out_all[i]; }
    P.rank = rank;
    P.pad = 0;
    ag_gemm_kernel<<<GRID, NWAVES * 64, 0, stream>>>(P);
    HIP_CHECK(hipGetLastError());
}
```

```cpp
#include <hip/hip_runtime.h>
#include <rccl/rccl.h>
#include <cstdio>
#include <cstdint>

#define HIP_CHECK(expr) do { hipError_t _e = (expr); if (_e != hipSuccess) { \
    fprintf(stderr, "HIP error %s at %s:%d\n", hipGetErrorString(_e), __FILE__, __LINE__); \
    return; } } while (0)

#define GAS __attribute__((address_space(1)))
#define LAS __attribute__((address_space(3)))
typedef unsigned u32x4 __attribute__((ext_vector_type(4)));
typedef int i32x4 __attribute__((ext_vector_type(4)));
typedef float f32x4 __attribute__((ext_vector_type(4)));
typedef GAS unsigned gu32;
typedef GAS unsigned long long gu64;
typedef __amdgpu_buffer_rsrc_t rsrc_t;

#define NGB 6
#define PW 4
#define PWI 4
#define TAU32 227
#define T_FIRST 40
#define CREDIT 60
#define ND 2
#define AHEAD 3
#define POLL_SLEEP 1
#define OWN_AUX 17
constexpr int NR = 8;
constexpr int MS = 512;
constexpr int KD = 4096;
constexpr int NS = 512;
constexpr int NAB = 32 - NGB;
constexpr int GRID = 256;
constexpr int NWAVES = 8;
constexpr int NPUSH = 56;
constexpr int RS = 12;
constexpr int NLIST = 128 + NGB;
constexpr int MAXT = (NLIST + RS - 1) / RS;
constexpr int TSTR = 65;
constexpr unsigned SHARD = MS * KD;
constexpr unsigned OUTB = NR * MS * NS * 4;
constexpr unsigned PAR_BYTES = (NR + 1) * SHARD;
constexpr unsigned OFF_W = NR * SHARD;
constexpr unsigned OFF_Y = 2 * PAR_BYTES;
constexpr unsigned OFF_XF = OFF_Y + 8 * 256 * 4096;
constexpr unsigned OFF_WF = OFF_XF + NR * 32 * 8 * 128;
constexpr unsigned OFF_DF = OFF_WF + 8192;
constexpr unsigned OFF_Q = OFF_DF + 4096;
constexpr unsigned OFF_QT = OFF_Q + 4096;
constexpr unsigned OFF_CNT = OFF_QT + 4096;
constexpr unsigned OFF_TMO = OFF_CNT + GRID * 128;
constexpr unsigned OFF_DBG = OFF_TMO + 4096;
constexpr unsigned WS_NEED = OFF_DBG + GRID * NWAVES * 64 * 8;
constexpr int NQ = 2 * (128 - NGB - 3 * ND);
constexpr int NSTAT = 128 + 16 * NGB;
static_assert(8 * PW + 2 * RS <= 64 && NQ <= 256, "flag / queue layout");
constexpr unsigned long long DEADLINE_TICKS = 50000000ull;
constexpr int SYS = 17;

struct Params {
    const signed char* x;
    const signed char* w;
    const float* sx;
    const float* sw;
    char* ws[NR];
    char* out[NR];
    int rank;
    int pad;
};

__device__ __forceinline__ unsigned long long rt() { return __builtin_amdgcn_s_memrealtime(); }
__device__ __forceinline__ rsrc_t mk(const void* p, unsigned bytes) {
    return __builtin_amdgcn_make_buffer_rsrc((void*)p, 0, (int)bytes, 0x00020000);
}
__device__ __forceinline__ unsigned rfl(unsigned v) { return __builtin_amdgcn_readfirstlane(v); }

#define STAMP(i) do { } while (0)
#define LSTAMP(i) do { } while (0)

__device__ __forceinline__ void transpose16x4(const unsigned (&r)[16], u32x4 (&o)[4]) {
#pragma unroll
    for (int q = 0; q < 4; ++q) {
        const unsigned r0 = r[4 * q], r1 = r[4 * q + 1], r2 = r[4 * q + 2], r3 = r[4 * q + 3];
        const unsigned lo01 = __builtin_amdgcn_perm(r1, r0, 0x05010400);
        const unsigned hi01 = __builtin_amdgcn_perm(r1, r0, 0x07030602);
        const unsigned lo23 = __builtin_amdgcn_perm(r3, r2, 0x05010400);
        const unsigned hi23 = __builtin_amdgcn_perm(r3, r2, 0x07030602);
        o[0][q] = __builtin_amdgcn_perm(lo23, lo01, 0x05040100);
        o[1][q] = __builtin_amdgcn_perm(lo23, lo01, 0x07060302);
        o[2][q] = __builtin_amdgcn_perm(hi23, hi01, 0x05040100);
        o[3][q] = __builtin_amdgcn_perm(hi23, hi01, 0x07060302);
    }
}

__device__ __forceinline__ bool expired(unsigned& spins, unsigned long long t0, gu32* tmo, unsigned call, int lane) {
    __builtin_amdgcn_s_sleep(POLL_SLEEP);
    spins = rfl(spins + 1u);
    if ((spins & 63u) != 0u) return false;
    const bool late = rfl(__hip_atomic_load(tmo, __ATOMIC_RELAXED, __HIP_MEMORY_SCOPE_AGENT)) == call
                      || rfl(rt() - t0 > DEADLINE_TICKS ? 1u : 0u) != 0u;
    if (late) __hip_atomic_store(tmo, call, __ATOMIC_RELAXED, __HIP_MEMORY_SCOPE_AGENT);
    return late;
}

__global__ void __launch_bounds__(NWAVES * 64, 2) ag_gemm_kernel(Params P) {
    __shared__ __attribute__((aligned(16))) int smem[MAXT * 16 * TSTR + 32];
    const int b = blockIdx.x;
    const int lane = threadIdx.x & 63;
    const int wave = rfl(threadIdx.x >> 6);
    const unsigned rank = (unsigned)P.rank & 7u;
    const unsigned myidx = rank & 3u, mycls = rank & 4u, otcls = mycls ^ 4u;
    char* myws = P.ws[rank];
    gu32* cntp = (gu32*)(myws + OFF_CNT + b * 128 + wave * 4);
    gu32* tmo = (gu32*)(myws + OFF_TMO);
    LSTAMP(0);
    const unsigned cntv = __hip_atomic_load(cntp, __ATOMIC_RELAXED, __HIP_MEMORY_SCOPE_AGENT);
    const rsrc_t xr = mk(P.x, SHARD);
    const rsrc_t wr = mk(P.w, SHARD);
    const int c = lane & 15, kq = lane >> 4;

    if (b < NPUSH) {
        const unsigned di = (unsigned)b % 7u, cu = (unsigned)b / 7u;
        const bool cross = di >= 3u;
        if (wave >= (cross ? PW : PWI)) return;
        const int NPW = cross ? 8 * PW : 8 * PWI;
        const unsigned dst = (cross ? (otcls | (di - 3u)) : (mycls | ((myidx + 1u + di) & 3u))) & 7u;
        char* pw = P.ws[dst];
        const rsrc_t dr = mk(pw, OFF_Y);
        const rsrc_t dor = mk(P.out[dst], OUTB);
        const rsrc_t yr = mk(myws + OFF_Y + (dst & 3u) * (512u * 4096u), 512u * 4096u);
        const gu32* qp = (const gu32*)(myws + OFF_Q + (dst & 3u) * 1024u);
        const unsigned xv = (unsigned)(c * KD + kq * 16);
        const unsigned quart = ((myidx + dst) & 3u) * 128u;
        const unsigned wv = (unsigned)(kq * 16 * NS + c * 4) + quart;
        const int nstat = cross ? NSTAT : 16 * NAB;
        const int nitems = cross ? NSTAT + NQ : 16 * NAB;
        u32x4 d[4], e[4];
        auto fetch = [&](int n, unsigned ent) {
            if (n >= nstat) {
#pragma unroll
                for (int j = 0; j < 4; ++j) e[j] = __builtin_amdgcn_raw_buffer_load_b128(yr, lane * 16 + j * 1024, (ent & 511u) * 4096u, SYS);
            } else if (cross && n < 128) {
                const unsigned so = (unsigned)(n & 63) * 64u * NS + (unsigned)(n >> 6) * 64u;
#pragma unroll
                for (int i = 0; i < 16; ++i) e[i >> 2][i & 3] = __builtin_amdgcn_raw_buffer_load_b32(wr, wv + i * NS, so, 0);
            } else {
                const int m = cross ? n - 128 : n + 16 * NGB;
                const unsigned so = (unsigned)(m >> 4) * 16u * KD + (unsigned)(m & 15) * 256u;
#pragma unroll
                for (int j = 0; j < 4; ++j) e[j] = __builtin_amdgcn_raw_buffer_load_b128(xr, xv + j * 64, so, 0);
            }
        };
        auto settle = [&](int n) {
            if (cross && n < 128) {
                unsigned r[16];
#pragma unroll
                for (int i = 0; i < 16; ++i) r[i] = e[i >> 2][i & 3];
                transpose16x4(r, d);
            } else {
#pragma unroll
                for (int j = 0; j < 4; ++j) d[j] = e[j];
            }
        };
        auto qslot = [&](int n) { return qp + ((unsigned)(n - NSTAT) & 255u); };
        int n = wave * 8 + (int)cu, n1 = n + NPW;
        const unsigned period = (unsigned)(TAU32 * NPW / 32);
        unsigned slotT = (unsigned)rt() + T_FIRST + (unsigned)(n * TAU32 / 32);
        fetch(n, 0);
        const unsigned call = rfl(cntv) + 1;
        const unsigned ctag = call & 0x7fffffu;
        const unsigned pbase = (call & 1u) * PAR_BYTES;
        gu32* tick = (gu32*)(myws + OFF_QT + (8u + di) * 128u);
        const unsigned tbase = (call - 1u) * (unsigned)(nitems - NPW) - 2u * (unsigned)NPW;
        const unsigned long long t0 = rt();
        unsigned spins = 0;
        unsigned ent = 0, ent1 = 0;
        bool dead = false;
        settle(n);
        STAMP(1);
        while (n < nitems) {
            {
                unsigned now = (unsigned)rt();
                while ((int)(slotT - now) > 0) {
                    if ((int)(slotT - now) > 8) __builtin_amdgcn_s_sleep(1);
                    now = (unsigned)rt();
                }
                if ((int)(now - slotT) > CREDIT) slotT = now - CREDIT;
                slotT += period;
            }
            const bool pre = n1 < nstat || (n1 < nitems && (ent1 >> 9) == ctag);
            if (pre) fetch(n1, ent1);
            unsigned tk;
            asm volatile("s_mov_b64 exec, 1\n\tglobal_atomic_add %0, %1, %2, off sc0\n\ts_mov_b64 exec, -1"
                         : "=&v"(tk) : "v"(tick), "v"(n1 < nitems ? 1u : 0u) : "memory");
            __builtin_amdgcn_sched_barrier(0);
            unsigned fo = 0;
            if (n >= nstat) {
                const unsigned oo = ((ent & 255u) * 16u * NS + quart + ((ent >> 8) & 1u) * 64u) * 4u + (unsigned)(kq * NS * 4 + c * 16);
                if (!dead) {
#pragma unroll
                    for (int j = 0; j < 4; ++j) __builtin_amdgcn_raw_buffer_store_b128(d[j], dor, oo + j * 4 * NS * 4, 0, SYS);
                }
            } else {
                unsigned so;
                if (cross && n < 128) {
                    const unsigned u = myidx * 128u + (unsigned)n;
                    so = pbase + OFF_W + u * 4096u;
                    fo = OFF_WF + (u >> 3) * 128u + (u & 7u) * 4u;
                } else {
                    const unsigned m = (unsigned)(cross ? n - 128 : n + 16 * NGB);
                    so = pbase + rank * SHARD + m * 4096u;
                    fo = OFF_XF + (rank * 256u + (m >> 1)) * 128u + (m & 1u) * 4u;
                }
#pragma unroll
                for (int j = 0; j < 4; ++j) __builtin_amdgcn_raw_buffer_store_b128(d[j], dr, so + lane * 16 + j * 1024, 0, SYS);
            }
            unsigned entv = 0;
            if (n1 + 63 >= nstat && n1 < nitems && cross)
                asm volatile("global_atomic_add %0, %1, %2, off sc0" : "+v"(entv) : "v"(qslot(n1 + lane)), "v"(0u) : "memory");
            asm volatile("s_waitcnt vmcnt(0)" : "+v"(e[0]), "+v"(e[1]), "+v"(e[2]), "+v"(e[3]), "+v"(entv), "+v"(tk) :: "memory");
            if (n < nstat && lane == 0) __hip_atomic_store((gu32*)(pw + fo), call, __ATOMIC_RELAXED, __HIP_MEMORY_SCOPE_SYSTEM);
            if (!pre && n1 < nitems) {
                ent1 = rfl(entv);
                while ((ent1 >> 9) != ctag && !dead) {
                    dead = expired(spins, t0, tmo, call, lane);
                    unsigned ev;
                    asm volatile("s_mov_b64 exec, 1\n\tglobal_atomic_add %0, %1, %2, off sc0\n\ts_mov_b64 exec, -1\n\ts_waitcnt vmcnt(0)"
                                 : "=&v"(ev) : "v"(qslot(n1)), "v"(0u) : "memory");
                    ent1 = rfl(ev);
                }
                asm volatile("" ::: "memory");
                fetch(n1, ent1);
                asm volatile("" : "+v"(e[0]), "+v"(e[1]), "+v"(e[2]), "+v"(e[3]));
            }
            settle(n1);
            ent = ent1;
            n = n1;
            if (n1 < nitems) n1 = (int)((rfl(tk) - tbase) & 0x7fffffffu);
            ent1 = (unsigned)(n1 - n) < 64u ? __builtin_amdgcn_readlane(entv, (n1 - n) & 63) : 0u;
        }
        if (lane == 0) {
            if (cross && !dead)
                __hip_atomic_store((gu32*)(pw + OFF_DF + (myidx * 64u + cu * PW + (unsigned)wave) * 4u), call,
                                   __ATOMIC_RELAXED, __HIP_MEMORY_SCOPE_SYSTEM);
            __hip_atomic_store(cntp, call, __ATOMIC_RELAXED, __HIP_MEMORY_SCOPE_AGENT);
        }
        return;
    }
    if (b < 64) {
        if (b != NPUSH || wave != 0) return;
        const unsigned call = rfl(cntv) + 1;
        if (lane == 0) __hip_atomic_store(cntp, call, __ATOMIC_RELAXED, __HIP_MEMORY_SCOPE_AGENT);
        const gu32* fp = (const gu32*)(myws + OFF_DF + lane * 4);
        const int lrs = (lane - 8 * PW) % RS;
        const bool need = lane < 8 * PW || (lane < 8 * PW + 2 * RS && (lrs + (3 * NAB - 1 - lrs) / RS * RS) / 3 >= NAB - ND);
        const unsigned long long t0 = rt();
        unsigned spins = 0;
        for (;;) {
            bool ok = true;
#pragma unroll
            for (int i = 0; i < 4; ++i) ok &= __hip_atomic_load(fp + i * 64, __ATOMIC_RELAXED, __HIP_MEMORY_SCOPE_SYSTEM) >= call;
            ok |= !need;
            if (__all(ok)) break;
            if (expired(spins, t0, tmo, call, lane)) {
                if (lane == 0) __hip_atomic_store((gu32*)(P.out[rank] + (size_t)rank * MS * NS * 4), 0x7fc00000u,
                                                  __ATOMIC_RELAXED, __HIP_MEMORY_SCOPE_SYSTEM);
                break;
            }
        }
        LSTAMP(1);
        return;
    }

    const int xcd = b & 7, slot = (b - 64) >> 3;
    const int rs = 3 * (xcd >> 1) + (slot >> 3);
    const bool remote = (slot & 4) != 0;
    const unsigned dj = (unsigned)slot & 3u;
    const unsigned cblk = (unsigned)xcd & 1u;
    const unsigned owner = remote ? (otcls | dj) : rank;
    const unsigned col0 = remote ? ((myidx + dj) & 3u) * 128u + cblk * 64u : (cblk * 4u + dj) * 64u;
    LAS int* tcnt = (LAS int*)smem + MAXT * 16 * TSTR;

    u32x4 wf[8][4];
    if (!remote) {
        const unsigned wv = (unsigned)(kq * 16 * NS + c * 4);
#pragma unroll
        for (int s = 0; s < 8; ++s) {
            unsigned r[16];
            const unsigned so = (unsigned)(wave * 8 + s) * 64u * NS + col0;
#pragma unroll
            for (int i = 0; i < 16; ++i) r[i] = __builtin_amdgcn_raw_buffer_load_b32(wr, wv + i * NS, so, 0);
            transpose16x4(r, wf[s]);
        }
    }
    for (int i = threadIdx.x; i < MAXT * 16 * TSTR + 32; i += NWAVES * 64) smem[i] = 0;
    __syncthreads();
    const unsigned call = rfl(cntv) + 1;
    if (lane == 0) __hip_atomic_store(cntp, call, __ATOMIC_RELAXED, __HIP_MEMORY_SCOPE_AGENT);
    const float scale = P.sx[0] * P.sw[0];
    const unsigned pbase = (call & 1u) * PAR_BYTES;
    const rsrc_t sr = mk(myws + pbase, PAR_BYTES);
    const rsrc_t orr = mk(P.out[owner & 7u], OUTB);
    const rsrc_t yr = mk(myws + OFF_Y + dj * (512u * 4096u), 512u * 4096u);
    int tlast = (3 * NAB - 1 - rs) / RS;
    if ((rs + tlast * RS) / 3 < NAB - ND) tlast = -1;
    const unsigned long long t0 = rt();
    unsigned spins = 0;
    bool dead = false;

    unsigned tsrc, trb;
    bool tvalid;
    {
        const unsigned idx = (unsigned)rs + RS * (unsigned)lane;
        if (idx < 3u * NAB) { tsrc = mycls | ((myidx + 1u + idx % 3u) & 3u); trb = NGB + idx / 3u; tvalid = true; }
        else if (idx < 4u * NAB) { tsrc = rank; trb = NGB + idx - 3u * NAB; tvalid = true; }
        else if (idx < 128u) { const unsigned u = idx - 4u * NAB; tsrc = otcls | (u & 3u); trb = u >> 2; tvalid = tsrc != owner; }
        else { tsrc = rank; trb = (idx - 128u) & 31u; tvalid = !remote && idx < NLIST; }
    }
    const unsigned tdesc = tsrc * 32u + trb;
    const unsigned long long validmask = __ballot(tvalid);
    const unsigned long long ownmask = __ballot(tvalid && tsrc == rank);
    const gu64* flagp = (tvalid && tsrc != rank) ? (const gu64*)(myws + OFF_XF + (tdesc * 8u + (unsigned)wave) * 128u)
                                                 : (const gu64*)tmo;
    const unsigned vown = (unsigned)(c * KD + kq * 16), vrem = (unsigned)(lane * 16);

    unsigned pend = (unsigned)validmask;
    auto refill = [&](unsigned ready, u32x4 (&a)[8]) {
        const unsigned todo = ready & pend;
        const int t = todo ? __builtin_ctz(todo) : -1;
        pend &= ~(todo & (0u - todo));
        const unsigned desc = __builtin_amdgcn_readlane(tdesc, t & 63);
        const bool own = (desc >> 5) == rank;
        const rsrc_t r = mk(own ? (const char*)P.x : myws + pbase, todo ? (own ? SHARD : OFF_W) : 0u);
        const unsigned so = own ? (desc & 31u) * 65536u + (unsigned)wave * 512u : desc * 65536u + (unsigned)wave * 8192u;
        const unsigned step = own ? 64u : 1024u;
        const unsigned vo = own ? vown : vrem;
#pragma unroll
        for (int s = 0; s < 8; ++s) a[s] = __builtin_amdgcn_raw_buffer_load_b128(r, vo, so + s * step, SYS);
        return t;
    };
    auto arrive = [&](int t) {
        asm volatile("s_waitcnt lgkmcnt(0)" ::: "memory");
        int old = 0;
        if (lane == 0) old = __hip_atomic_fetch_add(tcnt + t, 1, __ATOMIC_RELAXED, __HIP_MEMORY_SCOPE_WORKGROUP);
        old = rfl(old);
        if (old == NWAVES - 1 && lane == 0) __hip_atomic_fetch_or(tcnt + 16, 1 << t, __ATOMIC_RELAXED, __HIP_MEMORY_SCOPE_WORKGROUP);
        return old;
    };
    auto store_tile = [&](int t) {
        const unsigned desc = __builtin_amdgcn_readlane(tdesc, t);
        const bool bad = __hip_atomic_load(tcnt + 17, __ATOMIC_RELAXED, __HIP_MEMORY_SCOPE_WORKGROUP) != 0;
        const float sc = bad ? __builtin_nanf("") : scale;
        const LAS int* tp = (const LAS int*)smem + t * 16 * TSTR + kq * TSTR + c * 4;
        f32x4 y[4];
#pragma unroll
        for (int i = 0; i < 4; ++i)
#pragma unroll
            for (int e = 0; e < 4; ++e) y[i][e] = (float)tp[i * 4 * TSTR + e] * sc;
        if (!remote || t == tlast) {
            const unsigned oo = (desc * 16u * NS + col0) * 4u + (unsigned)(kq * NS * 4 + c * 16);
#pragma unroll
            for (int i = 0; i < 4; ++i) {
                if (OWN_AUX == SYS || remote)
                    __builtin_amdgcn_raw_buffer_store_b128(__builtin_bit_cast(u32x4, y[i]), orr, oo + i * 4 * NS * 4, 0, SYS);
                else
                    __builtin_amdgcn_raw_buffer_store_b128(__builtin_bit_cast(u32x4, y[i]), orr, oo + i * 4 * NS * 4, 0, OWN_AUX);
            }
            if (remote) {
                asm volatile("s_waitcnt vmcnt(0)" ::: "memory");
                if (lane == 0)
                    __hip_atomic_store((gu32*)(P.ws[owner & 7u] + OFF_DF + (myidx * 64u + 8u * PW + cblk * RS + (unsigned)rs) * 4u), call,
                                       __ATOMIC_RELAXED, __HIP_MEMORY_SCOPE_SYSTEM);
            }
        } else {
            unsigned ticket = 0;
            asm volatile("s_mov_b64 exec, 1\n\tglobal_atomic_add %0, %1, %2, off sc0\n\ts_mov_b64 exec, -1"
                         : "=&v"(ticket) : "v"(myws + OFF_QT + dj * 128u), "v"(1u) : "memory");
            const unsigned id = cblk * 256u + desc;
#pragma unroll
            for (int i = 0; i < 4; ++i)
                __builtin_amdgcn_raw_buffer_store_b128(__builtin_bit_cast(u32x4, y[i]), yr, id * 4096u + i * 1024 + lane * 16, 0, SYS);
            asm volatile("s_waitcnt vmcnt(0)" : "+v"(ticket) :: "memory");
            if (lane == 0)
                asm volatile("global_atomic_swap %0, %1, off" :: "v"(myws + OFF_Q + dj * 1024u + ((ticket - (call - 1u) * NQ) & 255u) * 4u),
                             "v"(((call & 0x7fffffu) << 9) | id) : "memory");
        }
        STAMP(8 + t);
    };
    u32x4 a0[8], a1[8];
    int t0i = refill((unsigned)ownmask, a0);
    int t1i = refill((unsigned)ownmask, a1);
    if (remote) {
        const unsigned u0 = (dj * 2u + cblk) * 64u + (unsigned)wave * 8u;
        const gu32* fp = (const gu32*)(myws + OFF_WF + u0 * 16u + (lane & 7) * 4u);
        unsigned have = 0;
#pragma unroll
        for (int s = 0; s < 8; ++s) {
            while (((have >> s) & 1u) == 0u && !dead) {
                have = (unsigned)__ballot(__hip_atomic_load(fp, __ATOMIC_RELAXED, __HIP_MEMORY_SCOPE_SYSTEM) >= call);
                if (((have >> s) & 1u) == 0u) dead = expired(spins, t0, tmo, call, lane);
            }
            asm volatile("" ::: "memory");
#pragma unroll
            for (int e = 0; e < 4; ++e)
                wf[s][e] = __builtin_amdgcn_raw_buffer_load_b128(sr, lane * 16 + e * 1024, OFF_W + (u0 + s) * 4096u, SYS);
        }
#pragma unroll
        for (int s = 0; s < 8; s += 2)
            asm volatile("" : "+v"(wf[s][0]), "+v"(wf[s][1]), "+v"(wf[s][2]), "+v"(wf[s][3]),
                              "+v"(wf[s + 1][0]), "+v"(wf[s + 1][1]), "+v"(wf[s + 1][2]), "+v"(wf[s + 1][3]));
    }
    STAMP(1);

    unsigned long long fv = __hip_atomic_load(flagp, __ATOMIC_RELAXED, __HIP_MEMORY_SCOPE_SYSTEM);
    asm volatile("" : "+v"(fv));
    bool done = false;
    auto claim = [&]() {
        const unsigned fin = rfl(__hip_atomic_load(tcnt + 16, __ATOMIC_RELAXED, __HIP_MEMORY_SCOPE_WORKGROUP));
        const unsigned clm = rfl(__hip_atomic_load(tcnt + 18, __ATOMIC_RELAXED, __HIP_MEMORY_SCOPE_WORKGROUP));
        if ((fin & ~clm) == 0u) return clm == (unsigned)validmask ? -2 : -1;
        const int ts = __builtin_ctz(fin & ~clm);
        int old = 0;
        if (lane == 0) old = __hip_atomic_fetch_or(tcnt + 18, 1 << ts, __ATOMIC_RELAXED, __HIP_MEMORY_SCOPE_WORKGROUP);
        asm volatile("" ::: "memory");
        return ((rfl(old) >> ts) & 1u) != 0u ? -1 : ts;
    };
    auto half = [&](u32x4 (&a)[8], int& t, int other) {
        i32x4 acc[4];
        int pos = NWAVES;
        const int tc = t;
        if (tc >= 0) {
#pragma unroll
            for (int e = 0; e < 4; ++e) acc[e] = (i32x4){0, 0, 0, 0};
#pragma unroll
            for (int s = 0; s < 8; ++s)
#pragma unroll
                for (int e = 0; e < 4; ++e)
                    acc[e] = __builtin_amdgcn_mfma_i32_16x16x64_i8(__builtin_bit_cast(i32x4, wf[s][e]), __builtin_bit_cast(i32x4, a[s]), acc[e], 0, 0, 0);
        }
        const unsigned rdy = (unsigned)__ballot((unsigned)fv >= call && (unsigned)(fv >> 32) >= call) | (unsigned)ownmask;
        if (__builtin_amdgcn_readlane((unsigned)fv, 63) == call) dead = true;
        asm volatile("" ::: "memory");
        fv = __hip_atomic_load(flagp, __ATOMIC_RELAXED, __HIP_MEMORY_SCOPE_SYSTEM);
        t = refill(dead ? 0u : rdy, a);
        __builtin_amdgcn_sched_barrier(0);
        if (tc >= 0) {
            LAS int* tp = (LAS int*)smem + tc * 16 * TSTR + c * TSTR + kq * 16;
#pragma unroll
            for (int r = 0; r < 4; ++r)
#pragma unroll
                for (int e = 0; e < 4; ++e)
                    __hip_atomic_fetch_add(tp + r * 4 + e, acc[e][r], __ATOMIC_RELAXED, __HIP_MEMORY_SCOPE_WORKGROUP);
            pos = arrive(tc);
            STAMP(25 + 3 * tc);
        }
        if (t >= 0 || other >= 0) {
            if (pos < AHEAD) {
                const int ts = claim();
                if (ts >= 0) store_tile(ts);
            }
            return;
        }
        asm volatile("" : "+v"(fv));
        for (;;) {
            if (dead && pend != 0u) {
                if (lane == 0) __hip_atomic_store(tcnt + 17, 1, __ATOMIC_RELAXED, __HIP_MEMORY_SCOPE_WORKGROUP);
                for (; pend != 0u; pend &= pend - 1u) arrive(__builtin_ctz(pend));
            }
            const int ts = claim();
            if (ts >= 0) store_tile(ts);
            if (pend == 0u) {
                if (ts == -2) { done = true; return; }
                if (ts < 0) __builtin_amdgcn_s_sleep(1);
                continue;
            }
            if (((unsigned)__ballot((unsigned)fv >= call && (unsigned)(fv >> 32) >= call) & pend) != 0u) return;
            if (expired(spins, t0, tmo, call, lane)) dead = true;
            fv = __hip_atomic_load(flagp, __ATOMIC_RELAXED, __HIP_MEMORY_SCOPE_SYSTEM);
            asm volatile("" : "+v"(fv));
        }
    };
    auto leave = [&]() {
        STAMP(2);
        LSTAMP(3);
        asm volatile("s_endpgm");
    };
    for (;;) {
        half(a0, t0i, t1i);
        if (done) leave();
        half(a1, t1i, t0i);
        if (done) leave();
    }
}

extern "C" void kernel_launch(int rank, int nranks, ncclComm_t comm,
                              hipStream_t stream,
                              void* const* d_in, const int* in_sizes, int n_in,
                              void* d_out, int out_size,
                              const int* in_dtypes, int out_dtype,
                              void* const* d_out_all,
                              void* const* d_ws_all, size_t ws_bytes) {
    if (nranks != NR || (unsigned)rank >= (unsigned)NR || n_in != 4 || in_sizes[0] != MS * KD || in_sizes[1] != KD * NS
        || out_size != NR * MS * NS || ws_bytes < WS_NEED) {
        fprintf(stderr, "kernel_launch: unexpected shapes\n");
        return;
    }
    Params P;
    P.x = (const signed char*)d_in[0];
    P.w = (const signed char*)d_in[1];
    P.sx = (const float*)d_in[2];
    P.sw = (const float*)d_in[3];
    for (int i = 0; i < NR; ++i) { P.ws[i] = (char*)d_ws_all[i]; P.out[i] = (char*)d_out_all[i]; }
    P.rank = rank;
    P.pad = 0;
    ag_gemm_kernel<<<GRID, NWAVES * 64, 0, stream>>>(P);
    HIP_CHECK(hipGetLastError());
}
```

```cpp
#include <hip/hip_runtime.h>
#include <rccl/rccl.h>
#include <cstdio>
#include <cstdint>

#define HIP_CHECK(expr) do { hipError_t _e = (expr); if (_e != hipSuccess) { \
    fprintf(stderr, "HIP error %s at %s:%d\n", hipGetErrorString(_e), __FILE__, __LINE__); \
    return; } } while (0)

#define GAS __attribute__((address_space(1)))
#define LAS __attribute__((address_space(3)))
typedef unsigned u32x4 __attribute__((ext_vector_type(4)));
typedef int i32x4 __attribute__((ext_vector_type(4)));
typedef float f32x4 __attribute__((ext_vector_type(4)));
typedef unsigned u32x3 __attribute__((ext_vector_type(3)));
typedef GAS unsigned gu32;
typedef GAS unsigned long long gu64;
typedef __amdgpu_buffer_rsrc_t rsrc_t;

#define NGB 8
#define PW 8
#define TAU32 227
#define TAUP32 158
#define T_FIRST 70
#define LEAD 200
#define CREDIT 60
#define ND 2
#define NRAW 3
#define AHEAD 3
constexpr int NR = 8;
constexpr int MS = 512;
constexpr int KD = 4096;
constexpr int NS = 512;
constexpr int NAB = 32 - NGB;
constexpr int GRID = 256;
constexpr int NWAVES = 8;
constexpr int NPUSH = 56;
constexpr int NPW = 8 * PW;
constexpr int RS = 12;
constexpr int NLIST = 128 + NGB;
constexpr int MAXT = (NLIST + RS - 1) / RS;
constexpr int TSTR = 65;
constexpr unsigned SHARD = MS * KD;
constexpr unsigned OUTB = NR * MS * NS * 4;
constexpr unsigned PAR_BYTES = (NR + 1) * SHARD;
constexpr unsigned OFF_W = NR * SHARD;
constexpr unsigned OFF_IN = 2 * PAR_BYTES;
constexpr unsigned OFF_Y = OFF_IN + 4 * 256 * 4096;
constexpr unsigned OFF_XF = OFF_Y + 8 * 256 * 4096;
constexpr unsigned OFF_WF = OFF_XF + NR * 32 * 8 * 128;
constexpr unsigned OFF_IF = OFF_WF + 8192;
constexpr unsigned OFF_DF = OFF_IF + 4096;
constexpr unsigned OFF_Q = OFF_DF + 4096;
constexpr unsigned OFF_QT = OFF_Q + 4096;
constexpr unsigned OFF_CNT = OFF_QT + 4096;
constexpr unsigned OFF_TMO = OFF_CNT + GRID * 128;
constexpr unsigned OFF_DBG = OFF_TMO + 4096;
constexpr unsigned WS_NEED = OFF_DBG + GRID * NWAVES * 64 * 8;
constexpr int NQ = 2 * (128 - NGB - 3 * ND);
constexpr int NSTAT = 128 + 16 * NGB;
static_assert(2 * RS <= 32 && NQ <= 256 && MAXT <= 16 && ND <= 4, "flag / queue layout");
constexpr unsigned long long DEADLINE_TICKS = 50000000ull;
constexpr int SYS = 17;

struct Params {
    const signed char* x;
    const signed char* w;
    const float* sx;
    const float* sw;
    char* ws[NR];
    char* out[NR];
    int rank;
    int pad;
};

__device__ __forceinline__ unsigned long long rt() { return __builtin_amdgcn_s_memrealtime(); }
__device__ __forceinline__ rsrc_t mk(const void* p, unsigned bytes) {
    return __builtin_amdgcn_make_buffer_rsrc((void*)p, 0, (int)bytes, 0x00020000);
}
__device__ __forceinline__ unsigned rfl(unsigned v) { return __builtin_amdgcn_readfirstlane(v); }

#define STAMP(i) do { } while (0)
#define CSTAMP(i) do { } while (0)
#define SMEM_EXTRA 0
#define LSTAMP(i) do { } while (0)

__device__ __forceinline__ void transpose16x4(const unsigned (&r)[16], u32x4 (&o)[4]) {
#pragma unroll
    for (int q = 0; q < 4; ++q) {
        const unsigned r0 = r[4 * q], r1 = r[4 * q + 1], r2 = r[4 * q + 2], r3 = r[4 * q + 3];
        const unsigned lo01 = __builtin_amdgcn_perm(r1, r0, 0x05010400);
        const unsigned hi01 = __builtin_amdgcn_perm(r1, r0, 0x07030602);
        const unsigned lo23 = __builtin_amdgcn_perm(r3, r2, 0x05010400);
        const unsigned hi23 = __builtin_amdgcn_perm(r3, r2, 0x07030602);
        o[0][q] = __builtin_amdgcn_perm(lo23, lo01, 0x05040100);
        o[1][q] = __builtin_amdgcn_perm(lo23, lo01, 0x07060302);
        o[2][q] = __builtin_amdgcn_perm(hi23, hi01, 0x05040100);
        o[3][q] = __builtin_amdgcn_perm(hi23, hi01, 0x07060302);
    }
}

__device__ __forceinline__ bool expired(unsigned& spins, unsigned long long t0, gu32* tmo, unsigned call, int lane) {
    __builtin_amdgcn_s_sleep(1);
    spins = rfl(spins + 1u);
    if ((spins & 63u) != 0u) return false;
    const bool late = rfl(__hip_atomic_load(tmo, __ATOMIC_RELAXED, __HIP_MEMORY_SCOPE_AGENT)) == call
                      || rfl(rt() - t0 > DEADLINE_TICKS ? 1u : 0u) != 0u;
    if (late) __hip_atomic_store(tmo, call, __ATOMIC_RELAXED, __HIP_MEMORY_SCOPE_AGENT);
    return late;
}

__global__ void __launch_bounds__(NWAVES * 64, 2) ag_gemm_kernel(Params P) {
    __shared__ __attribute__((aligned(16))) int smem[MAXT * 16 * TSTR + 32 + SMEM_EXTRA];
    const int b = blockIdx.x;
    const int lane = threadIdx.x & 63;
    const int wave = rfl(threadIdx.x >> 6);
    const unsigned rank = (unsigned)P.rank & 7u;
    const unsigned myidx = rank & 3u, mycls = rank & 4u, otcls = mycls ^ 4u;
    char* myws = P.ws[7];
    gu32* cntp = (gu32*)(myws + OFF_CNT + b * 128 + wave * 4);
    gu32* tmo = (gu32*)(myws + OFF_TMO);
    LSTAMP(0);
    const unsigned cntv = __hip_atomic_load(cntp, __ATOMIC_RELAXED, __HIP_MEMORY_SCOPE_AGENT);
    const rsrc_t xr = mk(P.x, SHARD);
    const rsrc_t wr = mk(P.w, SHARD);
    const int c = lane & 15, kq = lane >> 4;

    if (b < NPUSH) {
        if (wave >= PW) return;
        const unsigned tstart = (unsigned)rt();
        const unsigned di = (unsigned)b % 7u, cu = (unsigned)b / 7u;
        const bool cross = di >= 3u;
        const unsigned dj = (di - 3u) & 3u;
        char* pw = P.ws[di];
        const rsrc_t dr = mk(pw, OFF_Y);
        const rsrc_t dor = mk(P.out[di], OUTB);
        const rsrc_t yr = mk(myws + OFF_Y + dj * (512u * 4096u), 512u * 4096u);
        const gu32* qp = (const gu32*)(myws + OFF_Q + dj * 1024u);
        const unsigned xv = (unsigned)(c * KD + kq * 16);
        const unsigned quart = ((myidx + dj) & 3u) * 128u;
        const unsigned wv = (unsigned)(kq * 16 * NS + c * 4) + quart;
        const int nstat = cross ? NSTAT : 16 * NAB;
        const int nitems = cross ? NSTAT + NQ : 16 * NAB;
        u32x4 d[4];
        auto fetch = [&](int n, unsigned ent) {
            if (n >= nstat) {
#pragma unroll
                for (int j = 0; j < 4; ++j) d[j] = __builtin_amdgcn_raw_buffer_load_b128(yr, lane * 16 + j * 1024, (ent & 511u) * 4096u, SYS);
            } else if (cross && n < 128) {
                unsigned r[16];
                const unsigned so = (unsigned)(n & 63) * 64u * NS + (unsigned)(n >> 6) * 64u;
#pragma unroll
                for (int i = 0; i < 16; ++i) r[i] = __builtin_amdgcn_raw_buffer_load_b32(wr, wv + i * NS, so, 0);
                transpose16x4(r, d);
            } else {
                const int m = cross ? n - 128 : n + 16 * NGB;
                const unsigned so = (unsigned)(m >> 4) * 16u * KD + (unsigned)(m & 15) * 256u;
#pragma unroll
                for (int j = 0; j < 4; ++j) d[j] = __builtin_amdgcn_raw_buffer_load_b128(xr, xv + j * 64, so, 0);
            }
            asm volatile("" : "+v"(d[0]), "+v"(d[1]), "+v"(d[2]), "+v"(d[3]));
        };
        int n = wave * 8 + (int)cu;
        while ((int)(tstart + T_FIRST + n * TAU32 / 32 - LEAD - (unsigned)rt()) > 0) __builtin_amdgcn_s_sleep(1);
        fetch(n, 0);
        const unsigned call = rfl(cntv) + 1;
        const unsigned ctag = call & 0x3fffffu;
        const unsigned pbase = (call & 1u) * PAR_BYTES;
        const unsigned long long t0 = rt();
        unsigned spins = 0, ent = 0;
        unsigned origin = tstart + T_FIRST;
        bool dead = false;
        STAMP(1);
        for (;;) {
            {
                const unsigned slotT = origin + (unsigned)(n <= NSTAT || !cross ? n * TAU32 : NSTAT * TAU32 + (n - NSTAT) * TAUP32) / 32u;
                unsigned now = (unsigned)rt();
                while ((int)(slotT - now) > 0) {
                    if ((int)(slotT - now) > 8) __builtin_amdgcn_s_sleep(1);
                    now = (unsigned)rt();
                }
                if (n >= nstat && (int)(now - slotT) > CREDIT) origin += now - slotT - CREDIT;
            }
            unsigned fo, fval = call;
            if (n >= nstat) {
                const unsigned sl = (unsigned)(n - NSTAT);
                fo = OFF_IF + (myidx * 256u + (sl & 15u) * 16u + (sl >> 4)) * 4u;
                fval = ent;
                if (ent & 512u) {
                    const unsigned so = OFF_IN + (myidx * 256u + sl) * 4096u + lane * 16;
                    __builtin_amdgcn_raw_buffer_store_b128(d[0], dr, so, 0, SYS);
                    __builtin_amdgcn_raw_buffer_store_b128(d[1], dr, so + 1024, 0, SYS);
                    if (lane < 48) __builtin_amdgcn_raw_buffer_store_b128(d[2], dr, so + 2048, 0, SYS);
                } else {
                    const unsigned oo = ((ent & 255u) * 16u * NS + quart + ((ent >> 8) & 1u) * 64u) * 4u + (unsigned)(kq * NS * 4 + c * 16);
#pragma unroll
                    for (int j = 0; j < 4; ++j) __builtin_amdgcn_raw_buffer_store_b128(d[j], dor, oo + j * 4 * NS * 4, 0, SYS);
                    origin += (TAU32 - TAUP32) * NPW / 32;
                }
            } else {
                unsigned so;
                if (cross && n < 128) {
                    const unsigned u = myidx * 128u + (unsigned)n;
                    so = pbase + OFF_W + u * 4096u;
                    fo = OFF_WF + (u >> 3) * 128u + (u & 7u) * 4u;
                } else {
                    const unsigned m = (unsigned)(cross ? n - 128 : n + 16 * NGB);
                    so = pbase + rank * SHARD + m * 4096u;
                    fo = OFF_XF + (rank * 256u + (m >> 1)) * 128u + (m & 1u) * 4u;
                }
#pragma unroll
                for (int j = 0; j < 4; ++j) __builtin_amdgcn_raw_buffer_store_b128(d[j], dr, so + lane * 16 + j * 1024, 0, SYS);
            }
            const int n1 = n + NPW;
            const bool queued = n1 >= nstat && n1 < nitems;
            const gu32* q1 = qp + ((unsigned)(n1 - NSTAT) & 255u);
            unsigned ev = 0;
            if (queued) asm volatile("s_mov_b64 exec, 1\n\tglobal_atomic_add %0, %1, %2, off sc0\n\ts_mov_b64 exec, -1"
                                     : "+v"(ev) : "v"(q1), "v"(0u) : "memory");
            asm volatile("s_waitcnt vmcnt(0)" : "+v"(ev) :: "memory");
            if (lane == 0) __hip_atomic_store((gu32*)(pw + fo), fval, __ATOMIC_RELAXED, __HIP_MEMORY_SCOPE_SYSTEM);
            if (n1 >= nitems) break;
            ent = rfl(ev);
            while (queued && (ent >> 10) != ctag && !dead) {
                dead = expired(spins, t0, tmo, call, lane);
                asm volatile("s_mov_b64 exec, 1\n\tglobal_atomic_add %0, %1, %2, off sc0\n\ts_mov_b64 exec, -1\n\ts_waitcnt vmcnt(0)"
                             : "=&v"(ev) : "v"(q1), "v"(0u) : "memory");
                ent = rfl(ev);
            }
            if (dead) break;
            n = n1;
            fetch(n, ent);
        }
        if (lane == 0) __hip_atomic_store(cntp, call, __ATOMIC_RELAXED, __HIP_MEMORY_SCOPE_AGENT);
        return;
    }
    if (b < 64) {
        const unsigned u = (unsigned)(b - NPUSH) * NWAVES + (unsigned)wave, si = u & 3u, s0 = u >> 2;
        const unsigned call = rfl(cntv) + 1;
        const unsigned ctag = call & 0x3fffffu;
        if (lane == 0) __hip_atomic_store(cntp, call, __ATOMIC_RELAXED, __HIP_MEMORY_SCOPE_AGENT);
        const float scale = P.sx[0] * P.sw[0];
        const rsrc_t orr = mk(P.out[7], OUTB);
        const int lrs = (lane - 16) % RS;
        const bool need = lane < 16 ? s0 + 16u * lane < NQ
                                    : (s0 == 0u && lane < 16 + 2 * RS && (lrs + (3 * NAB - 1 - lrs) / RS * RS) / 3 >= NAB - ND);
        const gu32* fp = (const gu32*)(myws + (lane < 16 ? OFF_IF + (si * 256u + s0 * 16u + lane) * 4u : OFF_DF + (si * 32u + ((lane - 16) & 31)) * 4u));
        const unsigned obase = ((si + myidx) & 3u) * 512u + (unsigned)(kq * NS * 4 + c * 16);
        unsigned long long pend = __ballot(need);
        const unsigned long long t0 = rt();
        unsigned spins = 0;
        while (pend != 0ull) {
            const unsigned fv = __hip_atomic_load(fp, __ATOMIC_RELAXED, __HIP_MEMORY_SCOPE_SYSTEM);
            const unsigned long long rdy = __ballot((fv >> 10) == ctag) & pend;
            asm volatile("" ::: "memory");
            if (rdy == 0ull) {
                if (expired(spins, t0, tmo, call, lane)) {
                    if (lane == 0) __hip_atomic_store((gu32*)(P.out[7] + (size_t)rank * MS * NS * 4), 0x7fc00000u,
                                                      __ATOMIC_RELAXED, __HIP_MEMORY_SCOPE_SYSTEM);
                    break;
                }
                continue;
            }
            unsigned todo = (unsigned)(rdy & __ballot((fv & 512u) != 0u));
            pend &= ~rdy | (unsigned long long)todo;
            constexpr int UB = 4;
            u32x4 lo[UB][2];
            u32x3 hi[UB];
            int k[UB];
#pragma unroll
            for (int i = 0; i < UB; ++i) {
                k[i] = todo ? __builtin_ctz(todo) : -1;
                const rsrc_t r = mk(myws + OFF_IN + si * (256u * 4096u), todo ? 256u * 4096u : 0u);
                pend &= ~(unsigned long long)(todo & (0u - todo));
                todo &= todo - 1u;
                if (i == 0 || k[i] >= 0) {
                    const unsigned so = (s0 + 16u * ((unsigned)k[i] & 15u)) * 4096u;
                    lo[i][0] = __builtin_amdgcn_raw_buffer_load_b128(r, lane * 16, so, SYS);
                    lo[i][1] = __builtin_amdgcn_raw_buffer_load_b128(r, lane * 16 + 1024, so, SYS);
                    hi[i] = __builtin_amdgcn_raw_buffer_load_b96(r, lane * 12 + 2048, so, SYS);
                }
            }
#pragma unroll
            for (int i = 0; i < UB; ++i) {
                if (k[i] < 0) break;
                const unsigned ent = __builtin_amdgcn_readlane(fv, k[i]);
                const unsigned oo = (ent & 255u) * (16u * NS * 4u) + (ent & 256u) + obase;
                const unsigned h3 = ((hi[i][0] >> 6) & 0x03030303u) | ((hi[i][1] >> 4) & 0x0c0c0c0cu) | ((hi[i][2] >> 2) & 0x30303030u);
#pragma unroll
                for (int g = 0; g < 4; ++g) {
                    const unsigned h = g < 3 ? hi[i][g < 3 ? g : 0] : h3;
                    const unsigned l0 = lo[i][g >> 1][(2 * g) & 3], l1 = lo[i][g >> 1][(2 * g + 1) & 3];
                    f32x4 y;
                    y[0] = (float)(int)__builtin_amdgcn_sbfe(__builtin_amdgcn_perm(h, l0, 0x0c040100), 0, 22) * scale;
                    y[1] = (float)(int)__builtin_amdgcn_sbfe(__builtin_amdgcn_perm(h, l0, 0x0c050302), 0, 22) * scale;
                    y[2] = (float)(int)__builtin_amdgcn_sbfe(__builtin_amdgcn_perm(h, l1, 0x0c060100), 0, 22) * scale;
                    y[3] = (float)(int)__builtin_amdgcn_sbfe(__builtin_amdgcn_perm(h, l1, 0x0c070302), 0, 22) * scale;
                    __builtin_amdgcn_raw_buffer_store_b128(__builtin_bit_cast(u32x4, y), orr, oo + g * 4 * NS * 4, 0, SYS);
                }
            }
        }
        LSTAMP(3);
        return;
    }

    const int xcd = b & 7, slot = (b - 64) >> 3;
    const int rs = 3 * (xcd >> 1) + (slot >> 3);
    const bool remote = (slot & 4) != 0;
    const unsigned dj = (unsigned)slot & 3u;
    const unsigned cblk = (unsigned)xcd & 1u;
    const unsigned owner = remote ? (otcls | dj) : rank;
    const unsigned col0 = remote ? ((myidx + dj) & 3u) * 128u + cblk * 64u : (cblk * 4u + dj) * 64u;
    LAS int* tcnt = (LAS int*)smem + MAXT * 16 * TSTR;

    u32x4 wf[8][4];
    if (!remote) {
        const unsigned wv = (unsigned)(kq * 16 * NS + c * 4);
#pragma unroll
        for (int s = 0; s < 8; ++s) {
            unsigned r[16];
            const unsigned so = (unsigned)(wave * 8 + s) * 64u * NS + col0;
#pragma unroll
            for (int i = 0; i < 16; ++i) r[i] = __builtin_amdgcn_raw_buffer_load_b32(wr, wv + i * NS, so, 0);
            transpose16x4(r, wf[s]);
        }
    }
    for (int i = threadIdx.x; i < MAXT * 16 * TSTR + 32 + SMEM_EXTRA; i += NWAVES * 64) smem[i] = 0;
    __syncthreads();
    const unsigned call = rfl(cntv) + 1;
    if (lane == 0) __hip_atomic_store(cntp, call, __ATOMIC_RELAXED, __HIP_MEMORY_SCOPE_AGENT);
    const float scale = P.sx[0] * P.sw[0];
    const unsigned pbase = (call & 1u) * PAR_BYTES;
    const rsrc_t sr = mk(myws + pbase, PAR_BYTES);
    const rsrc_t orr = mk(P.out[remote ? 3u + dj : 7u], OUTB);
    const rsrc_t yr = mk(myws + OFF_Y + dj * (512u * 4096u), 512u * 4096u);
    gu32* dflag = (gu32*)(P.ws[3u + dj] + OFF_DF + (myidx * 32u + cblk * RS + (unsigned)rs) * 4u);
    asm volatile("" : "+s"(dflag));
    int tlast = (3 * NAB - 1 - rs) / RS;
    const int traw = (rs + tlast * RS) / 3 >= NAB - NRAW ? tlast : -1;
    if ((rs + tlast * RS) / 3 < NAB - ND) tlast = -1;
    const unsigned long long t0 = rt();
    unsigned spins = 0;
    bool dead = false;

    unsigned tsrc, trb;
    bool tvalid;
    {
        const unsigned idx = (unsigned)rs + RS * (unsigned)lane;
        if (idx < 3u * NAB) { tsrc = mycls | ((myidx + 1u + idx % 3u) & 3u); trb = NGB + idx / 3u; tvalid = true; }
        else if (idx < 4u * NAB) { tsrc = rank; trb = NGB + idx - 3u * NAB; tvalid = true; }
        else if (idx < 128u) { const unsigned u = idx - 4u * NAB; tsrc = otcls | (u & 3u); trb = u >> 2; tvalid = tsrc != owner; }
        else { tsrc = rank; trb = (idx - 128u) & 31u; tvalid = !remote && idx < NLIST; }
    }
    const unsigned tdesc = tsrc * 32u + trb;
    const unsigned long long validmask = __ballot(tvalid);
    const unsigned long long ownmask = __ballot(tvalid && tsrc == rank);
    const gu64* flagp = (tvalid && tsrc != rank) ? (const gu64*)(myws + OFF_XF + (tdesc * 8u + (unsigned)wave) * 128u)
                                                 : (const gu64*)tmo;
    const unsigned vown = (unsigned)(c * KD + kq * 16), vrem = (unsigned)(lane * 16);

    unsigned pend = (unsigned)validmask;
    auto refill = [&](unsigned ready, u32x4 (&a)[8]) {
        const unsigned todo = ready & pend;
        const int t = todo ? __builtin_ctz(todo) : -1;
        pend &= ~(todo & (0u - todo));
        const unsigned desc = __builtin_amdgcn_readlane(tdesc, t & 63);
        const bool own = (desc >> 5) == rank;
        const rsrc_t r = mk(own ? (const char*)P.x : myws + pbase, todo ? (own ? SHARD : OFF_W) : 0u);
        const unsigned so = own ? (desc & 31u) * 65536u + (unsigned)wave * 512u : desc * 65536u + (unsigned)wave * 8192u;
        const unsigned step = own ? 64u : 1024u;
        const unsigned vo = own ? vown : vrem;
#pragma unroll
        for (int s = 0; s < 8; ++s) a[s] = __builtin_amdgcn_raw_buffer_load_b128(r, vo, so + s * step, SYS);
        return t;
    };
    auto arrive = [&](int t, bool take) {
        asm volatile("s_waitcnt lgkmcnt(0)" ::: "memory");
        int old = 0;
        if (lane == 0) old = __hip_atomic_fetch_add(tcnt + t, 1, __ATOMIC_RELAXED, __HIP_MEMORY_SCOPE_WORKGROUP);
        old = rfl(old);
        if (old == NWAVES - 1 && lane == 0) __hip_atomic_fetch_or(tcnt + (take ? 18 : 16), 1 << t, __ATOMIC_RELAXED, __HIP_MEMORY_SCOPE_WORKGROUP);
        asm volatile("" ::: "memory");
        return old;
    };
    auto store_tile = [&](int t) {
        const unsigned desc = __builtin_amdgcn_readlane(tdesc, t);
        const bool bad = __hip_atomic_load(tcnt + 17, __ATOMIC_RELAXED, __HIP_MEMORY_SCOPE_WORKGROUP) != 0;
        const float sc = bad ? __builtin_nanf("") : scale;
        const LAS int* tp = (const LAS int*)smem + t * 16 * TSTR + kq * TSTR + c * 4;
        unsigned v[16];
        unsigned span = 0;
#pragma unroll
        for (int i = 0; i < 16; ++i) { v[i] = (unsigned)tp[(i >> 2) * 4 * TSTR + (i & 3)]; span |= v[i] + 0x200000u; }
        const bool queued = remote && t != tlast;
        const bool pack = queued && !bad && t != traw && __all((span >> 22) == 0u);
        const unsigned id = cblk * 256u + desc;
        unsigned ticket = 0;
        if (queued) asm volatile("s_mov_b64 exec, 1\n\tglobal_atomic_add %0, %1, %2, off sc0\n\ts_mov_b64 exec, -1"
                                 : "+v"(ticket) : "v"(myws + OFF_QT + dj * 128u), "v"(1u) : "memory");
        if (pack) {
            unsigned h[4];
            u32x4 lo[2];
#pragma unroll
            for (int g = 0; g < 4; ++g) {
                h[g] = __builtin_amdgcn_perm(v[4 * g + 1], v[4 * g], 0x0c0c0602) | __builtin_amdgcn_perm(v[4 * g + 3], v[4 * g + 2], 0x06020c0c);
                lo[g >> 1][(2 * g) & 3] = __builtin_amdgcn_perm(v[4 * g + 1], v[4 * g], 0x05040100);
                lo[g >> 1][(2 * g + 1) & 3] = __builtin_amdgcn_perm(v[4 * g + 3], v[4 * g + 2], 0x05040100);
            }
            u32x3 hi;
#pragma unroll
            for (int g = 0; g < 3; ++g) hi[g] = (h[g] & 0x3f3f3f3fu) | ((h[3] << (6 - 2 * g)) & 0xc0c0c0c0u);
            __builtin_amdgcn_raw_buffer_store_b128(lo[0], yr, id * 4096u + lane * 16, 0, SYS);
            __builtin_amdgcn_raw_buffer_store_b128(lo[1], yr, id * 4096u + 1024 + lane * 16, 0, SYS);
            __builtin_amdgcn_raw_buffer_store_b96(hi, yr, id * 4096u + 2048 + lane * 12, 0, SYS);
        } else {
            const rsrc_t r = queued ? yr : orr;
            const unsigned oo = queued ? id * 4096u + lane * 16 : (desc * 16u * NS + col0) * 4u + (unsigned)(kq * NS * 4 + c * 16);
            const unsigned st = queued ? 1024u : 4u * NS * 4u;
#pragma unroll
            for (int i = 0; i < 4; ++i) {
                f32x4 y;
#pragma unroll
                for (int e = 0; e < 4; ++e) y[e] = (float)(int)v[4 * i + e] * sc;
                __builtin_amdgcn_raw_buffer_store_b128(__builtin_bit_cast(u32x4, y), r, oo + i * st, 0, SYS);
            }
        }
        if (remote) {
            asm volatile("s_waitcnt vmcnt(0)" : "+v"(ticket) :: "memory");
            const unsigned tag = (call & 0x3fffffu) << 10;
            if (lane == 0) {
                if (queued)
                    asm volatile("global_atomic_swap %0, %1, off" :: "v"(myws + OFF_Q + dj * 1024u + ((ticket - (call - 1u) * NQ) & 255u) * 4u),
                                 "v"(tag | (pack ? 512u : 0u) | id) : "memory");
                else
                    __hip_atomic_store(dflag, tag, __ATOMIC_RELAXED, __HIP_MEMORY_SCOPE_SYSTEM);
            }
        }
        CSTAMP(8 + t);
    };
    u32x4 a0[8], a1[8];
    int t0i = refill((unsigned)ownmask, a0);
    int t1i = refill((unsigned)ownmask, a1);
    if (remote) {
        const unsigned u0 = (dj * 2u + cblk) * 64u + (unsigned)wave * 8u;
        const gu32* fp = (const gu32*)(myws + OFF_WF + u0 * 16u + (lane & 7) * 4u);
        unsigned have = 0;
#pragma unroll
        for (int s = 0; s < 8; ++s) {
            while (((have >> s) & 1u) == 0u && !dead) {
                have = (unsigned)__ballot(__hip_atomic_load(fp, __ATOMIC_RELAXED, __HIP_MEMORY_SCOPE_SYSTEM) >= call);
                if (((have >> s) & 1u) == 0u) dead = expired(spins, t0, tmo, call, lane);
            }
            asm volatile("" ::: "memory");
#pragma unroll
            for (int e = 0; e < 4; ++e)
                wf[s][e] = __builtin_amdgcn_raw_buffer_load_b128(sr, lane * 16 + e * 1024, OFF_W + (u0 + s) * 4096u, SYS);
        }
#pragma unroll
        for (int s = 0; s < 8; s += 2)
            asm volatile("" : "+v"(wf[s][0]), "+v"(wf[s][1]), "+v"(wf[s][2]), "+v"(wf[s][3]),
                              "+v"(wf[s + 1][0]), "+v"(wf[s + 1][1]), "+v"(wf[s + 1][2]), "+v"(wf[s + 1][3]));
    }
    CSTAMP(1);

    unsigned long long fv = __hip_atomic_load(flagp, __ATOMIC_RELAXED, __HIP_MEMORY_SCOPE_SYSTEM);
    asm volatile("" : "+v"(fv));
    bool done = false;
    auto claim = [&]() {
        const unsigned fin = rfl(__hip_atomic_load(tcnt + 16, __ATOMIC_RELAXED, __HIP_MEMORY_SCOPE_WORKGROUP));
        const unsigned clm = rfl(__hip_atomic_load(tcnt + 18, __ATOMIC_RELAXED, __HIP_MEMORY_SCOPE_WORKGROUP));
        if ((fin & ~clm) == 0u) return clm == (unsigned)validmask ? -2 : -1;
        const int ts = __builtin_ctz(fin & ~clm);
        int old = 0;
        if (lane == 0) old = __hip_atomic_fetch_or(tcnt + 18, 1 << ts, __ATOMIC_RELAXED, __HIP_MEMORY_SCOPE_WORKGROUP);
        asm volatile("" ::: "memory");
        return ((rfl(old) >> ts) & 1u) != 0u ? -1 : ts;
    };
    auto half = [&](u32x4 (&a)[8], int& t, int other) {
        i32x4 acc[4];
        int pos = NWAVES;
        const int tc = t;
        if (tc >= 0) {
#pragma unroll
            for (int e = 0; e < 4; ++e) acc[e] = (i32x4){0, 0, 0, 0};
#pragma unroll
            for (int s = 0; s < 8; ++s)
#pragma unroll
                for (int e = 0; e < 4; ++e)
                    acc[e] = __builtin_amdgcn_mfma_i32_16x16x64_i8(__builtin_bit_cast(i32x4, wf[s][e]), __builtin_bit_cast(i32x4, a[s]), acc[e], 0, 0, 0);
        }
        const unsigned rdy = (unsigned)__ballot((unsigned)fv >= call && (unsigned)(fv >> 32) >= call) | (unsigned)ownmask;
        if (__builtin_amdgcn_readlane((unsigned)fv, 63) == call) dead = true;
        asm volatile("" ::: "memory");
        fv = __hip_atomic_load(flagp, __ATOMIC_RELAXED, __HIP_MEMORY_SCOPE_SYSTEM);
        t = refill(dead ? 0u : rdy, a);
        __builtin_amdgcn_sched_barrier(0);
        if (t >= 0) CSTAMP(24 + 3 * t);
        const bool busy = t >= 0 || other >= 0;
        if (tc >= 0) {
            LAS int* tp = (LAS int*)smem + tc * 16 * TSTR + c * TSTR + kq * 16;
#pragma unroll
            for (int r = 0; r < 4; ++r)
#pragma unroll
                for (int e = 0; e < 4; ++e)
                    __hip_atomic_fetch_add(tp + r * 4 + e, acc[e][r], __ATOMIC_RELAXED, __HIP_MEMORY_SCOPE_WORKGROUP);
            pos = arrive(tc, !busy);
            CSTAMP(25 + 3 * tc);
        }
        if (busy ? pos < AHEAD : pos == NWAVES - 1) {
            const int ts = busy ? claim() : tc;
            if (ts >= 0) store_tile(ts);
        }
        if (busy) return;
        asm volatile("" : "+v"(fv));
        for (;;) {
            if (dead && pend != 0u) {
                if (lane == 0) __hip_atomic_store(tcnt + 17, 1, __ATOMIC_RELAXED, __HIP_MEMORY_SCOPE_WORKGROUP);
                for (; pend != 0u; pend &= pend - 1u) arrive(__builtin_ctz(pend), false);
            }
            const int ts = claim();
            if (ts >= 0) store_tile(ts);
            if (pend == 0u) {
                if (ts == -2) { done = true; return; }
                if (ts < 0) __builtin_amdgcn_s_sleep(1);
                continue;
            }
            if (((unsigned)__ballot((unsigned)fv >= call && (unsigned)(fv >> 32) >= call) & pend) != 0u) return;
            if (expired(spins, t0, tmo, call, lane)) dead = true;
            fv = __hip_atomic_load(flagp, __ATOMIC_RELAXED, __HIP_MEMORY_SCOPE_SYSTEM);
            asm volatile("" : "+v"(fv));
        }
    };
    auto leave = [&]() {
        CSTAMP(2);
        LSTAMP(3);
        asm volatile("s_endpgm");
    };
    for (;;) {
        half(a0, t0i, t1i);
        if (done) leave();
        half(a1, t1i, t0i);
        if (done) leave();
    }
}

extern "C" void kernel_launch(int rank, int nranks, ncclComm_t comm,
                              hipStream_t stream,
                              void* const* d_in, const int* in_sizes, int n_in,
                              void* d_out, int out_size,
                              const int* in_dtypes, int out_dtype,
                              void* const* d_out_all,
                              void* const* d_ws_all, size_t ws_bytes) {
    if (nranks != NR || (unsigned)rank >= (unsigned)NR || n_in != 4 || in_sizes[0] != MS * KD || in_sizes[1] != KD * NS
        || out_size != NR * MS * NS || ws_bytes < WS_NEED) {
        fprintf(stderr, "kernel_launch: unexpected shapes\n");
        return;
    }
    Params P;
    P.x = (const signed char*)d_in[0];
    P.w = (const signed char*)d_in[1];
    P.sx = (const float*)d_in[2];
    P.sw = (const float*)d_in[3];
    for (int i = 0; i < NR; ++i) {
        const int p = i < 3 ? ((rank & 4) | ((rank + 1 + i) & 3)) : (i < 7 ? (((rank & 4) ^ 4) | (i - 3)) : rank);
        P.ws[i] = (char*)d_ws_all[p];
        P.out[i] = (char*)d_out_all[p];
    }
    P.rank = rank;
    P.pad = 0;
    ag_gemm_kernel<<<GRID, NWAVES * 64, 0, stream>>>(P);
    HIP_CHECK(hipGetLastError());
}
```

```cpp
#include <hip/hip_runtime.h>
#include <rccl/rccl.h>
#include <cstdio>
#include <cstdint>

#define HIP_CHECK(expr) do { hipError_t _e = (expr); if (_e != hipSuccess) { \
    fprintf(stderr, "HIP error %s at %s:%d\n", hipGetErrorString(_e), __FILE__, __LINE__); \
    return; } } while (0)

#define GAS __attribute__((address_space(1)))
#define LAS __attribute__((address_space(3)))
typedef unsigned u32x4 __attribute__((ext_vector_type(4)));
typedef int i32x4 __attribute__((ext_vector_type(4)));
typedef float f32x4 __attribute__((ext_vector_type(4)));
typedef GAS unsigned gu32;
typedef GAS unsigned long long gu64;
typedef __amdgpu_buffer_rsrc_t rsrc_t;

#define NGB 9
#define PW 8
#define TAU32 226
#define TAUI32 222
#define T_FIRSTI 70
#define TAUP32 152
#define T_FIRST 80
#define LEAD 200
#define CREDIT 60
#define NDL 2
#define WATCH 260
#define HADJ 280
#define POLL_SLEEP 1
#define AHEAD 3
constexpr int NR = 8;
constexpr int MS = 512;
constexpr int KD = 4096;
constexpr int NS = 512;
constexpr int NAB = 32 - NGB;
constexpr int GRID = 256;
constexpr int NWAVES = 8;
constexpr int NPUSH = 56;
constexpr int NPW = 8 * PW;
constexpr int RS = 12;
constexpr int NIN = NAB + NDL;
constexpr int NLIST = 128 + NGB + 3 * NDL;
constexpr int MAXT = (NLIST + RS - 1) / RS;
constexpr int TSTR = 65;
constexpr unsigned SHARD = MS * KD;
constexpr unsigned OUTB = NR * MS * NS * 4;
constexpr unsigned PAR_BYTES = (NR + 1) * SHARD;
constexpr unsigned OFF_W = NR * SHARD;
constexpr unsigned OFF_Y = 2 * PAR_BYTES;
constexpr unsigned OFF_XF = OFF_Y + 8 * 256 * 4096;
constexpr unsigned OFF_WF = OFF_XF + NR * 32 * 8 * 128;
constexpr unsigned OFF_Q = OFF_WF + 8192;
constexpr unsigned OFF_QT = OFF_Q + 4096;
constexpr unsigned OFF_CNT = OFF_QT + 4096;
constexpr unsigned OFF_TMO = OFF_CNT + GRID * 128;
constexpr unsigned OFF_DBG = OFF_TMO + 4096;
constexpr unsigned WS_NEED = OFF_DBG + GRID * NWAVES * 64 * 8;
constexpr int NQ = 2 * (4 * NAB + 3 * (NGB - NDL));
constexpr int NSTAT = 128 + 16 * NGB;
static_assert(NPW == 64 && NQ <= 256 && MAXT <= 13 && NDL <= NGB, "flag / queue layout");
constexpr unsigned long long DEADLINE_TICKS = 50000000ull;
constexpr int SYS = 17;

struct Params {
    const signed char* x;
    const signed char* w;
    const float* sx;
    const float* sw;
    char* ws[NR];
    char* out;
    int rank;
    int pad;
};

__device__ __forceinline__ unsigned long long rt() { return __builtin_amdgcn_s_memrealtime(); }
__device__ __forceinline__ rsrc_t mk(const void* p, unsigned bytes) {
    return __builtin_amdgcn_make_buffer_rsrc((void*)p, 0, (int)bytes, 0x00020000);
}
__device__ __forceinline__ unsigned rfl(unsigned v) { return __builtin_amdgcn_readfirstlane(v); }

#define STAMP(i) do { } while (0)
#define CSTAMP(i) do { } while (0)
#define SMEM_EXTRA 0
#define LSTAMP(i) do { } while (0)

__device__ __forceinline__ void transpose16x4(const unsigned (&r)[16], u32x4 (&o)[4]) {
#pragma unroll
    for (int q = 0; q < 4; ++q) {
        const unsigned r0 = r[4 * q], r1 = r[4 * q + 1], r2 = r[4 * q + 2], r3 = r[4 * q + 3];
        const unsigned lo01 = __builtin_amdgcn_perm(r1, r0, 0x05010400);
        const unsigned hi01 = __builtin_amdgcn_perm(r1, r0, 0x07030602);
        const unsigned lo23 = __builtin_amdgcn_perm(r3, r2, 0x05010400);
        const unsigned hi23 = __builtin_amdgcn_perm(r3, r2, 0x07030602);
        o[0][q] = __builtin_amdgcn_perm(lo23, lo01, 0x05040100);
        o[1][q] = __builtin_amdgcn_perm(lo23, lo01, 0x07060302);
        o[2][q] = __builtin_amdgcn_perm(hi23, hi01, 0x05040100);
        o[3][q] = __builtin_amdgcn_perm(hi23, hi01, 0x07060302);
    }
}

__device__ __forceinline__ bool expired(unsigned& spins, unsigned long long t0, gu32* tmo, unsigned call, int lane) {
    if (POLL_SLEEP) __builtin_amdgcn_s_sleep(POLL_SLEEP);
    spins = rfl(spins + 1u);
    if ((spins & 63u) != 0u) return false;
    const bool late = rfl(__hip_atomic_load(tmo, __ATOMIC_RELAXED, __HIP_MEMORY_SCOPE_AGENT)) == call
                      || rfl(rt() - t0 > DEADLINE_TICKS ? 1u : 0u) != 0u;
    if (late) __hip_atomic_store(tmo, call, __ATOMIC_RELAXED, __HIP_MEMORY_SCOPE_AGENT);
    return late;
}

__global__ void __launch_bounds__(NWAVES * 64, 2) ag_gemm_kernel(Params P) {
    __shared__ __attribute__((aligned(16))) int smem[MAXT * 16 * TSTR + 32 + SMEM_EXTRA];
    const int b = blockIdx.x;
    const int lane = threadIdx.x & 63;
    const int wave = rfl(threadIdx.x >> 6);
    const unsigned rank = (unsigned)P.rank & 7u;
    const unsigned myidx = rank & 3u, mycls = rank & 4u, otcls = mycls ^ 4u;
    char* myws = P.ws[7];
    gu32* cntp = (gu32*)(myws + OFF_CNT + b * 128 + wave * 4);
    gu32* tmo = (gu32*)(myws + OFF_TMO);
    LSTAMP(0);
    const unsigned cntv = __hip_atomic_load(cntp, __ATOMIC_RELAXED, __HIP_MEMORY_SCOPE_AGENT);
    const rsrc_t xr = mk(P.x, SHARD);
    const rsrc_t wr = mk(P.w, SHARD);
    const int c = lane & 15, kq = lane >> 4;

    if (b < NPUSH) {
        const unsigned tstart = (unsigned)rt();
        const unsigned di = (unsigned)b % 7u, cu = (unsigned)b / 7u;
        const bool cross = di >= 3u;
        const unsigned dj = (di - 3u) & 3u;
        char* pw = P.ws[di];
        const rsrc_t dr = mk(pw, OFF_Y);
        const unsigned xv = (unsigned)(c * KD + kq * 16);
        const unsigned quart = ((myidx + dj) & 3u) * 128u;
        const unsigned wv = (unsigned)(kq * 16 * NS + c * 4) + quart;
        const int npairs = cross ? NSTAT / 2 : 8 * NIN;
        auto until = [&](unsigned t) {
            unsigned now = (unsigned)rt();
            while ((int)(t - now) > 0) {
                if ((int)(t - now) > 8) __builtin_amdgcn_s_sleep(1);
                now = (unsigned)rt();
            }
            return now;
        };
        const int q = wave * 8 + (int)cu;
        unsigned origin = tstart + (cross ? T_FIRST : T_FIRSTI);
        const int tau = cross ? TAU32 : TAUI32;
        unsigned call = 0;
        for (int g = q; g < npairs; g += NPW) {
            const int sl = 2 * g - (int)cu;
            until(origin + sl * tau / 32 - LEAD);
            const bool isw = cross && g < 64;
            const unsigned m = (unsigned)(cross ? g - 64 : (g < 8 * NAB ? g + 8 * NGB : g - 8 * (NAB - NGB + NDL)));
            u32x4 e[2][4];
            if (isw) {
                const unsigned so = (unsigned)(g & 31) * 128u * NS + (unsigned)(g >> 5) * 64u;
#pragma unroll
                for (int u = 0; u < 2; ++u) {
                    unsigned r[16];
#pragma unroll
                    for (int i = 0; i < 16; ++i) r[i] = __builtin_amdgcn_raw_buffer_load_b32(wr, wv + i * NS, so + u * 64 * NS, 0);
                    transpose16x4(r, e[u]);
                }
            } else {
                const unsigned so = (m >> 3) * 16u * KD + (m & 7u) * 512u;
#pragma unroll
                for (int u = 0; u < 2; ++u)
#pragma unroll
                    for (int j = 0; j < 4; ++j) e[u][j] = __builtin_amdgcn_raw_buffer_load_b128(xr, xv + u * 256 + j * 64, so, 0);
            }
            call = rfl(cntv) + 1;
            const unsigned gw = myidx * 64u + (unsigned)g;
            const unsigned so = (call & 1u) * PAR_BYTES + (isw ? OFF_W + gw * 8192u : rank * SHARD + m * 8192u) + lane * 16;
            const unsigned fo = isw ? OFF_WF + (gw >> 2) * 128u + (gw & 3u) * 4u : OFF_XF + (rank * 256u + m) * 128u;
#pragma unroll
            for (int u = 0; u < 2; ++u) asm volatile("" : "+v"(e[u][0]), "+v"(e[u][1]), "+v"(e[u][2]), "+v"(e[u][3]));
            if (g == q) STAMP(1);
#pragma unroll
            for (int u = 0; u < 2; ++u) {
                until(origin + (sl + 8 * u) * tau / 32);
#pragma unroll
                for (int j = 0; j < 4; ++j) __builtin_amdgcn_raw_buffer_store_b128(e[u][j], dr, so + u * 4096 + j * 1024, 0, SYS);
            }
            asm volatile("s_waitcnt vmcnt(0)" ::: "memory");
            if (lane == 0) __hip_atomic_store((gu32*)(pw + fo), call, __ATOMIC_RELAXED, __HIP_MEMORY_SCOPE_SYSTEM);
        }
        call = rfl(cntv) + 1;
        if (lane == 0) __hip_atomic_store(cntp, call, __ATOMIC_RELAXED, __HIP_MEMORY_SCOPE_AGENT);
        if (!cross) return;

        const unsigned ctag = call & 0x3fffffu;
        const float scale = P.sx[0] * P.sw[0];
        const rsrc_t yr = mk(pw + OFF_Y + myidx * (512u * 4096u), 512u * 4096u);
        const rsrc_t orr = mk(P.out, OUTB);
        const unsigned obase = quart * 4u + (unsigned)(kq * NS * 4 + c * 16);
        const int p = (q + NPW - NSTAT / 2 % NPW) % NPW;
        const gu32* hp = (const gu32*)(myws + OFF_XF + ((otcls | dj) * 256u + 8 * (NGB - 4)) * 128u);
        u32x4 d[4];
        const gu32* fp = (const gu32*)(pw + OFF_Q + (myidx * 256u + (unsigned)p * 4u + (lane & 3)) * 4u);
        const unsigned long long t0 = rt();
        unsigned spins = 0;
        bool dead = false;
        origin += NSTAT * TAU32 / 32;
        LAS unsigned* op = (LAS unsigned*)smem;
        if (p < 8) {
            unsigned tp = until(origin - WATCH);
            unsigned v[8], ti[8];
#pragma unroll
            for (int i = 0; i < 8; ++i) { v[i] = 0; ti[i] = 0; }
            for (bool found = false; !found && !dead;) {
#pragma unroll
                for (int i = 0; i < 8; ++i) {
                    if (!found && rfl(v[i]) >= call) { found = true; origin = ti[i] + HADJ; }
                    unsigned now;
                    do now = (unsigned)rt(); while ((int)(tp + 11u - now) > 0 && !found);
                    tp = ti[i] = now;
                    v[i] = __hip_atomic_load(hp, __ATOMIC_RELAXED, __HIP_MEMORY_SCOPE_SYSTEM);
                    __builtin_amdgcn_sched_barrier(0);
                }
                if (!found && rt() - t0 > DEADLINE_TICKS) { dead = true; origin = tp; }
            }
            if (lane == 0) __hip_atomic_store(op, origin, __ATOMIC_RELAXED, __HIP_MEMORY_SCOPE_WORKGROUP);
        } else {
            for (;;) {
                origin = rfl(__hip_atomic_load(op, __ATOMIC_RELAXED, __HIP_MEMORY_SCOPE_WORKGROUP));
                if (origin - tstart <= (unsigned)rt() + HADJ - tstart) break;
                __builtin_amdgcn_s_sleep(2);
            }
        }
        unsigned fv = 0;
        for (int k = 0; p + k * NPW < NQ; ++k) {
            unsigned ent = __builtin_amdgcn_readlane(fv, k);
            if ((ent >> 10) != ctag) until(origin + (p + k * NPW) * TAUP32 / 32 - 110u);
            while ((ent >> 10) != ctag && !dead) {
                fv = __hip_atomic_load(fp, __ATOMIC_RELAXED, __HIP_MEMORY_SCOPE_SYSTEM);
                ent = __builtin_amdgcn_readlane(fv, k);
                if ((ent >> 10) != ctag) dead = expired(spins, t0, tmo, call, lane);
            }
            if (dead) break;
            asm volatile("" ::: "memory");
            const unsigned slotT = origin + (p + k * NPW) * TAUP32 / 32;
            const unsigned now = until(slotT);
            if ((int)(now - slotT) > CREDIT) origin += now - slotT - CREDIT;
            const unsigned so = (ent & 511u) * 4096u;
            const unsigned oo = (ent & 255u) * (16u * NS * 4u) + (ent & 256u) + obase;
            if (ent & 512u) {
                const u32x4 lo0 = __builtin_amdgcn_raw_buffer_load_b128(yr, lane * 16, so, SYS);
                const u32x4 lo1 = __builtin_amdgcn_raw_buffer_load_b128(yr, lane * 16 + 1024, so, SYS);
                unsigned hi[3];
#pragma unroll
                for (int g = 0; g < 3; ++g) hi[g] = __builtin_amdgcn_raw_buffer_load_b32(yr, lane * 4 + 2048 + g * 256, so, SYS);
                const unsigned h3 = ((hi[0] >> 6) & 0x03030303u) | ((hi[1] >> 4) & 0x0c0c0c0cu) | ((hi[2] >> 2) & 0x30303030u);
#pragma unroll
                for (int g = 0; g < 4; ++g) {
                    const unsigned h = g < 3 ? hi[g < 3 ? g : 0] : h3;
                    const unsigned l0 = (g < 2 ? lo0 : lo1)[(2 * g) & 3], l1 = (g < 2 ? lo0 : lo1)[(2 * g + 1) & 3];
                    f32x4 y;
                    y[0] = (float)(int)__builtin_amdgcn_sbfe(__builtin_amdgcn_perm(h, l0, 0x0c040100), 0, 22) * scale;
                    y[1] = (float)(int)__builtin_amdgcn_sbfe(__builtin_amdgcn_perm(h, l0, 0x0c050302), 0, 22) * scale;
                    y[2] = (float)(int)__builtin_amdgcn_sbfe(__builtin_amdgcn_perm(h, l1, 0x0c060100), 0, 22) * scale;
                    y[3] = (float)(int)__builtin_amdgcn_sbfe(__builtin_amdgcn_perm(h, l1, 0x0c070302), 0, 22) * scale;
                    d[g] = __builtin_bit_cast(u32x4, y);
                }
            } else {
#pragma unroll
                for (int j = 0; j < 4; ++j) d[j] = __builtin_amdgcn_raw_buffer_load_b128(yr, lane * 16 + j * 1024, so, SYS);
                origin += (TAU32 - TAUP32) * NPW / 32;
            }
#pragma unroll
            for (int g = 0; g < 4; ++g) __builtin_amdgcn_raw_buffer_store_b128(d[g], orr, oo + g * 4 * NS * 4, 0, SYS);
        }
        if (dead && lane == 0)
            __hip_atomic_store((gu32*)(P.out + (size_t)rank * MS * NS * 4), 0x7fc00000u, __ATOMIC_RELAXED, __HIP_MEMORY_SCOPE_SYSTEM);
        LSTAMP(63);
        return;
    }
    if (b < 64) return;

    const int xcd = b & 7, slot = (b - 64) >> 3;
    const int rs = 3 * (xcd >> 1) + (slot >> 3);
    const bool remote = (slot & 4) != 0;
    const unsigned dj = (unsigned)slot & 3u;
    const unsigned cblk = (unsigned)xcd & 1u;
    const unsigned owner = remote ? (otcls | dj) : rank;
    const unsigned col0 = (cblk * 4u + dj) * 64u;
    LAS int* tcnt = (LAS int*)smem + MAXT * 16 * TSTR;

    u32x4 wf[8][4];
    if (!remote) {
        const unsigned wv = (unsigned)(kq * 16 * NS + c * 4);
#pragma unroll
        for (int s = 0; s < 8; ++s) {
            unsigned r[16];
            const unsigned so = (unsigned)(wave * 8 + s) * 64u * NS + col0;
#pragma unroll
            for (int i = 0; i < 16; ++i) r[i] = __builtin_amdgcn_raw_buffer_load_b32(wr, wv + i * NS, so, 0);
            transpose16x4(r, wf[s]);
        }
    }
    for (int i = threadIdx.x; i < MAXT * 16 * TSTR + 32 + SMEM_EXTRA; i += NWAVES * 64) smem[i] = 0;
    __syncthreads();
    const unsigned call = rfl(cntv) + 1;
    if (lane == 0) __hip_atomic_store(cntp, call, __ATOMIC_RELAXED, __HIP_MEMORY_SCOPE_AGENT);
    const float scale = P.sx[0] * P.sw[0];
    const unsigned pbase = (call & 1u) * PAR_BYTES;
    const rsrc_t sr = mk(myws + pbase, PAR_BYTES);
    const rsrc_t orr = remote ? mk(myws + OFF_Y + dj * (512u * 4096u), 512u * 4096u) : mk(P.out, OUTB);
    const unsigned long long t0 = rt();
    unsigned spins = 0;
    bool dead = false;

    unsigned tsrc, trb;
    bool tvalid;
    {
        const unsigned idx = (unsigned)rs + RS * (unsigned)lane;
        constexpr unsigned E1 = 3 * NIN, E2 = E1 + NAB, E3 = E2 + 4 * NGB;
        if (idx < E1) {
            const unsigned u = idx / 3u;
            tsrc = mycls | ((myidx + 1u + idx % 3u) & 3u); trb = u < NAB ? NGB + u : u - (NAB - NGB + NDL); tvalid = !remote || u < NAB;
        } else if (idx < E2) {
            tsrc = rank; trb = NGB + idx - E1; tvalid = true;
        } else if (idx < E3) {
            const unsigned u = idx - E2;
            tsrc = otcls | (u & 3u); trb = u >> 2; tvalid = !remote || (tsrc != owner && trb < NGB - NDL);
        } else {
            tsrc = rank; trb = (idx - E3) & 31u; tvalid = !remote && idx < NLIST;
        }
    }
    const unsigned tdesc = tsrc * 32u + trb;
    const unsigned long long validmask = __ballot(tvalid);
    const unsigned long long ownmask = __ballot(tvalid && tsrc == rank);
    const gu32* flagp = (tvalid && tsrc != rank) ? (const gu32*)(myws + OFF_XF + (tdesc * 8u + (unsigned)wave) * 128u) : tmo;
    const unsigned vown = (unsigned)(c * KD + kq * 16), vrem = (unsigned)(lane * 16);

    unsigned pend = (unsigned)validmask;
    auto refill = [&](unsigned ready, u32x4 (&a)[8]) {
        const unsigned todo = ready & pend;
        const int t = todo ? __builtin_ctz(todo) : -1;
        pend &= ~(todo & (0u - todo));
        const unsigned desc = __builtin_amdgcn_readlane(tdesc, t & 63);
        const bool own = (desc >> 5) == rank;
        const rsrc_t r = mk(own ? (const char*)P.x : myws + pbase, todo ? (own ? SHARD : OFF_W) : 0u);
        const unsigned so = own ? (desc & 31u) * 65536u + (unsigned)wave * 512u : desc * 65536u + (unsigned)wave * 8192u;
        const unsigned step = own ? 64u : 1024u;
        const unsigned vo = own ? vown : vrem;
#pragma unroll
        for (int s = 0; s < 8; ++s) a[s] = __builtin_amdgcn_raw_buffer_load_b128(r, vo, so + s * step, SYS);
        if (t >= 0) CSTAMP(24 + 3 * t);
        return t;
    };
    auto arrive = [&](int t, bool take) {
        asm volatile("s_waitcnt lgkmcnt(0)" ::: "memory");
        int old = 0;
        if (lane == 0) old = __hip_atomic_fetch_add(tcnt + t, 1, __ATOMIC_RELAXED, __HIP_MEMORY_SCOPE_WORKGROUP);
        old = rfl(old);
        if (old == NWAVES - 1 && lane == 0) __hip_atomic_fetch_or(tcnt + (take ? 18 : 16), 1 << t, __ATOMIC_RELAXED, __HIP_MEMORY_SCOPE_WORKGROUP);
        asm volatile("" ::: "memory");
        return old;
    };
    auto store_tile = [&](int t) {
        unsigned ticket = 0;
        if (remote) asm volatile("s_mov_b64 exec, 1\n\tglobal_atomic_add %0, %1, %2, off sc0\n\ts_mov_b64 exec, -1"
                                 : "+v"(ticket) : "v"(myws + OFF_QT + dj * 128u), "v"(1u) : "memory");
        const unsigned desc = __builtin_amdgcn_readlane(tdesc, t);
        const LAS int* tp = (const LAS int*)smem + t * 16 * TSTR + kq * TSTR + c * 4;
        unsigned v[16];
#pragma unroll
        for (int i = 0; i < 16; ++i) v[i] = (unsigned)tp[(i >> 2) * 4 * TSTR + (i & 3)];
        const bool bad = __hip_atomic_load(tcnt + 17, __ATOMIC_RELAXED, __HIP_MEMORY_SCOPE_WORKGROUP) != 0;
        const float sc = bad ? __builtin_nanf("") : scale;
        unsigned span = 0;
#pragma unroll
        for (int i = 0; i < 16; ++i) span |= v[i] + 0x200000u;
        const bool pack = remote && __all((span >> 22) == 0u && !bad);
        const unsigned id = cblk * 256u + desc;
        if (pack) {
            unsigned h[4];
            u32x4 lo[2];
#pragma unroll
            for (int g = 0; g < 4; ++g) {
                h[g] = __builtin_amdgcn_perm(v[4 * g + 1], v[4 * g], 0x0c0c0602) | __builtin_amdgcn_perm(v[4 * g + 3], v[4 * g + 2], 0x06020c0c);
                lo[g >> 1][(2 * g) & 3] = __builtin_amdgcn_perm(v[4 * g + 1], v[4 * g], 0x05040100);
                lo[g >> 1][(2 * g + 1) & 3] = __builtin_amdgcn_perm(v[4 * g + 3], v[4 * g + 2], 0x05040100);
            }
            __builtin_amdgcn_raw_buffer_store_b128(lo[0], orr, id * 4096u + lane * 16, 0, SYS);
            __builtin_amdgcn_raw_buffer_store_b128(lo[1], orr, id * 4096u + 1024 + lane * 16, 0, SYS);
#pragma unroll
            for (int g = 0; g < 3; ++g)
                __builtin_amdgcn_raw_buffer_store_b32((h[g] & 0x3f3f3f3fu) | ((h[3] << (6 - 2 * g)) & 0xc0c0c0c0u), orr,
                                                      id * 4096u + 2048 + g * 256 + lane * 4, 0, SYS);
        } else {
            const unsigned oo = remote ? id * 4096u + lane * 16 : (desc * 16u * NS + col0) * 4u + (unsigned)(kq * NS * 4 + c * 16);
            const unsigned st = remote ? 1024u : 4u * NS * 4u;
#pragma unroll
            for (int i = 0; i < 4; ++i) {
                f32x4 y;
#pragma unroll
                for (int e = 0; e < 4; ++e) y[e] = (float)(int)v[4 * i + e] * sc;
                __builtin_amdgcn_raw_buffer_store_b128(__builtin_bit_cast(u32x4, y), orr, oo + i * st, 0, SYS);
            }
        }
        if (remote) {
            asm volatile("s_waitcnt vmcnt(0)" : "+v"(ticket) :: "memory");
            const unsigned slot = ticket - (call - 1u) * NQ;
            if (lane == 0)
                __hip_atomic_store((gu32*)(myws + OFF_Q + dj * 1024u) + ((slot & 63u) * 4u + ((slot >> 6) & 3u)),
                                   ((call & 0x3fffffu) << 10) | (pack ? 512u : 0u) | id, __ATOMIC_RELAXED, __HIP_MEMORY_SCOPE_SYSTEM);
        }
        CSTAMP(8 + t);
    };
    u32x4 a0[8], a1[8];
    int t0i = refill((unsigned)ownmask, a0);
    int t1i = refill((unsigned)ownmask, a1);
    if (remote) {
        const unsigned u0 = (dj * 2u + cblk) * 64u + (unsigned)wave * 8u;
        const gu32* fp = (const gu32*)(myws + OFF_WF + u0 * 16u + ((lane >> 1) & 3) * 4u);
        unsigned have = 0;
#pragma unroll
        for (int s = 0; s < 8; ++s) {
            while (((have >> s) & 1u) == 0u && !dead) {
                have = (unsigned)__ballot(__hip_atomic_load(fp, __ATOMIC_RELAXED, __HIP_MEMORY_SCOPE_SYSTEM) >= call);
                if (((have >> s) & 1u) == 0u) dead = expired(spins, t0, tmo, call, lane);
            }
            asm volatile("" ::: "memory");
#pragma unroll
            for (int e = 0; e < 4; ++e)
                wf[s][e] = __builtin_amdgcn_raw_buffer_load_b128(sr, lane * 16 + e * 1024, OFF_W + (u0 + s) * 4096u, SYS);
        }
#pragma unroll
        for (int s = 0; s < 8; s += 2)
            asm volatile("" : "+v"(wf[s][0]), "+v"(wf[s][1]), "+v"(wf[s][2]), "+v"(wf[s][3]),
                              "+v"(wf[s + 1][0]), "+v"(wf[s + 1][1]), "+v"(wf[s + 1][2]), "+v"(wf[s + 1][3]));
    }
    CSTAMP(1);

    unsigned fv = __hip_atomic_load(flagp, __ATOMIC_RELAXED, __HIP_MEMORY_SCOPE_SYSTEM);
    asm volatile("" : "+v"(fv));
    auto claim = [&]() {
        const unsigned fin = rfl(__hip_atomic_load(tcnt + 16, __ATOMIC_RELAXED, __HIP_MEMORY_SCOPE_WORKGROUP));
        const unsigned clm = rfl(__hip_atomic_load(tcnt + 18, __ATOMIC_RELAXED, __HIP_MEMORY_SCOPE_WORKGROUP));
        if ((fin & ~clm) == 0u) return clm == (unsigned)validmask ? -2 : -1;
        const int ts = __builtin_ctz(fin & ~clm);
        int old = 0;
        if (lane == 0) old = __hip_atomic_fetch_or(tcnt + 18, 1 << ts, __ATOMIC_RELAXED, __HIP_MEMORY_SCOPE_WORKGROUP);
        asm volatile("" ::: "memory");
        return ((rfl(old) >> ts) & 1u) != 0u ? -1 : ts;
    };
    auto mma = [&](const u32x4 (&a)[8], i32x4 (&acc)[4]) {
#pragma unroll
        for (int e = 0; e < 4; ++e) acc[e] = (i32x4){0, 0, 0, 0};
#pragma unroll
        for (int s = 0; s < 8; ++s)
#pragma unroll
            for (int e = 0; e < 4; ++e)
                acc[e] = __builtin_amdgcn_mfma_i32_16x16x64_i8(__builtin_bit_cast(i32x4, wf[s][e]), __builtin_bit_cast(i32x4, a[s]), acc[e], 0, 0, 0);
    };
    auto add_tile = [&](int t, const i32x4 (&acc)[4], bool take) {
        LAS int* tp = (LAS int*)smem + t * 16 * TSTR + c * TSTR + kq * 16;
#pragma unroll
        for (int r = 0; r < 4; ++r)
#pragma unroll
            for (int e = 0; e < 4; ++e)
                __hip_atomic_fetch_add(tp + r * 4 + e, acc[e][r], __ATOMIC_RELAXED, __HIP_MEMORY_SCOPE_WORKGROUP);
        const int pos = arrive(t, take);
        CSTAMP(25 + 3 * t);
        return pos;
    };
    auto ready = [&]() {
        if (__builtin_amdgcn_readlane(fv, 63) == call) dead = true;
        const unsigned r = (unsigned)__ballot(fv >= call) | (unsigned)ownmask;
        asm volatile("" ::: "memory");
        return dead ? 0u : r;
    };
    auto half = [&](u32x4 (&a)[8], int& t, int other) {
        i32x4 acc[4];
        const int tc = t;
        if (tc >= 0) mma(a, acc);
        const unsigned rdy = ready();
        fv = __hip_atomic_load(flagp, __ATOMIC_RELAXED, __HIP_MEMORY_SCOPE_SYSTEM);
        t = refill(rdy, a);
        __builtin_amdgcn_sched_barrier(0);
        if (tc >= 0) {
            const bool busy = t >= 0 || other >= 0;
            const int pos = add_tile(tc, acc, !busy);
            if (busy ? pos < AHEAD : pos == NWAVES - 1) {
                const int ts = busy ? claim() : tc;
                if (ts >= 0) store_tile(ts);
            }
        }
    };
    for (;;) {
        do {
            half(a0, t0i, t1i);
            half(a1, t1i, t0i);
        } while (t0i >= 0 || t1i >= 0);
        unsigned rdy;
        for (;;) {
            rdy = ready() & pend;
            if ((rdy & (rdy - 1u)) != 0u) break;
            int ts;
            if (rdy != 0u) {
                const int t = refill(rdy, a0);
                fv = __hip_atomic_load(flagp, __ATOMIC_RELAXED, __HIP_MEMORY_SCOPE_SYSTEM);
                __builtin_amdgcn_sched_barrier(0);
                i32x4 acc[4];
                mma(a0, acc);
                ts = add_tile(t, acc, true) == NWAVES - 1 ? t : -1;
            } else {
                if (dead && pend != 0u) {
                    if (lane == 0) __hip_atomic_store(tcnt + 17, 1, __ATOMIC_RELAXED, __HIP_MEMORY_SCOPE_WORKGROUP);
                    for (; pend != 0u; pend &= pend - 1u) arrive(__builtin_ctz(pend), false);
                }
                if (pend != 0u) {
                    if (expired(spins, t0, tmo, call, lane)) dead = true;
                    fv = __hip_atomic_load(flagp, __ATOMIC_RELAXED, __HIP_MEMORY_SCOPE_SYSTEM);
                } else {
                    __builtin_amdgcn_s_sleep(1);
                }
                ts = claim();
                if (ts == -2) {
                    CSTAMP(2);
                    LSTAMP(63);
                    asm volatile("s_endpgm");
                }
            }
            asm volatile("" : "+v"(fv));
            if (ts >= 0) store_tile(ts);
        }
        t0i = refill(rdy, a0);
        t1i = refill(rdy, a1);
    }
}

extern "C" void kernel_launch(int rank, int nranks, ncclComm_t comm,
                              hipStream_t stream,
                              void* const* d_in, const int* in_sizes, int n_in,
                              void* d_out, int out_size,
                              const int* in_dtypes, int out_dtype,
                              void* const* d_out_all,
                              void* const* d_ws_all, size_t ws_bytes) {
    if (nranks != NR || (unsigned)rank >= (unsigned)NR || n_in != 4 || in_sizes[0] != MS * KD || in_sizes[1] != KD * NS
        || out_size != NR * MS * NS || ws_bytes < WS_NEED) {
        fprintf(stderr, "kernel_launch: unexpected shapes\n");
        return;
    }
    Params P;
    P.x = (const signed char*)d_in[0];
    P.w = (const signed char*)d_in[1];
    P.sx = (const float*)d_in[2];
    P.sw = (const float*)d_in[3];
    for (int i = 0; i < NR; ++i) {
        const int p = i < 3 ? ((rank & 4) | ((rank + 1 + i) & 3)) : (i < 7 ? (((rank & 4) ^ 4) | (i - 3)) : rank);
        P.ws[i] = (char*)d_ws_all[p];
    }
    P.out = (char*)d_out;
    P.rank = rank;
    P.pad = 0;
    ag_gemm_kernel<<<GRID, NWAVES * 64, 0, stream>>>(P);
    HIP_CHECK(hipGetLastError());
}
```

```cpp
#include <hip/hip_runtime.h>
#include <rccl/rccl.h>
#include <cstdio>
#include <cstdint>

#define HIP_CHECK(expr) do { hipError_t _e = (expr); if (_e != hipSuccess) { \
    fprintf(stderr, "HIP error %s at %s:%d\n", hipGetErrorString(_e), __FILE__, __LINE__); \
    return; } } while (0)

#define GAS __attribute__((address_space(1)))
#define LAS __attribute__((address_space(3)))
typedef unsigned u32x4 __attribute__((ext_vector_type(4)));
typedef int i32x4 __attribute__((ext_vector_type(4)));
typedef float f32x4 __attribute__((ext_vector_type(4)));
typedef GAS unsigned gu32;
typedef GAS unsigned long long gu64;
typedef __amdgpu_buffer_rsrc_t rsrc_t;

#define NGB 9
#define PW 8
#define TAU32 226
#define TAUI32 224
#define T_FIRSTI 70
#define TAUP32 152
#define T_FIRST 80
#define ACK1 150
#define WIN 3
#define WINP 4
#define WMAX 400
#define LEAD 200
#define CREDIT 60
#define NDL 2
#define WATCH 260
#define HADJ 280
#define POLL_SLEEP 1
#define AHEAD 3
constexpr int NR = 8;
constexpr int MS = 512;
constexpr int KD = 4096;
constexpr int NS = 512;
constexpr int NAB = 32 - NGB;
constexpr int GRID = 256;
constexpr int NWAVES = 8;
constexpr int NPUSH = 56;
constexpr int NPW = 8 * PW;
constexpr int RS = 12;
constexpr int NIN = NAB + NDL;
constexpr int NLIST = 128 + NGB + 3 * NDL;
constexpr int MAXT = (NLIST + RS - 1) / RS;
constexpr int TSTR = 65;
constexpr unsigned SHARD = MS * KD;
constexpr unsigned OUTB = NR * MS * NS * 4;
constexpr unsigned PAR_BYTES = (NR + 1) * SHARD;
constexpr unsigned OFF_W = NR * SHARD;
constexpr unsigned OFF_Y = 2 * PAR_BYTES;
constexpr unsigned OFF_XF = OFF_Y + 8 * 256 * 4096;
constexpr unsigned OFF_WF = OFF_XF + NR * 32 * 8 * 128;
constexpr unsigned OFF_Q = OFF_WF + 8192;
constexpr unsigned OFF_QT = OFF_Q + 4096;
constexpr unsigned OFF_CNT = OFF_QT + 4096;
constexpr unsigned OFF_TMO = OFF_CNT + GRID * 128;
constexpr unsigned OFF_DBG = OFF_TMO + 4096;
constexpr unsigned WS_NEED = OFF_DBG + GRID * NWAVES * 64 * 8;
constexpr int NQ = 2 * (4 * NAB + 3 * (NGB - NDL));
constexpr int NSTAT = 128 + 16 * NGB;
static_assert(NPW == 64 && NQ <= 256 && MAXT <= 13 && NDL <= NGB, "flag / queue layout");
constexpr unsigned long long DEADLINE_TICKS = 50000000ull;
constexpr int SYS = 17;

struct Params {
    const signed char* x;
    const signed char* w;
    const float* sx;
    const float* sw;
    char* ws[NR];
    char* out;
    int rank;
    int pad;
};

__device__ __forceinline__ unsigned long long rt() { return __builtin_amdgcn_s_memrealtime(); }
__device__ __forceinline__ rsrc_t mk(const void* p, unsigned bytes) {
    return __builtin_amdgcn_make_buffer_rsrc((void*)p, 0, (int)bytes, 0x00020000);
}
__device__ __forceinline__ unsigned rfl(unsigned v) { return __builtin_amdgcn_readfirstlane(v); }

#define STAMP(i) do { } while (0)
#define CSTAMP(i) do { } while (0)
#define SMEM_EXTRA 0
#define LSTAMP(i) do { } while (0)

__device__ __forceinline__ void transpose16x4(const unsigned (&r)[16], u32x4 (&o)[4]) {
#pragma unroll
    for (int q = 0; q < 4; ++q) {
        const unsigned r0 = r[4 * q], r1 = r[4 * q + 1], r2 = r[4 * q + 2], r3 = r[4 * q + 3];
        const unsigned lo01 = __builtin_amdgcn_perm(r1, r0, 0x05010400);
        const unsigned hi01 = __builtin_amdgcn_perm(r1, r0, 0x07030602);
        const unsigned lo23 = __builtin_amdgcn_perm(r3, r2, 0x05010400);
        const unsigned hi23 = __builtin_amdgcn_perm(r3, r2, 0x07030602);
        o[0][q] = __builtin_amdgcn_perm(lo23, lo01, 0x05040100);
        o[1][q] = __builtin_amdgcn_perm(lo23, lo01, 0x07060302);
        o[2][q] = __builtin_amdgcn_perm(hi23, hi01, 0x05040100);
        o[3][q] = __builtin_amdgcn_perm(hi23, hi01, 0x07060302);
    }
}

__device__ __forceinline__ bool expired(unsigned& spins, unsigned long long t0, gu32* tmo, unsigned call, int lane) {
    if (POLL_SLEEP) __builtin_amdgcn_s_sleep(POLL_SLEEP);
    spins = rfl(spins + 1u);
    if ((spins & 63u) != 0u) return false;
    const bool late = rfl(__hip_atomic_load(tmo, __ATOMIC_RELAXED, __HIP_MEMORY_SCOPE_AGENT)) == call
                      || rfl(rt() - t0 > DEADLINE_TICKS ? 1u : 0u) != 0u;
    if (late) __hip_atomic_store(tmo, call, __ATOMIC_RELAXED, __HIP_MEMORY_SCOPE_AGENT);
    return late;
}

__global__ void __launch_bounds__(NWAVES * 64, 2) ag_gemm_kernel(Params P) {
    __shared__ __attribute__((aligned(16))) int smem[MAXT * 16 * TSTR + 32 + SMEM_EXTRA];
    const int b = blockIdx.x;
    const int lane = threadIdx.x & 63;
    const int wave = rfl(threadIdx.x >> 6);
    const unsigned rank = (unsigned)P.rank & 7u;
    const unsigned myidx = rank & 3u, mycls = rank & 4u, otcls = mycls ^ 4u;
    char* myws = P.ws[7];
    gu32* cntp = (gu32*)(myws + OFF_CNT + b * 128 + wave * 4);
    gu32* tmo = (gu32*)(myws + OFF_TMO);
    LSTAMP(0);
    const unsigned cntv = __hip_atomic_load(cntp, __ATOMIC_RELAXED, __HIP_MEMORY_SCOPE_AGENT);
    const rsrc_t xr = mk(P.x, SHARD);
    const rsrc_t wr = mk(P.w, SHARD);
    const int c = lane & 15, kq = lane >> 4;

    if (b < NPUSH) {
        const unsigned tstart = (unsigned)rt();
        const unsigned di = (unsigned)b % 7u, cu = (unsigned)b / 7u;
        const bool cross = di >= 3u;
        const unsigned dj = (di - 3u) & 3u;
        char* pw = P.ws[di];
        const rsrc_t dr = mk(pw, OFF_Y);
        const unsigned xv = (unsigned)(c * KD + kq * 16);
        const unsigned quart = ((myidx + dj) & 3u) * 128u;
        const unsigned wv = (unsigned)(kq * 16 * NS + c * 4) + quart;
        const int npairs = cross ? NSTAT / 2 : 8 * NIN;
        auto until = [&](unsigned t) {
            unsigned now = (unsigned)rt();
            while ((int)(t - now) > 0) {
                if ((int)(t - now) > 8) __builtin_amdgcn_s_sleep(1);
                now = (unsigned)rt();
            }
            return now;
        };
        const int q = wave * 8 + (int)cu;
        unsigned origin = tstart + (cross ? T_FIRST : T_FIRSTI);
        const int tau = cross ? TAU32 : TAUI32;
        LAS unsigned* op = (LAS unsigned*)smem;
        if (threadIdx.x == 0) { op[1] = 0; op[2] = 0; op[3] = 0; }
        unsigned sh = 0;
        auto gate = [&](unsigned t, int need, bool shifted) {
            unsigned now;
            for (;;) {
                const int fin = (int)rfl(__hip_atomic_load(op + 1, __ATOMIC_RELAXED, __HIP_MEMORY_SCOPE_WORKGROUP));
                if (shifted) sh = rfl(__hip_atomic_load(op + 3, __ATOMIC_RELAXED, __HIP_MEMORY_SCOPE_WORKGROUP)) & 511u;
                now = (unsigned)rt();
                const int left = (int)(t + (shifted ? sh : 0u) - now);
                if (left <= 0 && (fin >= need || left < -WMAX)) break;
                if (left > 12) __builtin_amdgcn_s_sleep(1);
            }
            return now;
        };
        auto finished = [&](int n) {
            if (lane == 0) __hip_atomic_fetch_max((LAS int*)op + 1, n, __ATOMIC_RELAXED, __HIP_MEMORY_SCOPE_WORKGROUP);
        };
        unsigned call = 0;
        for (int g = q; g < npairs; g += NPW) {
            const int sl = 2 * g - (int)cu;
            until(origin + sl * tau / 32 - LEAD);
            const bool isw = cross && g < 64;
            const unsigned m = (unsigned)(cross ? g - 64 : (g < 8 * NAB ? g + 8 * NGB : g - 8 * (NAB - NGB + NDL)));
            u32x4 e[2][4];
            if (isw) {
                const unsigned so = (unsigned)(g & 31) * 128u * NS + (unsigned)(g >> 5) * 64u;
#pragma unroll
                for (int u = 0; u < 2; ++u) {
                    unsigned r[16];
#pragma unroll
                    for (int i = 0; i < 16; ++i) r[i] = __builtin_amdgcn_raw_buffer_load_b32(wr, wv + i * NS, so + u * 64 * NS, 0);
                    transpose16x4(r, e[u]);
                }
            } else {
                const unsigned so = (m >> 3) * 16u * KD + (m & 7u) * 512u;
#pragma unroll
                for (int u = 0; u < 2; ++u)
#pragma unroll
                    for (int j = 0; j < 4; ++j) e[u][j] = __builtin_amdgcn_raw_buffer_load_b128(xr, xv + u * 256 + j * 64, so, 0);
            }
            call = rfl(cntv) + 1;
            const unsigned gw = myidx * 64u + (unsigned)g;
            const unsigned so = (call & 1u) * PAR_BYTES + (isw ? OFF_W + gw * 8192u : rank * SHARD + m * 8192u) + lane * 16;
            const unsigned fo = isw ? OFF_WF + (gw >> 2) * 128u + (gw & 3u) * 4u : OFF_XF + (rank * 256u + m) * 128u;
#pragma unroll
            for (int u = 0; u < 2; ++u) asm volatile("" : "+v"(e[u][0]), "+v"(e[u][1]), "+v"(e[u][2]), "+v"(e[u][3]));
            if (g == q) STAMP(1);
            const int j0 = (g >> 3) * 2;
#pragma unroll
            for (int u = 0; u < 2; ++u) {
                const unsigned t = origin + (sl + 8 * u) * tau / 32;
                const unsigned now = gate(t, j0 + u + 1 - WIN, true);
#pragma unroll
                for (int j = 0; j < 4; ++j) __builtin_amdgcn_raw_buffer_store_b128(e[u][j], dr, so + u * 4096 + j * 1024, 0, SYS);
                if (u == 1 && lane == 0) op[2] = now - t;
            }
            asm volatile("s_waitcnt vmcnt(4)" ::: "memory");
            finished(j0 + 1);
            if (g < 8) {
                const int lag = (int)((unsigned)rt() - origin) - ACK1;
                if (lag > 0 && lane == 0) op[3] = lag < 511 ? lag : 511;
            }
            asm volatile("s_waitcnt vmcnt(0)" ::: "memory");
            if (lane == 0) __hip_atomic_store((gu32*)(pw + fo), call, __ATOMIC_RELAXED, __HIP_MEMORY_SCOPE_SYSTEM);
            finished(j0 + 2);
        }
        call = rfl(cntv) + 1;
        if (lane == 0) __hip_atomic_store(cntp, call, __ATOMIC_RELAXED, __HIP_MEMORY_SCOPE_AGENT);
        if (!cross) return;

        const unsigned ctag = call & 0x3fffffu;
        const float scale = P.sx[0] * P.sw[0];
        const rsrc_t yr = mk(pw + OFF_Y + myidx * (512u * 4096u), 512u * 4096u);
        const rsrc_t orr = mk(P.out, OUTB);
        const unsigned obase = quart * 4u + (unsigned)(kq * NS * 4 + c * 16);
        const int p = (q + NPW - NSTAT / 2 % NPW) % NPW;
        const gu32* hp = (const gu32*)(myws + OFF_XF + ((otcls | dj) * 256u + 8 * (NGB - 4)) * 128u);
        u32x4 d[4];
        const gu32* fp = (const gu32*)(pw + OFF_Q + (myidx * 256u + (unsigned)p * 4u + (lane & 3)) * 4u);
        const unsigned long long t0 = rt();
        unsigned spins = 0;
        bool dead = false;
        origin += NSTAT * TAU32 / 32;
        if (p < 8) {
            until(origin - WATCH);
            unsigned tp = until(origin - WATCH + (rfl(op[2]) & 1023u));
            unsigned v[8], ti[8];
#pragma unroll
            for (int i = 0; i < 8; ++i) { v[i] = 0; ti[i] = 0; }
            for (bool found = false; !found && !dead;) {
#pragma unroll
                for (int i = 0; i < 8; ++i) {
                    if (!found && rfl(v[i]) >= call) { found = true; origin = ti[i] + HADJ; }
                    unsigned now;
                    do now = (unsigned)rt(); while ((int)(tp + 11u - now) > 0 && !found);
                    tp = ti[i] = now;
                    v[i] = __hip_atomic_load(hp, __ATOMIC_RELAXED, __HIP_MEMORY_SCOPE_SYSTEM);
                    __builtin_amdgcn_sched_barrier(0);
                }
                if (!found && rt() - t0 > DEADLINE_TICKS) { dead = true; origin = tp; }
            }
            if (lane == 0) __hip_atomic_store(op, origin, __ATOMIC_RELAXED, __HIP_MEMORY_SCOPE_WORKGROUP);
        } else {
            for (;;) {
                origin = rfl(__hip_atomic_load(op, __ATOMIC_RELAXED, __HIP_MEMORY_SCOPE_WORKGROUP));
                if (origin - tstart <= (unsigned)rt() + HADJ - tstart) break;
                __builtin_amdgcn_s_sleep(2);
            }
        }
        unsigned fv = 0;
        for (int k = 0; p + k * NPW < NQ; ++k) {
            unsigned ent = __builtin_amdgcn_readlane(fv, k);
            if ((ent >> 10) != ctag) until(origin + (p + k * NPW) * TAUP32 / 32 - 110u);
            while ((ent >> 10) != ctag && !dead) {
                fv = __hip_atomic_load(fp, __ATOMIC_RELAXED, __HIP_MEMORY_SCOPE_SYSTEM);
                ent = __builtin_amdgcn_readlane(fv, k);
                if ((ent >> 10) != ctag) dead = expired(spins, t0, tmo, call, lane);
            }
            if (dead) break;
            asm volatile("" ::: "memory");
            const unsigned slotT = origin + (p + k * NPW) * TAUP32 / 32;
            const int jp = NSTAT / 8 + (p >> 3) + k * 8;
            const unsigned now = gate(slotT, jp + 1 - WINP, false);
            if ((int)(now - slotT) > CREDIT) origin += now - slotT - CREDIT;
            const unsigned so = (ent & 511u) * 4096u;
            const unsigned oo = (ent & 255u) * (16u * NS * 4u) + (ent & 256u) + obase;
            if (ent & 512u) {
                const u32x4 lo0 = __builtin_amdgcn_raw_buffer_load_b128(yr, lane * 16, so, SYS);
                const u32x4 lo1 = __builtin_amdgcn_raw_buffer_load_b128(yr, lane * 16 + 1024, so, SYS);
                unsigned hi[3];
#pragma unroll
                for (int g = 0; g < 3; ++g) hi[g] = __builtin_amdgcn_raw_buffer_load_b32(yr, lane * 4 + 2048 + g * 256, so, SYS);
                const unsigned h3 = ((hi[0] >> 6) & 0x03030303u) | ((hi[1] >> 4) & 0x0c0c0c0cu) | ((hi[2] >> 2) & 0x30303030u);
#pragma unroll
                for (int g = 0; g < 4; ++g) {
                    const unsigned h = g < 3 ? hi[g < 3 ? g : 0] : h3;
                    const unsigned l0 = (g < 2 ? lo0 : lo1)[(2 * g) & 3], l1 = (g < 2 ? lo0 : lo1)[(2 * g + 1) & 3];
                    f32x4 y;
                    y[0] = (float)(int)__builtin_amdgcn_sbfe(__builtin_amdgcn_perm(h, l0, 0x0c040100), 0, 22) * scale;
                    y[1] = (float)(int)__builtin_amdgcn_sbfe(__builtin_amdgcn_perm(h, l0, 0x0c050302), 0, 22) * scale;
                    y[2] = (float)(int)__builtin_amdgcn_sbfe(__builtin_amdgcn_perm(h, l1, 0x0c060100), 0, 22) * scale;
                    y[3] = (float)(int)__builtin_amdgcn_sbfe(__builtin_amdgcn_perm(h, l1, 0x0c070302), 0, 22) * scale;
                    d[g] = __builtin_bit_cast(u32x4, y);
                }
            } else {
#pragma unroll
                for (int j = 0; j < 4; ++j) d[j] = __builtin_amdgcn_raw_buffer_load_b128(yr, lane * 16 + j * 1024, so, SYS);
                origin += (TAU32 - TAUP32) * NPW / 32;
            }
            asm volatile("" : "+v"(d[0]), "+v"(d[1]), "+v"(d[2]), "+v"(d[3]));
            finished(jp + 1);
#pragma unroll
            for (int g = 0; g < 4; ++g) __builtin_amdgcn_raw_buffer_store_b128(d[g], orr, oo + g * 4 * NS * 4, 0, SYS);
        }
        if (dead && lane == 0)
            __hip_atomic_store((gu32*)(P.out + (size_t)rank * MS * NS * 4), 0x7fc00000u, __ATOMIC_RELAXED, __HIP_MEMORY_SCOPE_SYSTEM);
        LSTAMP(63);
        return;
    }
    if (b < 64) return;

    const int xcd = b & 7, slot = (b - 64) >> 3;
    const int rs = 3 * (xcd >> 1) + (slot >> 3);
    const bool remote = (slot & 4) != 0;
    const unsigned dj = (unsigned)slot & 3u;
    const unsigned cblk = (unsigned)xcd & 1u;
    const unsigned owner = remote ? (otcls | dj) : rank;
    const unsigned col0 = (cblk * 4u + dj) * 64u;
    LAS int* tcnt = (LAS int*)smem + MAXT * 16 * TSTR;

    u32x4 wf[8][4];
    if (!remote) {
        const unsigned wv = (unsigned)(kq * 16 * NS + c * 4);
#pragma unroll
        for (int s = 0; s < 8; ++s) {
            unsigned r[16];
            const unsigned so = (unsigned)(wave * 8 + s) * 64u * NS + col0;
#pragma unroll
            for (int i = 0; i < 16; ++i) r[i] = __builtin_amdgcn_raw_buffer_load_b32(wr, wv + i * NS, so, 0);
            transpose16x4(r, wf[s]);
        }
    }
    for (int i = threadIdx.x; i < MAXT * 16 * TSTR + 32 + SMEM_EXTRA; i += NWAVES * 64) smem[i] = 0;
    __syncthreads();
    const unsigned call = rfl(cntv) + 1;
    if (lane == 0) __hip_atomic_store(cntp, call, __ATOMIC_RELAXED, __HIP_MEMORY_SCOPE_AGENT);
    const float scale = P.sx[0] * P.sw[0];
    const unsigned pbase = (call & 1u) * PAR_BYTES;
    const rsrc_t sr = mk(myws + pbase, PAR_BYTES);
    const rsrc_t orr = remote ? mk(myws + OFF_Y + dj * (512u * 4096u), 512u * 4096u) : mk(P.out, OUTB);
    const unsigned long long t0 = rt();
    unsigned spins = 0;
    bool dead = false;

    unsigned tsrc, trb;
    bool tvalid;
    {
        const unsigned idx = (unsigned)rs + RS * (unsigned)lane;
        constexpr unsigned E1 = 3 * NIN, E2 = E1 + NAB, E3 = E2 + 4 * NGB;
        if (idx < E1) {
            const unsigned u = idx / 3u;
            tsrc = mycls | ((myidx + 1u + idx % 3u) & 3u); trb = u < NAB ? NGB + u : u - (NAB - NGB + NDL); tvalid = !remote || u < NAB;
        } else if (idx < E2) {
            tsrc = rank; trb = NGB + idx - E1; tvalid = true;
        } else if (idx < E3) {
            const unsigned u = idx - E2;
            tsrc = otcls | (u & 3u); trb = u >> 2; tvalid = !remote || (tsrc != owner && trb < NGB - NDL);
        } else {
            tsrc = rank; trb = (idx - E3) & 31u; tvalid = !remote && idx < NLIST;
        }
    }
    const unsigned tdesc = tsrc * 32u + trb;
    const unsigned long long validmask = __ballot(tvalid);
    const unsigned long long ownmask = __ballot(tvalid && tsrc == rank);
    const gu32* flagp = (tvalid && tsrc != rank) ? (const gu32*)(myws + OFF_XF + (tdesc * 8u + (unsigned)wave) * 128u) : tmo;
    const unsigned vown = (unsigned)(c * KD + kq * 16), vrem = (unsigned)(lane * 16);

    unsigned pend = (unsigned)validmask;
    auto refill = [&](unsigned ready, u32x4 (&a)[8]) {
        const unsigned todo = ready & pend;
        const int t = todo ? __builtin_ctz(todo) : -1;
        pend &= ~(todo & (0u - todo));
        const unsigned desc = __builtin_amdgcn_readlane(tdesc, t & 63);
        const bool own = (desc >> 5) == rank;
        const rsrc_t r = mk(own ? (const char*)P.x : myws + pbase, todo ? (own ? SHARD : OFF_W) : 0u);
        const unsigned so = own ? (desc & 31u) * 65536u + (unsigned)wave * 512u : desc * 65536u + (unsigned)wave * 8192u;
        const unsigned step = own ? 64u : 1024u;
        const unsigned vo = own ? vown : vrem;
#pragma unroll
        for (int s = 0; s < 8; ++s) a[s] = __builtin_amdgcn_raw_buffer_load_b128(r, vo, so + s * step, SYS);
        if (t >= 0) CSTAMP(24 + 3 * t);
        return t;
    };
    auto arrive = [&](int t, bool take) {
        asm volatile("s_waitcnt lgkmcnt(0)" ::: "memory");
        int old = 0;
        if (lane == 0) old = __hip_atomic_fetch_add(tcnt + t, 1, __ATOMIC_RELAXED, __HIP_MEMORY_SCOPE_WORKGROUP);
        old = rfl(old);
        if (old == NWAVES - 1 && lane == 0) __hip_atomic_fetch_or(tcnt + (take ? 18 : 16), 1 << t, __ATOMIC_RELAXED, __HIP_MEMORY_SCOPE_WORKGROUP);
        asm volatile("" ::: "memory");
        return old;
    };
    auto store_tile = [&](int t) {
        unsigned ticket = 0;
        if (remote) asm volatile("s_mov_b64 exec, 1\n\tglobal_atomic_add %0, %1, %2, off sc0\n\ts_mov_b64 exec, -1"
                                 : "+v"(ticket) : "v"(myws + OFF_QT + dj * 128u), "v"(1u) : "memory");
        const unsigned desc = __builtin_amdgcn_readlane(tdesc, t);
        const LAS int* tp = (const LAS int*)smem + t * 16 * TSTR + kq * TSTR + c * 4;
        unsigned v[16];
#pragma unroll
        for (int i = 0; i < 16; ++i) v[i] = (unsigned)tp[(i >> 2) * 4 * TSTR + (i & 3)];
        const bool bad = __hip_atomic_load(tcnt + 17, __ATOMIC_RELAXED, __HIP_MEMORY_SCOPE_WORKGROUP) != 0;
        const float sc = bad ? __builtin_nanf("") : scale;
        unsigned span = 0;
#pragma unroll
        for (int i = 0; i < 16; ++i) span |= v[i] + 0x200000u;
        const bool pack = remote && __all((span >> 22) == 0u && !bad);
        const unsigned id = cblk * 256u + desc;
        if (pack) {
            unsigned h[4];
            u32x4 lo[2];
#pragma unroll
            for (int g = 0; g < 4; ++g) {
                h[g] = __builtin_amdgcn_perm(v[4 * g + 1], v[4 * g], 0x0c0c0602) | __builtin_amdgcn_perm(v[4 * g + 3], v[4 * g + 2], 0x06020c0c);
                lo[g >> 1][(2 * g) & 3] = __builtin_amdgcn_perm(v[4 * g + 1], v[4 * g], 0x05040100);
                lo[g >> 1][(2 * g + 1) & 3] = __builtin_amdgcn_perm(v[4 * g + 3], v[4 * g + 2], 0x05040100);
            }
            __builtin_amdgcn_raw_buffer_store_b128(lo[0], orr, id * 4096u + lane * 16, 0, SYS);
            __builtin_amdgcn_raw_buffer_store_b128(lo[1], orr, id * 4096u + 1024 + lane * 16, 0, SYS);
#pragma unroll
            for (int g = 0; g < 3; ++g)
                __builtin_amdgcn_raw_buffer_store_b32((h[g] & 0x3f3f3f3fu) | ((h[3] << (6 - 2 * g)) & 0xc0c0c0c0u), orr,
                                                      id * 4096u + 2048 + g * 256 + lane * 4, 0, SYS);
        } else {
            const unsigned oo = remote ? id * 4096u + lane * 16 : (desc * 16u * NS + col0) * 4u + (unsigned)(kq * NS * 4 + c * 16);
            const unsigned st = remote ? 1024u : 4u * NS * 4u;
#pragma unroll
            for (int i = 0; i < 4; ++i) {
                f32x4 y;
#pragma unroll
                for (int e = 0; e < 4; ++e) y[e] = (float)(int)v[4 * i + e] * sc;
                __builtin_amdgcn_raw_buffer_store_b128(__builtin_bit_cast(u32x4, y), orr, oo + i * st, 0, SYS);
            }
        }
        if (remote) {
            asm volatile("s_waitcnt vmcnt(0)" : "+v"(ticket) :: "memory");
            const unsigned slot = ticket - (call - 1u) * NQ;
            if (lane == 0)
                __hip_atomic_store((gu32*)(myws + OFF_Q + dj * 1024u) + ((slot & 63u) * 4u + ((slot >> 6) & 3u)),
                                   ((call & 0x3fffffu) << 10) | (pack ? 512u : 0u) | id, __ATOMIC_RELAXED, __HIP_MEMORY_SCOPE_SYSTEM);
        }
        CSTAMP(8 + t);
    };
    u32x4 a0[8], a1[8];
    int t0i = refill((unsigned)ownmask, a0);
    int t1i = refill((unsigned)ownmask, a1);
    if (remote) {
        const unsigned u0 = (dj * 2u + cblk) * 64u + (unsigned)wave * 8u;
        const gu32* fp = (const gu32*)(myws + OFF_WF + u0 * 16u + ((lane >> 1) & 3) * 4u);
        unsigned have = 0;
#pragma unroll
        for (int s = 0; s < 8; ++s) {
            while (((have >> s) & 1u) == 0u && !dead) {
                have = (unsigned)__ballot(__hip_atomic_load(fp, __ATOMIC_RELAXED, __HIP_MEMORY_SCOPE_SYSTEM) >= call);
                if (((have >> s) & 1u) == 0u) dead = expired(spins, t0, tmo, call, lane);
            }
            asm volatile("" ::: "memory");
#pragma unroll
            for (int e = 0; e < 4; ++e)
                wf[s][e] = __builtin_amdgcn_raw_buffer_load_b128(sr, lane * 16 + e * 1024, OFF_W + (u0 + s) * 4096u, SYS);
        }
#pragma unroll
        for (int s = 0; s < 8; s += 2)
            asm volatile("" : "+v"(wf[s][0]), "+v"(wf[s][1]), "+v"(wf[s][2]), "+v"(wf[s][3]),
                              "+v"(wf[s + 1][0]), "+v"(wf[s + 1][1]), "+v"(wf[s + 1][2]), "+v"(wf[s + 1][3]));
    }
    CSTAMP(1);

    unsigned fv = __hip_atomic_load(flagp, __ATOMIC_RELAXED, __HIP_MEMORY_SCOPE_SYSTEM);
    asm volatile("" : "+v"(fv));
    auto claim = [&]() {
        const unsigned fin = rfl(__hip_atomic_load(tcnt + 16, __ATOMIC_RELAXED, __HIP_MEMORY_SCOPE_WORKGROUP));
        const unsigned clm = rfl(__hip_atomic_load(tcnt + 18, __ATOMIC_RELAXED, __HIP_MEMORY_SCOPE_WORKGROUP));
        if ((fin & ~clm) == 0u) return clm == (unsigned)validmask ? -2 : -1;
        const int ts = __builtin_ctz(fin & ~clm);
        int old = 0;
        if (lane == 0) old = __hip_atomic_fetch_or(tcnt + 18, 1 << ts, __ATOMIC_RELAXED, __HIP_MEMORY_SCOPE_WORKGROUP);
        asm volatile("" ::: "memory");
        return ((rfl(old) >> ts) & 1u) != 0u ? -1 : ts;
    };
    auto mma = [&](const u32x4 (&a)[8], i32x4 (&acc)[4]) {
#pragma unroll
        for (int e = 0; e < 4; ++e) acc[e] = (i32x4){0, 0, 0, 0};
#pragma unroll
        for (int s = 0; s < 8; ++s)
#pragma unroll
            for (int e = 0; e < 4; ++e)
                acc[e] = __builtin_amdgcn_mfma_i32_16x16x64_i8(__builtin_bit_cast(i32x4, wf[s][e]), __builtin_bit_cast(i32x4, a[s]), acc[e], 0, 0, 0);
    };
    auto add_tile = [&](int t, const i32x4 (&acc)[4], bool take) {
        LAS int* tp = (LAS int*)smem + t * 16 * TSTR + c * TSTR + kq * 16;
#pragma unroll
        for (int r = 0; r < 4; ++r)
#pragma unroll
            for (int e = 0; e < 4; ++e)
                __hip_atomic_fetch_add(tp + r * 4 + e, acc[e][r], __ATOMIC_RELAXED, __HIP_MEMORY_SCOPE_WORKGROUP);
        const int pos = arrive(t, take);
        CSTAMP(25 + 3 * t);
        return pos;
    };
    auto ready = [&]() {
        if (__builtin_amdgcn_readlane(fv, 63) == call) dead = true;
        const unsigned r = (unsigned)__ballot(fv >= call) | (unsigned)ownmask;
        asm volatile("" ::: "memory");
        return dead ? 0u : r;
    };
    auto half = [&](u32x4 (&a)[8], int& t, int other) {
        i32x4 acc[4];
        const int tc = t;
        if (tc >= 0) mma(a, acc);
        const unsigned rdy = ready();
        fv = __hip_atomic_load(flagp, __ATOMIC_RELAXED, __HIP_MEMORY_SCOPE_SYSTEM);
        t = refill(rdy, a);
        __builtin_amdgcn_sched_barrier(0);
        if (tc >= 0) {
            const bool busy = t >= 0 || other >= 0;
            const int pos = add_tile(tc, acc, !busy);
            if (busy ? pos < AHEAD : pos == NWAVES - 1) {
                const int ts = busy ? claim() : tc;
                if (ts >= 0) store_tile(ts);
            }
        }
    };
    for (;;) {
        do {
            half(a0, t0i, t1i);
            half(a1, t1i, t0i);
        } while (t0i >= 0 || t1i >= 0);
        unsigned rdy;
        for (;;) {
            rdy = ready() & pend;
            if ((rdy & (rdy - 1u)) != 0u) break;
            int ts;
            if (rdy != 0u) {
                const int t = refill(rdy, a0);
                fv = __hip_atomic_load(flagp, __ATOMIC_RELAXED, __HIP_MEMORY_SCOPE_SYSTEM);
                __builtin_amdgcn_sched_barrier(0);
                i32x4 acc[4];
                mma(a0, acc);
                ts = add_tile(t, acc, true) == NWAVES - 1 ? t : -1;
            } else {
                if (dead && pend != 0u) {
                    if (lane == 0) __hip_atomic_store(tcnt + 17, 1, __ATOMIC_RELAXED, __HIP_MEMORY_SCOPE_WORKGROUP);
                    for (; pend != 0u; pend &= pend - 1u) arrive(__builtin_ctz(pend), false);
                }
                if (pend != 0u) {
                    if (expired(spins, t0, tmo, call, lane)) dead = true;
                    fv = __hip_atomic_load(flagp, __ATOMIC_RELAXED, __HIP_MEMORY_SCOPE_SYSTEM);
                } else {
                    __builtin_amdgcn_s_sleep(1);
                }
                ts = claim();
                if (ts == -2) {
                    CSTAMP(2);
                    LSTAMP(63);
                    asm volatile("s_endpgm");
                }
            }
            asm volatile("" : "+v"(fv));
            if (ts >= 0) store_tile(ts);
        }
        t0i = refill(rdy, a0);
        t1i = refill(rdy, a1);
    }
}

extern "C" void kernel_launch(int rank, int nranks, ncclComm_t comm,
                              hipStream_t stream,
                              void* const* d_in, const int* in_sizes, int n_in,
                              void* d_out, int out_size,
                              const int* in_dtypes, int out_dtype,
                              void* const* d_out_all,
                              void* const* d_ws_all, size_t ws_bytes) {
    if (nranks != NR || (unsigned)rank >= (unsigned)NR || n_in != 4 || in_sizes[0] != MS * KD || in_sizes[1] != KD * NS
        || out_size != NR * MS * NS || ws_bytes < WS_NEED) {
        fprintf(stderr, "kernel_launch: unexpected shapes\n");
        return;
    }
    Params P;
    P.x = (const signed char*)d_in[0];
    P.w = (const signed char*)d_in[1];
    P.sx = (const float*)d_in[2];
    P.sw = (const float*)d_in[3];
    for (int i = 0; i < NR; ++i) {
        const int p = i < 3 ? ((rank & 4) | ((rank + 1 + i) & 3)) : (i < 7 ? (((rank & 4) ^ 4) | (i - 3)) : rank);
        P.ws[i] = (char*)d_ws_all[p];
    }
    P.out = (char*)d_out;
    P.rank = rank;
    P.pad = 0;
    ag_gemm_kernel<<<GRID, NWAVES * 64, 0, stream>>>(P);
    HIP_CHECK(hipGetLastError());
}
```

```cpp
#include <hip/hip_runtime.h>
#include <rccl/rccl.h>
#include <cstdio>
#include <cstdint>

#define HIP_CHECK(expr) do { hipError_t _e = (expr); if (_e != hipSuccess) { \
    fprintf(stderr, "HIP error %s at %s:%d\n", hipGetErrorString(_e), __FILE__, __LINE__); \
    return; } } while (0)

#define GAS __attribute__((address_space(1)))
#define LAS __attribute__((address_space(3)))
typedef unsigned u32x4 __attribute__((ext_vector_type(4)));
typedef int i32x4 __attribute__((ext_vector_type(4)));
typedef float f32x4 __attribute__((ext_vector_type(4)));
typedef GAS unsigned gu32;
typedef GAS unsigned long long gu64;
typedef __amdgpu_buffer_rsrc_t rsrc_t;

#define NGB 9
#define PW 8
#define TAU32 225
#define TAUI32 224
#define T_FIRSTI 70
#define TAUP32 149
#define T_FIRST 80
#define ACK1 150
#define SHMAX 70
#define ELEAD 130
#define WIN 3
#define WINP 4
#define WMAX 400
#define LEAD 170
#define CREDIT 60
#define NDL 2
#define WATCH 258
#define HADJ 278
#define POLL_SLEEP 0
#define AHEAD 3
constexpr int NR = 8;
constexpr int MS = 512;
constexpr int KD = 4096;
constexpr int NS = 512;
constexpr int NAB = 32 - NGB;
constexpr int GRID = 256;
constexpr int NWAVES = 8;
constexpr int NPUSH = 56;
constexpr int NPW = 8 * PW;
constexpr int RS = 12;
constexpr int NIN = NAB + NDL;
constexpr int NLIST = 128 + NGB + 3 * NDL;
constexpr int MAXT = (NLIST + RS - 1) / RS;
constexpr int TSTR = 65;
constexpr unsigned SHARD = MS * KD;
constexpr unsigned OUTB = NR * MS * NS * 4;
constexpr unsigned PAR_BYTES = (NR + 1) * SHARD;
constexpr unsigned OFF_W = NR * SHARD;
constexpr unsigned OFF_Y = 2 * PAR_BYTES;
constexpr unsigned OFF_XF = OFF_Y + 8 * 256 * 4096;
constexpr unsigned OFF_WF = OFF_XF + NR * 32 * 8 * 128;
constexpr unsigned OFF_Q = OFF_WF + 8192;
constexpr unsigned OFF_QT = OFF_Q + 4096;
constexpr unsigned OFF_CNT = OFF_QT + 4096;
constexpr unsigned OFF_TMO = OFF_CNT + GRID * 128;
constexpr unsigned OFF_DBG = OFF_TMO + 4096;
constexpr unsigned WS_NEED = OFF_DBG + GRID * NWAVES * 64 * 8;
constexpr int NQ = 2 * (4 * NAB + 3 * (NGB - NDL));
constexpr int NSTAT = 128 + 16 * NGB;
static_assert(NPW == 64 && NQ <= 256 && MAXT <= 13 && NDL <= NGB, "flag / queue layout");
constexpr unsigned long long DEADLINE_TICKS = 50000000ull;
constexpr int SYS = 17;

struct Params {
    const signed char* x;
    const signed char* w;
    const float* sx;
    const float* sw;
    char* ws[NR];
    char* out;
    int rank;
    int pad;
};

__device__ __forceinline__ unsigned long long rt() { return __builtin_amdgcn_s_memrealtime(); }
__device__ __forceinline__ rsrc_t mk(const void* p, unsigned bytes) {
    return __builtin_amdgcn_make_buffer_rsrc((void*)p, 0, (int)bytes, 0x00020000);
}
__device__ __forceinline__ unsigned rfl(unsigned v) { return __builtin_amdgcn_readfirstlane(v); }

#define STAMP(i) do { } while (0)
#define CSTAMP(i) do { } while (0)
#define SMEM_EXTRA 0
#define LSTAMP(i) do { } while (0)

__device__ __forceinline__ void transpose16x4(const unsigned (&r)[16], u32x4 (&o)[4]) {
#pragma unroll
    for (int q = 0; q < 4; ++q) {
        const unsigned r0 = r[4 * q], r1 = r[4 * q + 1], r2 = r[4 * q + 2], r3 = r[4 * q + 3];
        const unsigned lo01 = __builtin_amdgcn_perm(r1, r0, 0x05010400);
        const unsigned hi01 = __builtin_amdgcn_perm(r1, r0, 0x07030602);
        const unsigned lo23 = __builtin_amdgcn_perm(r3, r2, 0x05010400);
        const unsigned hi23 = __builtin_amdgcn_perm(r3, r2, 0x07030602);
        o[0][q] = __builtin_amdgcn_perm(lo23, lo01, 0x05040100);
        o[1][q] = __builtin_amdgcn_perm(lo23, lo01, 0x07060302);
        o[2][q] = __builtin_amdgcn_perm(hi23, hi01, 0x05040100);
        o[3][q] = __builtin_amdgcn_perm(hi23, hi01, 0x07060302);
    }
}

__device__ __forceinline__ bool expired(unsigned& spins, unsigned long long t0, gu32* tmo, unsigned call, int lane) {
    if (POLL_SLEEP) __builtin_amdgcn_s_sleep(POLL_SLEEP);
    spins = rfl(spins + 1u);
    if ((spins & 63u) != 0u) return false;
    const bool late = rfl(__hip_atomic_load(tmo, __ATOMIC_RELAXED, __HIP_MEMORY_SCOPE_AGENT)) == call
                      || rfl(rt() - t0 > DEADLINE_TICKS ? 1u : 0u) != 0u;
    if (late) __hip_atomic_store(tmo, call, __ATOMIC_RELAXED, __HIP_MEMORY_SCOPE_AGENT);
    return late;
}

__global__ void __launch_bounds__(NWAVES * 64, 2) ag_gemm_kernel(Params P) {
    __shared__ __attribute__((aligned(16))) int smem[MAXT * 16 * TSTR + 32 + SMEM_EXTRA];
    const int b = blockIdx.x;
    const int lane = threadIdx.x & 63;
    const int wave = rfl(threadIdx.x >> 6);
    const unsigned rank = (unsigned)P.rank & 7u;
    const unsigned myidx = rank & 3u, mycls = rank & 4u, otcls = mycls ^ 4u;
    char* myws = P.ws[7];
    gu32* cntp = (gu32*)(myws + OFF_CNT + b * 128 + wave * 4);
    gu32* tmo = (gu32*)(myws + OFF_TMO);
    LSTAMP(0);
    const unsigned cntv = __hip_atomic_load(cntp, __ATOMIC_RELAXED, __HIP_MEMORY_SCOPE_AGENT);
    const rsrc_t xr = mk(P.x, SHARD);
    const rsrc_t wr = mk(P.w, SHARD);
    const int c = lane & 15, kq = lane >> 4;

    if (b < NPUSH) {
        const unsigned tstart = (unsigned)rt();
        const unsigned di = (unsigned)b % 7u, cu = (unsigned)b / 7u;
        const bool cross = di >= 3u;
        const unsigned dj = (di - 3u) & 3u;
        char* pw = P.ws[di];
        const rsrc_t dr = mk(pw, OFF_Y);
        const unsigned xv = (unsigned)(c * KD + kq * 16);
        const unsigned quart = ((myidx + dj) & 3u) * 128u;
        const unsigned wv = (unsigned)(kq * 16 * NS + c * 4) + quart;
        const int npairs = cross ? NSTAT / 2 : 8 * NIN;
        auto until = [&](unsigned t) {
            unsigned now = (unsigned)rt();
            while ((int)(t - now) > 0) {
                if ((int)(t - now) > 8) __builtin_amdgcn_s_sleep(1);
                now = (unsigned)rt();
            }
            return now;
        };
        const int q = wave * 8 + (int)cu;
        unsigned origin = tstart + (cross ? T_FIRST : T_FIRSTI);
        const int tau = cross ? TAU32 : TAUI32;
        LAS unsigned* op = (LAS unsigned*)smem;
        if (wave == NWAVES - 1) op[lane] = 0;
        if (wave >= 2) {
            asm volatile("s_waitcnt lgkmcnt(0)" ::: "memory");
            __builtin_amdgcn_s_barrier();
        }
        unsigned sh = 0;
        auto gate = [&](unsigned t, int need, bool shifted) {
            unsigned now;
            for (;;) {
                const int fin = (int)rfl(__hip_atomic_load(op + 1, __ATOMIC_RELAXED, __HIP_MEMORY_SCOPE_WORKGROUP));
                if (shifted) sh = rfl(__hip_atomic_load(op + 3, __ATOMIC_RELAXED, __HIP_MEMORY_SCOPE_WORKGROUP));
                now = (unsigned)rt();
                const int left = (int)(t + (shifted ? sh : 0u) - now);
                if (left <= 0 && (fin >= need || left < -WMAX)) break;
                if (left > 12) __builtin_amdgcn_s_sleep(1);
            }
            return now;
        };
        auto finished = [&](int n) {
            if (lane == 0) __hip_atomic_fetch_max((LAS int*)op + 1, n, __ATOMIC_RELAXED, __HIP_MEMORY_SCOPE_WORKGROUP);
        };
        unsigned call = 0;
        for (int g = q; g < npairs; g += NPW) {
            const int sl = 2 * g - (int)cu;
            if (g >= 16) until(origin + sl * tau / 32 - LEAD);
            const bool isw = cross && g < 64;
            const unsigned m = (unsigned)(cross ? g - 64 : (g < 8 * NAB ? g + 8 * NGB : g - 8 * (NAB - NGB + NDL)));
            u32x4 e[2][4];
            if (isw) {
                const unsigned so = (unsigned)(g & 31) * 128u * NS + (unsigned)(g >> 5) * 64u;
#pragma unroll
                for (int u = 0; u < 2; ++u)
#pragma unroll
                    for (int i = 0; i < 16; ++i) e[u][i >> 2][i & 3] = __builtin_amdgcn_raw_buffer_load_b32(wr, wv + i * NS, so + u * 64 * NS, 0);
            } else {
                const unsigned so = (m >> 3) * 16u * KD + (m & 7u) * 512u;
#pragma unroll
                for (int u = 0; u < 2; ++u)
#pragma unroll
                    for (int j = 0; j < 4; ++j) e[u][j] = __builtin_amdgcn_raw_buffer_load_b128(xr, xv + u * 256 + j * 64, so, 0);
            }
            if (g < 16) __builtin_amdgcn_s_barrier();
            if (isw) {
#pragma unroll
                for (int u = 0; u < 2; ++u) {
                    unsigned r[16];
#pragma unroll
                    for (int i = 0; i < 16; ++i) r[i] = e[u][i >> 2][i & 3];
                    transpose16x4(r, e[u]);
                }
            }
            call = rfl(cntv) + 1;
            const unsigned gw = myidx * 64u + (unsigned)g;
            const unsigned so = (call & 1u) * PAR_BYTES + (isw ? OFF_W + gw * 8192u : rank * SHARD + m * 8192u) + lane * 16;
            const unsigned fo = isw ? OFF_WF + (gw >> 2) * 128u + (gw & 3u) * 4u : OFF_XF + (rank * 256u + m) * 128u;
#pragma unroll
            for (int u = 0; u < 2; ++u) asm volatile("" : "+v"(e[u][0]), "+v"(e[u][1]), "+v"(e[u][2]), "+v"(e[u][3]));
            if (g == q) STAMP(1);
            const int j0 = (g >> 3) * 2;
#pragma unroll
            for (int u = 0; u < 2; ++u) {
                const unsigned t = origin + (sl + 8 * u) * tau / 32;
                const unsigned now = gate(t, j0 + u + 1 - WIN, true);
#pragma unroll
                for (int j = 0; j < 4; ++j) __builtin_amdgcn_raw_buffer_store_b128(e[u][j], dr, so + u * 4096 + j * 1024, 0, SYS);
                if (u == 1 && lane == 0) op[2] = now - t;
            }
            asm volatile("s_waitcnt vmcnt(4)" ::: "memory");
            finished(j0 + 1);
            if (g < 8) {
                const int lag = (int)((unsigned)rt() - origin) - ACK1;
                if (lag > 0 && lane == 0) op[3] = lag < SHMAX ? lag : SHMAX;
            }
            asm volatile("s_waitcnt vmcnt(0)" ::: "memory");
            if (lane == 0) __hip_atomic_store((gu32*)(pw + fo), call, __ATOMIC_RELAXED, __HIP_MEMORY_SCOPE_SYSTEM);
            finished(j0 + 2);
        }
        call = rfl(cntv) + 1;
        if (lane == 0) __hip_atomic_store(cntp, call, __ATOMIC_RELAXED, __HIP_MEMORY_SCOPE_AGENT);
        if (!cross) return;

        const unsigned ctag = call & 0x3fffffu;
        const float scale = P.sx[0] * P.sw[0];
        const rsrc_t yr = mk(pw + OFF_Y + myidx * (512u * 4096u), 512u * 4096u);
        const rsrc_t orr = mk(P.out, OUTB);
        const unsigned obase = quart * 4u + (unsigned)(kq * NS * 4 + c * 16);
        const int p = (q + NPW - NSTAT / 2 % NPW) % NPW;
        const gu32* hp = (const gu32*)(myws + OFF_XF + ((otcls | dj) * 256u + 8 * (NGB - 4)) * 128u);
        u32x4 d[4];
        const gu32* fp = (const gu32*)(pw + OFF_Q + (myidx * 256u + ((unsigned)p & 7u) * 32u + (lane & 31)) * 4u);
        const unsigned long long t0 = rt();
        unsigned spins = 0;
        bool dead = false;
        origin += NSTAT * TAU32 / 32;
        if (p < 8) {
            until(origin - WATCH);
            unsigned tp = until(origin - WATCH + rfl(op[2]));
            unsigned v[8], ti[8];
#pragma unroll
            for (int i = 0; i < 8; ++i) { v[i] = 0; ti[i] = 0; }
            for (bool found = false; !found && !dead;) {
#pragma unroll
                for (int i = 0; i < 8; ++i) {
                    if (!found && rfl(v[i]) >= call) { found = true; origin = ti[i] + HADJ; }
                    unsigned now;
                    do now = (unsigned)rt(); while ((int)(tp + 11u - now) > 0 && !found);
                    tp = ti[i] = now;
                    v[i] = __hip_atomic_load(hp, __ATOMIC_RELAXED, __HIP_MEMORY_SCOPE_SYSTEM);
                    __builtin_amdgcn_sched_barrier(0);
                }
                if (!found && rt() - t0 > DEADLINE_TICKS) { dead = true; origin = tp; }
            }
            if (lane == 0) __hip_atomic_store(op, origin | 1u, __ATOMIC_RELAXED, __HIP_MEMORY_SCOPE_WORKGROUP);
        } else {
            while ((origin = rfl(__hip_atomic_load(op, __ATOMIC_RELAXED, __HIP_MEMORY_SCOPE_WORKGROUP))) == 0u)
                __builtin_amdgcn_s_sleep(2);
        }
        for (int k = 0; p + k * NPW < NQ; ++k) {
            const int ei = (p >> 3) + k * 8;
            unsigned ent = rfl(__hip_atomic_load(op + 32 + ei, __ATOMIC_ACQUIRE, __HIP_MEMORY_SCOPE_WORKGROUP));
            if ((ent >> 10) != ctag) {
                until(origin + (p + k * NPW) * TAUP32 / 32 - ELEAD);
                ent = rfl(__hip_atomic_load(op + 32 + ei, __ATOMIC_ACQUIRE, __HIP_MEMORY_SCOPE_WORKGROUP));
            }
            while ((ent >> 10) != ctag && !dead) {
                const unsigned fv = __hip_atomic_load(fp, __ATOMIC_RELAXED, __HIP_MEMORY_SCOPE_SYSTEM);
                if ((fv >> 10) == ctag && lane < 32) __hip_atomic_store(op + 32 + lane, fv, __ATOMIC_RELEASE, __HIP_MEMORY_SCOPE_WORKGROUP);
                ent = __builtin_amdgcn_readlane(fv, ei);
                if ((ent >> 10) != ctag) dead = expired(spins, t0, tmo, call, lane);
            }
            if (dead) break;
            asm volatile("" ::: "memory");
            const unsigned slotT = origin + (p + k * NPW) * TAUP32 / 32;
            const int jp = NSTAT / 8 + (p >> 3) + k * 8;
            const unsigned now = gate(slotT, jp + 1 - WINP, false);
            if ((int)(now - slotT) > CREDIT) origin += now - slotT - CREDIT;
            const unsigned so = (ent & 511u) * 4096u;
            const unsigned oo = (ent & 255u) * (16u * NS * 4u) + (ent & 256u) + obase;
            if (ent & 512u) {
                const u32x4 lo0 = __builtin_amdgcn_raw_buffer_load_b128(yr, lane * 16, so, SYS);
                const u32x4 lo1 = __builtin_amdgcn_raw_buffer_load_b128(yr, lane * 16 + 1024, so, SYS);
                unsigned hi[3];
#pragma unroll
                for (int g = 0; g < 3; ++g) hi[g] = __builtin_amdgcn_raw_buffer_load_b32(yr, lane * 4 + 2048 + g * 256, so, SYS);
                const unsigned h3 = ((hi[0] >> 6) & 0x03030303u) | ((hi[1] >> 4) & 0x0c0c0c0cu) | ((hi[2] >> 2) & 0x30303030u);
#pragma unroll
                for (int g = 0; g < 4; ++g) {
                    const unsigned h = g < 3 ? hi[g < 3 ? g : 0] : h3;
                    const unsigned l0 = (g < 2 ? lo0 : lo1)[(2 * g) & 3], l1 = (g < 2 ? lo0 : lo1)[(2 * g + 1) & 3];
                    f32x4 y;
                    y[0] = (float)(int)__builtin_amdgcn_sbfe(__builtin_amdgcn_perm(h, l0, 0x0c040100), 0, 22) * scale;
                    y[1] = (float)(int)__builtin_amdgcn_sbfe(__builtin_amdgcn_perm(h, l0, 0x0c050302), 0, 22) * scale;
                    y[2] = (float)(int)__builtin_amdgcn_sbfe(__builtin_amdgcn_perm(h, l1, 0x0c060100), 0, 22) * scale;
                    y[3] = (float)(int)__builtin_amdgcn_sbfe(__builtin_amdgcn_perm(h, l1, 0x0c070302), 0, 22) * scale;
                    d[g] = __builtin_bit_cast(u32x4, y);
                }
            } else {
#pragma unroll
                for (int j = 0; j < 4; ++j) d[j] = __builtin_amdgcn_raw_buffer_load_b128(yr, lane * 16 + j * 1024, so, SYS);
                origin += (TAU32 - TAUP32) * NPW / 32;
            }
            asm volatile("" : "+v"(d[0]), "+v"(d[1]), "+v"(d[2]), "+v"(d[3]));
            finished(jp + 1);
#pragma unroll
            for (int g = 0; g < 4; ++g) __builtin_amdgcn_raw_buffer_store_b128(d[g], orr, oo + g * 4 * NS * 4, 0, SYS);
        }
        if (dead && lane == 0)
            __hip_atomic_store((gu32*)(P.out + (size_t)rank * MS * NS * 4), 0x7fc00000u, __ATOMIC_RELAXED, __HIP_MEMORY_SCOPE_SYSTEM);
        LSTAMP(63);
        return;
    }
    if (b < 64) return;

    const int xcd = b & 7, slot = (b - 64) >> 3;
    const int rs = 3 * (xcd >> 1) + (slot >> 3);
    const bool remote = (slot & 4) != 0;
    const unsigned dj = (unsigned)slot & 3u;
    const unsigned cblk = (unsigned)xcd & 1u;
    const unsigned owner = remote ? (otcls | dj) : rank;
    const unsigned col0 = (cblk * 4u + dj) * 64u;
    LAS int* tcnt = (LAS int*)smem + MAXT * 16 * TSTR;

    u32x4 wf[8][4];
    if (!remote) {
        const unsigned wv = (unsigned)(kq * 16 * NS + c * 4);
#pragma unroll
        for (int s = 0; s < 8; ++s) {
            unsigned r[16];
            const unsigned so = (unsigned)(wave * 8 + s) * 64u * NS + col0;
#pragma unroll
            for (int i = 0; i < 16; ++i) r[i] = __builtin_amdgcn_raw_buffer_load_b32(wr, wv + i * NS, so, 0);
            transpose16x4(r, wf[s]);
        }
    }
    for (int i = threadIdx.x; i < MAXT * 16 * TSTR + 32 + SMEM_EXTRA; i += NWAVES * 64) smem[i] = 0;
    __syncthreads();
    const unsigned call = rfl(cntv) + 1;
    if (lane == 0) __hip_atomic_store(cntp, call, __ATOMIC_RELAXED, __HIP_MEMORY_SCOPE_AGENT);
    const float scale = P.sx[0] * P.sw[0];
    const unsigned pbase = (call & 1u) * PAR_BYTES;
    const rsrc_t sr = mk(myws + pbase, PAR_BYTES);
    const rsrc_t orr = remote ? mk(myws + OFF_Y + dj * (512u * 4096u), 512u * 4096u) : mk(P.out, OUTB);
    const unsigned long long t0 = rt();
    unsigned spins = 0;
    bool dead = false;

    unsigned tsrc, trb;
    bool tvalid;
    {
        const unsigned idx = (unsigned)rs + RS * (unsigned)lane;
        constexpr unsigned E1 = 3 * NIN, E2 = E1 + NAB, E3 = E2 + 4 * NGB;
        if (idx < E1) {
            const unsigned u = idx / 3u;
            tsrc = mycls | ((myidx + 1u + idx % 3u) & 3u); trb = u < NAB ? NGB + u : u - (NAB - NGB + NDL); tvalid = !remote || u < NAB;
        } else if (idx < E2) {
            tsrc = rank; trb = NGB + idx - E1; tvalid = true;
        } else if (idx < E3) {
            const unsigned u = idx - E2;
            tsrc = otcls | (u & 3u); trb = u >> 2; tvalid = !remote || (tsrc != owner && trb < NGB - NDL);
        } else {
            tsrc = rank; trb = (idx - E3) & 31u; tvalid = !remote && idx < NLIST;
        }
    }
    const unsigned tdesc = tsrc * 32u + trb;
    const unsigned long long validmask = __ballot(tvalid);
    const unsigned long long ownmask = __ballot(tvalid && tsrc == rank);
    const gu32* flagp = (tvalid && tsrc != rank) ? (const gu32*)(myws + OFF_XF + (tdesc * 8u + (unsigned)wave) * 128u) : tmo;
    const unsigned vown = (unsigned)(c * KD + kq * 16), vrem = (unsigned)(lane * 16);

    unsigned pend = (unsigned)validmask;
    auto refill = [&](unsigned ready, u32x4 (&a)[8]) {
        const unsigned todo = ready & pend;
        const int t = todo ? __builtin_ctz(todo) : -1;
        pend &= ~(todo & (0u - todo));
        const unsigned desc = __builtin_amdgcn_readlane(tdesc, t & 63);
        const bool own = (desc >> 5) == rank;
        const rsrc_t r = mk(own ? (const char*)P.x : myws + pbase, todo ? (own ? SHARD : OFF_W) : 0u);
        const unsigned so = own ? (desc & 31u) * 65536u + (unsigned)wave * 512u : desc * 65536u + (unsigned)wave * 8192u;
        const unsigned step = own ? 64u : 1024u;
        const unsigned vo = own ? vown : vrem;
#pragma unroll
        for (int s = 0; s < 8; ++s) a[s] = __builtin_amdgcn_raw_buffer_load_b128(r, vo, so + s * step, SYS);
        if (t >= 0) CSTAMP(24 + 3 * t);
        return t;
    };
    auto arrive = [&](int t, bool take) {
        asm volatile("s_waitcnt lgkmcnt(0)" ::: "memory");
        int old = 0;
        if (lane == 0) old = __hip_atomic_fetch_add(tcnt + t, 1, __ATOMIC_RELAXED, __HIP_MEMORY_SCOPE_WORKGROUP);
        old = rfl(old);
        if (old == NWAVES - 1 && lane == 0) __hip_atomic_fetch_or(tcnt + (take ? 18 : 16), 1 << t, __ATOMIC_RELAXED, __HIP_MEMORY_SCOPE_WORKGROUP);
        asm volatile("" ::: "memory");
        return old;
    };
    auto store_tile = [&](int t) {
        unsigned ticket = 0;
        if (remote) asm volatile("s_mov_b64 exec, 1\n\tglobal_atomic_add %0, %1, %2, off sc0\n\ts_mov_b64 exec, -1"
                                 : "+v"(ticket) : "v"(myws + OFF_QT + dj * 128u), "v"(1u) : "memory");
        const unsigned desc = __builtin_amdgcn_readlane(tdesc, t);
        const LAS int* tp = (const LAS int*)smem + t * 16 * TSTR + kq * TSTR + c * 4;
        unsigned v[16];
#pragma unroll
        for (int i = 0; i < 16; ++i) v[i] = (unsigned)tp[(i >> 2) * 4 * TSTR + (i & 3)];
        const bool bad = __hip_atomic_load(tcnt + 17, __ATOMIC_RELAXED, __HIP_MEMORY_SCOPE_WORKGROUP) != 0;
        const float sc = bad ? __builtin_nanf("") : scale;
        unsigned span = 0;
#pragma unroll
        for (int i = 0; i < 16; ++i) span |= v[i] + 0x200000u;
        const bool pack = remote && __all((span >> 22) == 0u && !bad);
        const unsigned id = cblk * 256u + desc;
        if (pack) {
            unsigned h[4];
            u32x4 lo[2];
#pragma unroll
            for (int g = 0; g < 4; ++g) {
                h[g] = __builtin_amdgcn_perm(v[4 * g + 1], v[4 * g], 0x0c0c0602) | __builtin_amdgcn_perm(v[4 * g + 3], v[4 * g + 2], 0x06020c0c);
                lo[g >> 1][(2 * g) & 3] = __builtin_amdgcn_perm(v[4 * g + 1], v[4 * g], 0x05040100);
                lo[g >> 1][(2 * g + 1) & 3] = __builtin_amdgcn_perm(v[4 * g + 3], v[4 * g + 2], 0x05040100);
            }
            __builtin_amdgcn_raw_buffer_store_b128(lo[0], orr, id * 4096u + lane * 16, 0, SYS);
            __builtin_amdgcn_raw_buffer_store_b128(lo[1], orr, id * 4096u + 1024 + lane * 16, 0, SYS);
#pragma unroll
            for (int g = 0; g < 3; ++g)
                __builtin_amdgcn_raw_buffer_store_b32((h[g] & 0x3f3f3f3fu) | ((h[3] << (6 - 2 * g)) & 0xc0c0c0c0u), orr,
                                                      id * 4096u + 2048 + g * 256 + lane * 4, 0, SYS);
        } else {
            const unsigned oo = remote ? id * 4096u + lane * 16 : (desc * 16u * NS + col0) * 4u + (unsigned)(kq * NS * 4 + c * 16);
            const unsigned st = remote ? 1024u : 4u * NS * 4u;
#pragma unroll
            for (int i = 0; i < 4; ++i) {
                f32x4 y;
#pragma unroll
                for (int e = 0; e < 4; ++e) y[e] = (float)(int)v[4 * i + e] * sc;
                __builtin_amdgcn_raw_buffer_store_b128(__builtin_bit_cast(u32x4, y), orr, oo + i * st, 0, SYS);
            }
        }
        if (remote) {
            asm volatile("s_waitcnt vmcnt(0)" : "+v"(ticket) :: "memory");
            const unsigned slot = ticket - (call - 1u) * NQ;
            if (lane == 0)
                __hip_atomic_store((gu32*)(myws + OFF_Q + dj * 1024u) + ((slot & 7u) * 32u + ((slot >> 3) & 31u)),
                                   ((call & 0x3fffffu) << 10) | (pack ? 512u : 0u) | id, __ATOMIC_RELAXED, __HIP_MEMORY_SCOPE_SYSTEM);
        }
        CSTAMP(8 + t);
    };
    u32x4 a0[8], a1[8];
    int t0i = refill((unsigned)ownmask, a0);
    int t1i = refill((unsigned)ownmask, a1);
    if (remote) {
        const unsigned u0 = (dj * 2u + cblk) * 64u + (unsigned)wave * 8u;
        const gu32* fp = (const gu32*)(myws + OFF_WF + u0 * 16u + ((lane >> 1) & 3) * 4u);
        unsigned have = 0;
#pragma unroll
        for (int s = 0; s < 8; ++s) {
            while (((have >> s) & 1u) == 0u && !dead) {
                have = (unsigned)__ballot(__hip_atomic_load(fp, __ATOMIC_RELAXED, __HIP_MEMORY_SCOPE_SYSTEM) >= call);
                if (((have >> s) & 1u) == 0u) dead = expired(spins, t0, tmo, call, lane);
            }
            asm volatile("" ::: "memory");
#pragma unroll
            for (int e = 0; e < 4; ++e)
                wf[s][e] = __builtin_amdgcn_raw_buffer_load_b128(sr, lane * 16 + e * 1024, OFF_W + (u0 + s) * 4096u, SYS);
        }
#pragma unroll
        for (int s = 0; s < 8; s += 2)
            asm volatile("" : "+v"(wf[s][0]), "+v"(wf[s][1]), "+v"(wf[s][2]), "+v"(wf[s][3]),
                              "+v"(wf[s + 1][0]), "+v"(wf[s + 1][1]), "+v"(wf[s + 1][2]), "+v"(wf[s + 1][3]));
    }
    CSTAMP(1);

    unsigned fv = __hip_atomic_load(flagp, __ATOMIC_RELAXED, __HIP_MEMORY_SCOPE_SYSTEM);
    asm volatile("" : "+v"(fv));
    auto claim = [&]() {
        const unsigned fin = rfl(__hip_atomic_load(tcnt + 16, __ATOMIC_RELAXED, __HIP_MEMORY_SCOPE_WORKGROUP));
        const unsigned clm = rfl(__hip_atomic_load(tcnt + 18, __ATOMIC_RELAXED, __HIP_MEMORY_SCOPE_WORKGROUP));
        if ((fin & ~clm) == 0u) return clm == (unsigned)validmask ? -2 : -1;
        const int ts = __builtin_ctz(fin & ~clm);
        int old = 0;
        if (lane == 0) old = __hip_atomic_fetch_or(tcnt + 18, 1 << ts, __ATOMIC_RELAXED, __HIP_MEMORY_SCOPE_WORKGROUP);
        asm volatile("" ::: "memory");
        return ((rfl(old) >> ts) & 1u) != 0u ? -1 : ts;
    };
    auto mma = [&](const u32x4 (&a)[8], i32x4 (&acc)[4]) {
#pragma unroll
        for (int e = 0; e < 4; ++e) acc[e] = (i32x4){0, 0, 0, 0};
#pragma unroll
        for (int s = 0; s < 8; ++s)
#pragma unroll
            for (int e = 0; e < 4; ++e)
                acc[e] = __builtin_amdgcn_mfma_i32_16x16x64_i8(__builtin_bit_cast(i32x4, wf[s][e]), __builtin_bit_cast(i32x4, a[s]), acc[e], 0, 0, 0);
    };
    auto add_tile = [&](int t, const i32x4 (&acc)[4], bool take) {
        LAS int* tp = (LAS int*)smem + t * 16 * TSTR + c * TSTR + kq * 16;
#pragma unroll
        for (int r = 0; r < 4; ++r)
#pragma unroll
            for (int e = 0; e < 4; ++e)
                __hip_atomic_fetch_add(tp + r * 4 + e, acc[e][r], __ATOMIC_RELAXED, __HIP_MEMORY_SCOPE_WORKGROUP);
        const int pos = arrive(t, take);
        CSTAMP(25 + 3 * t);
        return pos;
    };
    auto ready = [&]() {
        if (__builtin_amdgcn_readlane(fv, 63) == call) dead = true;
        const unsigned r = (unsigned)__ballot(fv >= call) | (unsigned)ownmask;
        asm volatile("" ::: "memory");
        return dead ? 0u : r;
    };
    auto half = [&](u32x4 (&a)[8], int& t, int other) {
        i32x4 acc[4];
        const int tc = t;
        if (tc >= 0) mma(a, acc);
        const unsigned rdy = ready();
        fv = __hip_atomic_load(flagp, __ATOMIC_RELAXED, __HIP_MEMORY_SCOPE_SYSTEM);
        t = refill(rdy, a);
        __builtin_amdgcn_sched_barrier(0);
        if (tc >= 0) {
            const bool busy = t >= 0 || other >= 0;
            const int pos = add_tile(tc, acc, !busy);
            if (busy ? pos < AHEAD : pos == NWAVES - 1) {
                const int ts = busy ? claim() : tc;
                if (ts >= 0) store_tile(ts);
            }
        }
    };
    for (;;) {
        do {
            half(a0, t0i, t1i);
            half(a1, t1i, t0i);
        } while (t0i >= 0 || t1i >= 0);
        unsigned rdy;
        for (;;) {
            rdy = ready() & pend;
            if ((rdy & (rdy - 1u)) != 0u) break;
            int ts;
            if (rdy != 0u) {
                const int t = refill(rdy, a0);
                fv = __hip_atomic_load(flagp, __ATOMIC_RELAXED, __HIP_MEMORY_SCOPE_SYSTEM);
                __builtin_amdgcn_sched_barrier(0);
                i32x4 acc[4];
                mma(a0, acc);
                ts = add_tile(t, acc, true) == NWAVES - 1 ? t : -1;
            } else {
                if (dead && pend != 0u) {
                    if (lane == 0) __hip_atomic_store(tcnt + 17, 1, __ATOMIC_RELAXED, __HIP_MEMORY_SCOPE_WORKGROUP);
                    for (; pend != 0u; pend &= pend - 1u) arrive(__builtin_ctz(pend), false);
                }
                if (pend != 0u) {
                    if (expired(spins, t0, tmo, call, lane)) dead = true;
                    fv = __hip_atomic_load(flagp, __ATOMIC_RELAXED, __HIP_MEMORY_SCOPE_SYSTEM);
                } else {
                    __builtin_amdgcn_s_sleep(1);
                }
                ts = claim();
                if (ts == -2) {
                    CSTAMP(2);
                    LSTAMP(63);
                    asm volatile("s_endpgm");
                }
            }
            asm volatile("" : "+v"(fv));
            if (ts >= 0) store_tile(ts);
        }
        t0i = refill(rdy, a0);
        t1i = refill(rdy, a1);
    }
}

extern "C" void kernel_launch(int rank, int nranks, ncclComm_t comm,
                              hipStream_t stream,
                              void* const* d_in, const int* in_sizes, int n_in,
                              void* d_out, int out_size,
                              const int* in_dtypes, int out_dtype,
                              void* const* d_out_all,
                              void* const* d_ws_all, size_t ws_bytes) {
    if (nranks != NR || (unsigned)rank >= (unsigned)NR || n_in != 4 || in_sizes[0] != MS * KD || in_sizes[1] != KD * NS
        || out_size != NR * MS * NS || ws_bytes < WS_NEED) {
        fprintf(stderr, "kernel_launch: unexpected shapes\n");
        return;
    }
    Params P;
    P.x = (const signed char*)d_in[0];
    P.w = (const signed char*)d_in[1];
    P.sx = (const float*)d_in[2];
    P.sw = (const float*)d_in[3];
    for (int i = 0; i < NR; ++i) {
        const int p = i < 3 ? ((rank & 4) | ((rank + 1 + i) & 3)) : (i < 7 ? (((rank & 4) ^ 4) | (i - 3)) : rank);
        P.ws[i] = (char*)d_ws_all[p];
    }
    P.out = (char*)d_out;
    P.rank = rank;
    P.pad = 0;
    ag_gemm_kernel<<<GRID, NWAVES * 64, 0, stream>>>(P);
    HIP_CHECK(hipGetLastError());
}
```

```cpp
#include <hip/hip_runtime.h>
#include <rccl/rccl.h>
#include <cstdio>
#include <cstdint>

#define HIP_CHECK(expr) do { hipError_t _e = (expr); if (_e != hipSuccess) { \
    fprintf(stderr, "HIP error %s at %s:%d\n", hipGetErrorString(_e), __FILE__, __LINE__); \
    return; } } while (0)

#define GAS __attribute__((address_space(1)))
#define LAS __attribute__((address_space(3)))
typedef unsigned u32x4 __attribute__((ext_vector_type(4)));
typedef int i32x4 __attribute__((ext_vector_type(4)));
typedef float f32x4 __attribute__((ext_vector_type(4)));
typedef GAS unsigned gu32;
typedef __amdgpu_buffer_rsrc_t rsrc_t;

constexpr int NGB = 9;
constexpr int TAU32 = 225;
constexpr int TAUI32 = 224;
constexpr int T_FIRSTI = 70;
constexpr int TAUP32 = 149;
constexpr int T_FIRST = 80;
constexpr int ACK1 = 150;
constexpr int SHMAX = 70;
constexpr int ELEAD = 130;
constexpr int WIN = 3;
constexpr int WINP = 4;
constexpr int WMAX = 400;
constexpr int LEAD = 170;
constexpr int CREDIT = 60;
constexpr int NDL = 2;
constexpr int WATCH = 258;
constexpr int HADJ = 278;
constexpr int AHEAD = 3;
constexpr int NR = 8;
constexpr int MS = 512;
constexpr int KD = 4096;
constexpr int NS = 512;
constexpr int NAB = 32 - NGB;
constexpr int GRID = 256;
constexpr int NWAVES = 8;
constexpr int NPUSH = 56;
constexpr int NPW = 8 * NWAVES;
constexpr int RS = 12;
constexpr int NIN = NAB + NDL;
constexpr int NLIST = 128 + NGB + 3 * NDL;
constexpr int MAXT = (NLIST + RS - 1) / RS;
constexpr int TSTR = 65;
constexpr unsigned SHARD = MS * KD;
constexpr unsigned OUTB = NR * MS * NS * 4;
constexpr unsigned PAR_BYTES = (NR + 1) * SHARD;
constexpr unsigned OFF_W = NR * SHARD;
constexpr unsigned OFF_Y = 2 * PAR_BYTES;
constexpr unsigned OFF_XF = OFF_Y + 8 * 256 * 4096;
constexpr unsigned OFF_WF = OFF_XF + NR * 32 * 8 * 128;
constexpr unsigned OFF_Q = OFF_WF + 8192;
constexpr unsigned OFF_QT = OFF_Q + 4096;
constexpr unsigned OFF_CNT = OFF_QT + 4096;
constexpr unsigned OFF_TMO = OFF_CNT + GRID * 128;
constexpr unsigned WS_NEED = OFF_TMO + 4096;
constexpr int NQ = 2 * (4 * NAB + 3 * (NGB - NDL));
constexpr int NSTAT = 128 + 16 * NGB;
static_assert(NPW == 64 && NQ <= 256 && MAXT <= 13 && NDL <= NGB, "flag / queue layout");
constexpr unsigned long long DEADLINE_TICKS = 50000000ull;
constexpr int SYS = 17;

struct Params {
    const signed char* x;
    const signed char* w;
    const float* sx;
    const float* sw;
    char* ws[NR];
    char* out;
    int rank;
    int pad;
};

__device__ __forceinline__ unsigned long long rt() { return __builtin_amdgcn_s_memrealtime(); }
__device__ __forceinline__ rsrc_t mk(const void* p, unsigned bytes) {
    return __builtin_amdgcn_make_buffer_rsrc((void*)p, 0, (int)bytes, 0x00020000);
}
__device__ __forceinline__ unsigned rfl(unsigned v) { return __builtin_amdgcn_readfirstlane(v); }


__device__ __forceinline__ void transpose16x4(const unsigned (&r)[16], u32x4 (&o)[4]) {
#pragma unroll
    for (int q = 0; q < 4; ++q) {
        const unsigned r0 = r[4 * q], r1 = r[4 * q + 1], r2 = r[4 * q + 2], r3 = r[4 * q + 3];
        const unsigned lo01 = __builtin_amdgcn_perm(r1, r0, 0x05010400);
        const unsigned hi01 = __builtin_amdgcn_perm(r1, r0, 0x07030602);
        const unsigned lo23 = __builtin_amdgcn_perm(r3, r2, 0x05010400);
        const unsigned hi23 = __builtin_amdgcn_perm(r3, r2, 0x07030602);
        o[0][q] = __builtin_amdgcn_perm(lo23, lo01, 0x05040100);
        o[1][q] = __builtin_amdgcn_perm(lo23, lo01, 0x07060302);
        o[2][q] = __builtin_amdgcn_perm(hi23, hi01, 0x05040100);
        o[3][q] = __builtin_amdgcn_perm(hi23, hi01, 0x07060302);
    }
}

__device__ __forceinline__ unsigned call_tag(unsigned call) { return call % 0x3fffffu + 1u; }

__device__ __forceinline__ bool expired(unsigned& spins, unsigned long long t0, gu32* tmo, unsigned call) {
    spins = rfl(spins + 1u);
    if ((spins & 63u) != 0u) return false;
    const bool late = rfl(__hip_atomic_load(tmo, __ATOMIC_RELAXED, __HIP_MEMORY_SCOPE_AGENT)) == call
                      || rfl(rt() - t0 > DEADLINE_TICKS ? 1u : 0u) != 0u;
    if (late) __hip_atomic_store(tmo, call, __ATOMIC_RELAXED, __HIP_MEMORY_SCOPE_AGENT);
    return late;
}

__global__ void __launch_bounds__(NWAVES * 64, 2) ag_gemm_kernel(Params P) {
    __shared__ __attribute__((aligned(16))) int smem[MAXT * 16 * TSTR + 32];
    const int b = blockIdx.x;
    const int lane = threadIdx.x & 63;
    const int wave = rfl(threadIdx.x >> 6);
    const unsigned rank = (unsigned)P.rank & 7u;
    const unsigned myidx = rank & 3u, mycls = rank & 4u, otcls = mycls ^ 4u;
    char* myws = P.ws[7];
    gu32* cntp = (gu32*)(myws + OFF_CNT + b * 128 + wave * 4);
    gu32* tmo = (gu32*)(myws + OFF_TMO);
    const unsigned cntv = __hip_atomic_load(cntp, __ATOMIC_RELAXED, __HIP_MEMORY_SCOPE_AGENT);
    const rsrc_t xr = mk(P.x, SHARD);
    const rsrc_t wr = mk(P.w, SHARD);
    const int c = lane & 15, kq = lane >> 4;

    if (b < NPUSH) {
        const unsigned tstart = (unsigned)rt();
        const unsigned di = (unsigned)b % 7u, cu = (unsigned)b / 7u;
        const bool cross = di >= 3u;
        const unsigned dj = (di - 3u) & 3u;
        char* pw = P.ws[di];
        const rsrc_t dr = mk(pw, OFF_Y);
        const unsigned xv = (unsigned)(c * KD + kq * 16);
        const unsigned quart = ((myidx + dj) & 3u) * 128u;
        const unsigned wv = (unsigned)(kq * 16 * NS + c * 4) + quart;
        const int npairs = cross ? NSTAT / 2 : 8 * NIN;
        auto until = [&](unsigned t) {
            unsigned now = (unsigned)rt();
            while ((int)(t - now) > 0) {
                if ((int)(t - now) > 8) __builtin_amdgcn_s_sleep(1);
                now = (unsigned)rt();
            }
            return now;
        };
        const int q = wave * 8 + (int)cu;
        unsigned origin = tstart + (cross ? T_FIRST : T_FIRSTI);
        const int tau = cross ? TAU32 : TAUI32;
        LAS unsigned* op = (LAS unsigned*)smem;
        if (wave == NWAVES - 1) op[lane] = 0;
        if (wave >= 2) {
            asm volatile("s_waitcnt lgkmcnt(0)" ::: "memory");
            __builtin_amdgcn_s_barrier();
        }
        unsigned sh = 0;
        auto gate = [&](unsigned t, int need, bool shifted) {
            unsigned now;
            for (;;) {
                const int fin = (int)rfl(__hip_atomic_load(op + 1, __ATOMIC_RELAXED, __HIP_MEMORY_SCOPE_WORKGROUP));
                if (shifted) sh = rfl(__hip_atomic_load(op + 3, __ATOMIC_RELAXED, __HIP_MEMORY_SCOPE_WORKGROUP));
                now = (unsigned)rt();
                const int left = (int)(t + (shifted ? sh : 0u) - now);
                if (left <= 0 && (fin >= need || left < -WMAX)) break;
                if (left > 12) __builtin_amdgcn_s_sleep(1);
            }
            return now;
        };
        auto finished = [&](int n) {
            if (lane == 0) __hip_atomic_fetch_max((LAS int*)op + 1, n, __ATOMIC_RELAXED, __HIP_MEMORY_SCOPE_WORKGROUP);
        };
        unsigned call = 0;
        for (int g = q; g < npairs; g += NPW) {
            const int sl = 2 * g - (int)cu;
            if (g >= 16) until(origin + sl * tau / 32 - LEAD);
            const bool isw = cross && g < 64;
            const unsigned m = (unsigned)(cross ? g - 64 : (g < 8 * NAB ? g + 8 * NGB : g - 8 * (NAB - NGB + NDL)));
            u32x4 e[2][4];
            if (isw) {
                const unsigned so = (unsigned)(g & 31) * 128u * NS + (unsigned)(g >> 5) * 64u;
#pragma unroll
                for (int u = 0; u < 2; ++u)
#pragma unroll
                    for (int i = 0; i < 16; ++i) e[u][i >> 2][i & 3] = __builtin_amdgcn_raw_buffer_load_b32(wr, wv + i * NS, so + u * 64 * NS, 0);
            } else {
                const unsigned so = (m >> 3) * 16u * KD + (m & 7u) * 512u;
#pragma unroll
                for (int u = 0; u < 2; ++u)
#pragma unroll
                    for (int j = 0; j < 4; ++j) e[u][j] = __builtin_amdgcn_raw_buffer_load_b128(xr, xv + u * 256 + j * 64, so, 0);
            }
            if (g < 16) __builtin_amdgcn_s_barrier();
            if (isw) {
#pragma unroll
                for (int u = 0; u < 2; ++u) {
                    unsigned r[16];
#pragma unroll
                    for (int i = 0; i < 16; ++i) r[i] = e[u][i >> 2][i & 3];
                    transpose16x4(r, e[u]);
                }
            }
            call = rfl(cntv) + 1;
            const unsigned gw = myidx * 64u + (unsigned)g;
            const unsigned so = (call & 1u) * PAR_BYTES + (isw ? OFF_W + gw * 8192u : rank * SHARD + m * 8192u) + lane * 16;
            const unsigned fo = isw ? OFF_WF + (gw >> 2) * 128u + (gw & 3u) * 4u : OFF_XF + (rank * 256u + m) * 128u;
#pragma unroll
            for (int u = 0; u < 2; ++u) asm volatile("" : "+v"(e[u][0]), "+v"(e[u][1]), "+v"(e[u][2]), "+v"(e[u][3]));
            const int j0 = (g >> 3) * 2;
#pragma unroll
            for (int u = 0; u < 2; ++u) {
                const unsigned t = origin + (sl + 8 * u) * tau / 32;
                const unsigned now = gate(t, j0 + u + 1 - WIN, true);
#pragma unroll
                for (int j = 0; j < 4; ++j) __builtin_amdgcn_raw_buffer_store_b128(e[u][j], dr, so + u * 4096 + j * 1024, 0, SYS);
                if (u == 1 && lane == 0) op[2] = now - t;
            }
            asm volatile("s_waitcnt vmcnt(4)" ::: "memory");
            finished(j0 + 1);
            if (g < 8) {
                const int lag = (int)((unsigned)rt() - origin) - ACK1;
                if (lag > 0 && lane == 0) op[3] = lag < SHMAX ? lag : SHMAX;
            }
            asm volatile("s_waitcnt vmcnt(0)" ::: "memory");
            if (lane == 0) __hip_atomic_store((gu32*)(pw + fo), call, __ATOMIC_RELAXED, __HIP_MEMORY_SCOPE_SYSTEM);
            finished(j0 + 2);
        }
        call = rfl(cntv) + 1;
        if (lane == 0) __hip_atomic_store(cntp, call, __ATOMIC_RELAXED, __HIP_MEMORY_SCOPE_AGENT);
        if (!cross) return;

        const unsigned ctag = call_tag(call);
        const float scale = P.sx[0] * P.sw[0];
        const rsrc_t yr = mk(pw + OFF_Y + myidx * (512u * 4096u), 512u * 4096u);
        const rsrc_t orr = mk(P.out, OUTB);
        const unsigned obase = quart * 4u + (unsigned)(kq * NS * 4 + c * 16);
        const int p = (q + NPW - NSTAT / 2 % NPW) % NPW;
        const gu32* hp = (const gu32*)(myws + OFF_XF + ((otcls | dj) * 256u + 8 * (NGB - 4)) * 128u);
        u32x4 d[4];
        const gu32* fp = (const gu32*)(pw + OFF_Q + (myidx * 256u + ((unsigned)p & 7u) * 32u + (lane & 31)) * 4u);
        const unsigned long long t0 = rt();
        unsigned spins = 0;
        bool dead = false;
        origin += NSTAT * TAU32 / 32;
        if (p < 8) {
            until(origin - WATCH);
            unsigned tp = until(origin - WATCH + rfl(op[2]));
            unsigned v[8], ti[8];
#pragma unroll
            for (int i = 0; i < 8; ++i) { v[i] = 0; ti[i] = 0; }
            for (bool found = false; !found && !dead;) {
#pragma unroll
                for (int i = 0; i < 8; ++i) {
                    if (!found && rfl(v[i]) >= call) { found = true; origin = ti[i] + HADJ; }
                    unsigned now;
                    do now = (unsigned)rt(); while ((int)(tp + 11u - now) > 0 && !found);
                    tp = ti[i] = now;
                    v[i] = __hip_atomic_load(hp, __ATOMIC_RELAXED, __HIP_MEMORY_SCOPE_SYSTEM);
                    __builtin_amdgcn_sched_barrier(0);
                }
                if (!found && expired(spins, t0, tmo, call)) { dead = true; origin = tp; }
            }
            if (lane == 0) __hip_atomic_store(op, origin | 1u, __ATOMIC_RELAXED, __HIP_MEMORY_SCOPE_WORKGROUP);
        } else {
            while ((origin = rfl(__hip_atomic_load(op, __ATOMIC_RELAXED, __HIP_MEMORY_SCOPE_WORKGROUP))) == 0u)
                __builtin_amdgcn_s_sleep(2);
        }
        for (int k = 0; p + k * NPW < NQ; ++k) {
            const int ei = (p >> 3) + k * 8;
            unsigned ent = rfl(__hip_atomic_load(op + 32 + ei, __ATOMIC_ACQUIRE, __HIP_MEMORY_SCOPE_WORKGROUP));
            if ((ent >> 10) != ctag) {
                until(origin + (p + k * NPW) * TAUP32 / 32 - ELEAD);
                ent = rfl(__hip_atomic_load(op + 32 + ei, __ATOMIC_ACQUIRE, __HIP_MEMORY_SCOPE_WORKGROUP));
            }
            while ((ent >> 10) != ctag && !dead) {
                const unsigned fv = __hip_atomic_load(fp, __ATOMIC_RELAXED, __HIP_MEMORY_SCOPE_SYSTEM);
                if ((fv >> 10) == ctag && lane < 32) __hip_atomic_store(op + 32 + lane, fv, __ATOMIC_RELEASE, __HIP_MEMORY_SCOPE_WORKGROUP);
                ent = __builtin_amdgcn_readlane(fv, ei);
                if ((ent >> 10) != ctag) dead = expired(spins, t0, tmo, call);
            }
            if (dead) break;
            asm volatile("" ::: "memory");
            const unsigned slotT = origin + (p + k * NPW) * TAUP32 / 32;
            const int jp = NSTAT / 8 + (p >> 3) + k * 8;
            const unsigned now = gate(slotT, jp + 1 - WINP, false);
            if ((int)(now - slotT) > CREDIT) origin += now - slotT - CREDIT;
            const unsigned so = (ent & 511u) * 4096u;
            const unsigned oo = (ent & 255u) * (16u * NS * 4u) + (ent & 256u) + obase;
            if (ent & 512u) {
                const u32x4 lo0 = __builtin_amdgcn_raw_buffer_load_b128(yr, lane * 16, so, SYS);
                const u32x4 lo1 = __builtin_amdgcn_raw_buffer_load_b128(yr, lane * 16 + 1024, so, SYS);
                unsigned hi[3];
#pragma unroll
                for (int g = 0; g < 3; ++g) hi[g] = __builtin_amdgcn_raw_buffer_load_b32(yr, lane * 4 + 2048 + g * 256, so, SYS);
                const unsigned h3 = ((hi[0] >> 6) & 0x03030303u) | ((hi[1] >> 4) & 0x0c0c0c0cu) | ((hi[2] >> 2) & 0x30303030u);
#pragma unroll
                for (int g = 0; g < 4; ++g) {
                    const unsigned h = g < 3 ? hi[g < 3 ? g : 0] : h3;
                    const unsigned l0 = (g < 2 ? lo0 : lo1)[(2 * g) & 3], l1 = (g < 2 ? lo0 : lo1)[(2 * g + 1) & 3];
                    f32x4 y;
                    y[0] = (float)(int)__builtin_amdgcn_sbfe(__builtin_amdgcn_perm(h, l0, 0x0c040100), 0, 22) * scale;
                    y[1] = (float)(int)__builtin_amdgcn_sbfe(__builtin_amdgcn_perm(h, l0, 0x0c050302), 0, 22) * scale;
                    y[2] = (float)(int)__builtin_amdgcn_sbfe(__builtin_amdgcn_perm(h, l1, 0x0c060100), 0, 22) * scale;
                    y[3] = (float)(int)__builtin_amdgcn_sbfe(__builtin_amdgcn_perm(h, l1, 0x0c070302), 0, 22) * scale;
                    d[g] = __builtin_bit_cast(u32x4, y);
                }
            } else {
#pragma unroll
                for (int j = 0; j < 4; ++j) d[j] = __builtin_amdgcn_raw_buffer_load_b128(yr, lane * 16 + j * 1024, so, SYS);
                origin += (TAU32 - TAUP32) * NPW / 32;
            }
            asm volatile("" : "+v"(d[0]), "+v"(d[1]), "+v"(d[2]), "+v"(d[3]));
            finished(jp + 1);
#pragma unroll
            for (int g = 0; g < 4; ++g) __builtin_amdgcn_raw_buffer_store_b128(d[g], orr, oo + g * 4 * NS * 4, 0, SYS);
        }
        if (dead && lane == 0)
            __hip_atomic_store((gu32*)(P.out + (size_t)rank * MS * NS * 4), 0x7fc00000u, __ATOMIC_RELAXED, __HIP_MEMORY_SCOPE_SYSTEM);
        return;
    }
    if (b < 64) return;

    const int xcd = b & 7, slot = (b - 64) >> 3;
    const int rs = 3 * (xcd >> 1) + (slot >> 3);
    const bool remote = (slot & 4) != 0;
    const unsigned dj = (unsigned)slot & 3u;
    const unsigned cblk = (unsigned)xcd & 1u;
    const unsigned owner = remote ? (otcls | dj) : rank;
    const unsigned col0 = (cblk * 4u + dj) * 64u;
    LAS int* tcnt = (LAS int*)smem + MAXT * 16 * TSTR;

    u32x4 wf[8][4];
    if (!remote) {
        const unsigned wv = (unsigned)(kq * 16 * NS + c * 4);
#pragma unroll
        for (int s = 0; s < 8; ++s) {
            unsigned r[16];
            const unsigned so = (unsigned)(wave * 8 + s) * 64u * NS + col0;
#pragma unroll
            for (int i = 0; i < 16; ++i) r[i] = __builtin_amdgcn_raw_buffer_load_b32(wr, wv + i * NS, so, 0);
            transpose16x4(r, wf[s]);
        }
    }
    for (int i = threadIdx.x; i < MAXT * 16 * TSTR + 32; i += NWAVES * 64) smem[i] = 0;
    __syncthreads();
    const unsigned call = rfl(cntv) + 1;
    if (lane == 0) __hip_atomic_store(cntp, call, __ATOMIC_RELAXED, __HIP_MEMORY_SCOPE_AGENT);
    const float scale = P.sx[0] * P.sw[0];
    const unsigned pbase = (call & 1u) * PAR_BYTES;
    const rsrc_t sr = mk(myws + pbase, PAR_BYTES);
    const rsrc_t orr = remote ? mk(myws + OFF_Y + dj * (512u * 4096u), 512u * 4096u) : mk(P.out, OUTB);
    const unsigned long long t0 = rt();
    unsigned spins = 0;
    bool dead = false;

    unsigned tsrc, trb;
    bool tvalid;
    {
        const unsigned idx = (unsigned)rs + RS * (unsigned)lane;
        constexpr unsigned E1 = 3 * NIN, E2 = E1 + NAB, E3 = E2 + 4 * NGB;
        if (idx < E1) {
            const unsigned u = idx / 3u;
            tsrc = mycls | ((myidx + 1u + idx % 3u) & 3u); trb = u < NAB ? NGB + u : u - (NAB - NGB + NDL); tvalid = !remote || u < NAB;
        } else if (idx < E2) {
            tsrc = rank; trb = NGB + idx - E1; tvalid = true;
        } else if (idx < E3) {
            const unsigned u = idx - E2;
            tsrc = otcls | (u & 3u); trb = u >> 2; tvalid = !remote || (tsrc != owner && trb < NGB - NDL);
        } else {
            tsrc = rank; trb = (idx - E3) & 31u; tvalid = !remote && idx < NLIST;
        }
    }
    const unsigned tdesc = tsrc * 32u + trb;
    const unsigned long long validmask = __ballot(tvalid);
    const unsigned long long ownmask = __ballot(tvalid && tsrc == rank);
    const gu32* flagp = (tvalid && tsrc != rank) ? (const gu32*)(myws + OFF_XF + (tdesc * 8u + (unsigned)wave) * 128u) : tmo;
    const unsigned vown = (unsigned)(c * KD + kq * 16), vrem = (unsigned)(lane * 16);

    unsigned pend = (unsigned)validmask;
    auto refill = [&](unsigned ready, u32x4 (&a)[8]) {
        const unsigned todo = ready & pend;
        const int t = todo ? __builtin_ctz(todo) : -1;
        pend &= ~(todo & (0u - todo));
        const unsigned desc = __builtin_amdgcn_readlane(tdesc, t & 63);
        const bool own = (desc >> 5) == rank;
        const rsrc_t r = mk(own ? (const char*)P.x : myws + pbase, todo ? (own ? SHARD : OFF_W) : 0u);
        const unsigned so = own ? (desc & 31u) * 65536u + (unsigned)wave * 512u : desc * 65536u + (unsigned)wave * 8192u;
        const unsigned step = own ? 64u : 1024u;
        const unsigned vo = own ? vown : vrem;
#pragma unroll
        for (int s = 0; s < 8; ++s) a[s] = __builtin_amdgcn_raw_buffer_load_b128(r, vo, so + s * step, SYS);
        return t;
    };
    auto arrive = [&](int t, bool take) {
        asm volatile("s_waitcnt lgkmcnt(0)" ::: "memory");
        int old = 0;
        if (lane == 0) old = __hip_atomic_fetch_add(tcnt + t, 1, __ATOMIC_RELAXED, __HIP_MEMORY_SCOPE_WORKGROUP);
        old = rfl(old);
        if (old == NWAVES - 1 && lane == 0) __hip_atomic_fetch_or(tcnt + (take ? 18 : 16), 1 << t, __ATOMIC_RELAXED, __HIP_MEMORY_SCOPE_WORKGROUP);
        asm volatile("" ::: "memory");
        return old;
    };
    auto store_tile = [&](int t) {
        unsigned ticket = 0;
        if (remote) {
            unsigned long long keep;
            asm volatile("s_mov_b64 %1, exec\n\ts_mov_b64 exec, 1\n\tglobal_atomic_add %0, %2, %3, off sc0\n\ts_mov_b64 exec, %1"
                         : "+v"(ticket), "=&s"(keep) : "v"(myws + OFF_QT + dj * 128u), "v"(1u) : "memory");
        }
        const unsigned desc = __builtin_amdgcn_readlane(tdesc, t);
        const LAS int* tp = (const LAS int*)smem + t * 16 * TSTR + kq * TSTR + c * 4;
        unsigned v[16];
#pragma unroll
        for (int i = 0; i < 16; ++i) v[i] = (unsigned)tp[(i >> 2) * 4 * TSTR + (i & 3)];
        const bool bad = __hip_atomic_load(tcnt + 17, __ATOMIC_RELAXED, __HIP_MEMORY_SCOPE_WORKGROUP) != 0;
        const float sc = bad ? __builtin_nanf("") : scale;
        unsigned span = 0;
#pragma unroll
        for (int i = 0; i < 16; ++i) span |= v[i] + 0x200000u;
        const bool pack = remote && __all((span >> 22) == 0u && !bad);
        const unsigned id = cblk * 256u + desc;
        if (pack) {
            unsigned h[4];
            u32x4 lo[2];
#pragma unroll
            for (int g = 0; g < 4; ++g) {
                h[g] = __builtin_amdgcn_perm(v[4 * g + 1], v[4 * g], 0x0c0c0602) | __builtin_amdgcn_perm(v[4 * g + 3], v[4 * g + 2], 0x06020c0c);
                lo[g >> 1][(2 * g) & 3] = __builtin_amdgcn_perm(v[4 * g + 1], v[4 * g], 0x05040100);
                lo[g >> 1][(2 * g + 1) & 3] = __builtin_amdgcn_perm(v[4 * g + 3], v[4 * g + 2], 0x05040100);
            }
            __builtin_amdgcn_raw_buffer_store_b128(lo[0], orr, id * 4096u + lane * 16, 0, SYS);
            __builtin_amdgcn_raw_buffer_store_b128(lo[1], orr, id * 4096u + 1024 + lane * 16, 0, SYS);
#pragma unroll
            for (int g = 0; g < 3; ++g)
                __builtin_amdgcn_raw_buffer_store_b32((h[g] & 0x3f3f3f3fu) | ((h[3] << (6 - 2 * g)) & 0xc0c0c0c0u), orr,
                                                      id * 4096u + 2048 + g * 256 + lane * 4, 0, SYS);
        } else {
            const unsigned oo = remote ? id * 4096u + lane * 16 : (desc * 16u * NS + col0) * 4u + (unsigned)(kq * NS * 4 + c * 16);
            const unsigned st = remote ? 1024u : 4u * NS * 4u;
#pragma unroll
            for (int i = 0; i < 4; ++i) {
                f32x4 y;
#pragma unroll
                for (int e = 0; e < 4; ++e) y[e] = (float)(int)v[4 * i + e] * sc;
                __builtin_amdgcn_raw_buffer_store_b128(__builtin_bit_cast(u32x4, y), orr, oo + i * st, 0, SYS);
            }
        }
        if (remote) {
            asm volatile("s_waitcnt vmcnt(0)" : "+v"(ticket) :: "memory");
            const unsigned slot = ticket - (call - 1u) * NQ;
            if (lane == 0)
                __hip_atomic_store((gu32*)(myws + OFF_Q + dj * 1024u) + ((slot & 7u) * 32u + ((slot >> 3) & 31u)),
                                   (call_tag(call) << 10) | (pack ? 512u : 0u) | id, __ATOMIC_RELAXED, __HIP_MEMORY_SCOPE_SYSTEM);
        }
    };
    u32x4 a0[8], a1[8];
    int t0i = refill((unsigned)ownmask, a0);
    int t1i = refill((unsigned)ownmask, a1);
    if (remote) {
        const unsigned u0 = (dj * 2u + cblk) * 64u + (unsigned)wave * 8u;
        const gu32* fp = (const gu32*)(myws + OFF_WF + u0 * 16u + ((lane >> 1) & 3) * 4u);
        unsigned have = 0;
#pragma unroll
        for (int s = 0; s < 8; ++s) {
            while (((have >> s) & 1u) == 0u && !dead) {
                have = (unsigned)__ballot(__hip_atomic_load(fp, __ATOMIC_RELAXED, __HIP_MEMORY_SCOPE_SYSTEM) >= call);
                if (((have >> s) & 1u) == 0u) dead = expired(spins, t0, tmo, call);
            }
            asm volatile("" ::: "memory");
#pragma unroll
            for (int e = 0; e < 4; ++e)
                wf[s][e] = __builtin_amdgcn_raw_buffer_load_b128(sr, lane * 16 + e * 1024, OFF_W + (u0 + s) * 4096u, SYS);
        }
        if (dead && lane == 0) __hip_atomic_store(tcnt + 17, 1, __ATOMIC_RELAXED, __HIP_MEMORY_SCOPE_WORKGROUP);
#pragma unroll
        for (int s = 0; s < 8; s += 2)
            asm volatile("" : "+v"(wf[s][0]), "+v"(wf[s][1]), "+v"(wf[s][2]), "+v"(wf[s][3]),
                              "+v"(wf[s + 1][0]), "+v"(wf[s + 1][1]), "+v"(wf[s + 1][2]), "+v"(wf[s + 1][3]));
    }

    unsigned fv = __hip_atomic_load(flagp, __ATOMIC_RELAXED, __HIP_MEMORY_SCOPE_SYSTEM);
    asm volatile("" : "+v"(fv));
    auto claim = [&]() {
        const unsigned fin = rfl(__hip_atomic_load(tcnt + 16, __ATOMIC_RELAXED, __HIP_MEMORY_SCOPE_WORKGROUP));
        const unsigned clm = rfl(__hip_atomic_load(tcnt + 18, __ATOMIC_RELAXED, __HIP_MEMORY_SCOPE_WORKGROUP));
        if ((fin & ~clm) == 0u) return clm == (unsigned)validmask ? -2 : -1;
        const int ts = __builtin_ctz(fin & ~clm);
        int old = 0;
        if (lane == 0) old = __hip_atomic_fetch_or(tcnt + 18, 1 << ts, __ATOMIC_RELAXED, __HIP_MEMORY_SCOPE_WORKGROUP);
        asm volatile("" ::: "memory");
        return ((rfl(old) >> ts) & 1u) != 0u ? -1 : ts;
    };
    auto mma = [&](const u32x4 (&a)[8], i32x4 (&acc)[4]) {
#pragma unroll
        for (int e = 0; e < 4; ++e) acc[e] = (i32x4){0, 0, 0, 0};
#pragma unroll
        for (int s = 0; s < 8; ++s)
#pragma unroll
            for (int e = 0; e < 4; ++e)
                acc[e] = __builtin_amdgcn_mfma_i32_16x16x64_i8(__builtin_bit_cast(i32x4, wf[s][e]), __builtin_bit_cast(i32x4, a[s]), acc[e], 0, 0, 0);
    };
    auto add_tile = [&](int t, const i32x4 (&acc)[4], bool take) {
        LAS int* tp = (LAS int*)smem + t * 16 * TSTR + c * TSTR + kq * 16;
#pragma unroll
        for (int r = 0; r < 4; ++r)
#pragma unroll
            for (int e = 0; e < 4; ++e)
                __hip_atomic_fetch_add(tp + r * 4 + e, acc[e][r], __ATOMIC_RELAXED, __HIP_MEMORY_SCOPE_WORKGROUP);
        const int pos = arrive(t, take);
        return pos;
    };
    auto ready = [&]() {
        if (__builtin_amdgcn_readlane(fv, 63) == call) dead = true;
        const unsigned r = (unsigned)__ballot(fv >= call) | (unsigned)ownmask;
        asm volatile("" ::: "memory");
        return dead ? 0u : r;
    };
    auto half = [&](u32x4 (&a)[8], int& t, int other) {
        i32x4 acc[4];
        const int tc = t;
        if (tc >= 0) mma(a, acc);
        const unsigned rdy = ready();
        fv = __hip_atomic_load(flagp, __ATOMIC_RELAXED, __HIP_MEMORY_SCOPE_SYSTEM);
        t = refill(rdy, a);
        __builtin_amdgcn_sched_barrier(0);
        if (tc >= 0) {
            const bool busy = t >= 0 || other >= 0;
            const int pos = add_tile(tc, acc, !busy);
            if (busy ? pos < AHEAD : pos == NWAVES - 1) {
                const int ts = busy ? claim() : tc;
                if (ts >= 0) store_tile(ts);
            }
        }
    };
    for (;;) {
        do {
            half(a0, t0i, t1i);
            half(a1, t1i, t0i);
        } while (t0i >= 0 || t1i >= 0);
        unsigned rdy;
        for (;;) {
            rdy = ready() & pend;
            if ((rdy & (rdy - 1u)) != 0u) break;
            int ts;
            if (rdy != 0u) {
                const int t = refill(rdy, a0);
                fv = __hip_atomic_load(flagp, __ATOMIC_RELAXED, __HIP_MEMORY_SCOPE_SYSTEM);
                __builtin_amdgcn_sched_barrier(0);
                i32x4 acc[4];
                mma(a0, acc);
                ts = add_tile(t, acc, true) == NWAVES - 1 ? t : -1;
            } else {
                if (dead && pend != 0u) {
                    if (lane == 0) __hip_atomic_store(tcnt + 17, 1, __ATOMIC_RELAXED, __HIP_MEMORY_SCOPE_WORKGROUP);
                    for (; pend != 0u; pend &= pend - 1u) arrive(__builtin_ctz(pend), false);
                }
                if (pend != 0u) {
                    if (expired(spins, t0, tmo, call)) dead = true;
                    fv = __hip_atomic_load(flagp, __ATOMIC_RELAXED, __HIP_MEMORY_SCOPE_SYSTEM);
                } else {
                    __builtin_amdgcn_s_sleep(1);
                }
                ts = claim();
                if (ts == -2) {
                    asm volatile("s_endpgm");
                }
            }
            asm volatile("" : "+v"(fv));
            ts = (int)rfl((unsigned)ts);
            if (ts >= 0) store_tile(ts);
        }
        t0i = refill(rdy, a0);
        t1i = refill(rdy, a1);
    }
}

extern "C" void kernel_launch(int rank, int nranks, ncclComm_t comm,
                              hipStream_t stream,
                              void* const* d_in, const int* in_sizes, int n_in,
                              void* d_out, int out_size,
                              const int* in_dtypes, int out_dtype,
                              void* const* d_out_all,
                              void* const* d_ws_all, size_t ws_bytes) {
    if (nranks != NR || (unsigned)rank >= (unsigned)NR || n_in != 4 || in_sizes[0] != MS * KD || in_sizes[1] != KD * NS
        || out_size != NR * MS * NS || ws_bytes < WS_NEED) {
        fprintf(stderr, "kernel_launch: unexpected shapes\n");
        return;
    }
    Params P;
    P.x = (const signed char*)d_in[0];
    P.w = (const signed char*)d_in[1];
    P.sx = (const float*)d_in[2];
    P.sw = (const float*)d_in[3];
    for (int i = 0; i < NR; ++i) {
        const int p = i < 3 ? ((rank & 4) | ((rank + 1 + i) & 3)) : (i < 7 ? (((rank & 4) ^ 4) | (i - 3)) : rank);
        P.ws[i] = (char*)d_ws_all[p];
    }
    P.out = (char*)d_out;
    P.rank = rank;
    P.pad = 0;
    ag_gemm_kernel<<<GRID, NWAVES * 64, 0, stream>>>(P);
    HIP_CHECK(hipGetLastError());
}
```

```cpp
#include <hip/hip_runtime.h>
#include <rccl/rccl.h>
#include <cstdio>
#include <cstdint>

#define HIP_CHECK(expr) do { hipError_t _e = (expr); if (_e != hipSuccess) { \
    fprintf(stderr, "HIP error %s at %s:%d\n", hipGetErrorString(_e), __FILE__, __LINE__); \
    return; } } while (0)

#define GAS __attribute__((address_space(1)))
#define LAS __attribute__((address_space(3)))
typedef unsigned u32x4 __attribute__((ext_vector_type(4)));
typedef int i32x4 __attribute__((ext_vector_type(4)));
typedef float f32x4 __attribute__((ext_vector_type(4)));
typedef GAS unsigned gu32;
typedef __amdgpu_buffer_rsrc_t rsrc_t;

constexpr int NGB = 9;
constexpr int NDL = 2;
constexpr int TAU32 = 225;
constexpr int TAUI32 = 224;
constexpr int TAUP32 = 149;
constexpr int T_FIRST = 80;
constexpr int T_FIRSTI = 70;
constexpr int ACK1 = 125;
constexpr int TACC = 280;
constexpr int LEAD = 170;
constexpr int WIN = 3;
constexpr int WINP = 4;
constexpr int WMAX = 400;
constexpr int WATCH = 258;
constexpr int HADJ = 270;
constexpr int ELEAD = 100;
constexpr int CREDIT = 60;
constexpr int AHEAD = 3;

constexpr int NR = 8;
constexpr int MS = 512;
constexpr int KD = 4096;
constexpr int NS = 512;
constexpr int NAB = 32 - NGB;
constexpr int GRID = 256;
constexpr int NWAVES = 8;
constexpr int NPUSH = 56;
constexpr int NPW = 8 * NWAVES;
constexpr int RS = 12;
constexpr int NIN = NAB + NDL;
constexpr int NLIST = 128 + NGB + 3 * NDL;
constexpr int MAXT = (NLIST + RS - 1) / RS;
constexpr int TSTR = 65;
constexpr unsigned SHARD = MS * KD;
constexpr unsigned OUTB = NR * MS * NS * 4;
constexpr unsigned PAR_BYTES = (NR + 1) * SHARD;
constexpr unsigned OFF_W = NR * SHARD;
constexpr unsigned OFF_Y = 2 * PAR_BYTES;
constexpr unsigned OFF_XF = OFF_Y + 8 * 256 * 4096;
constexpr unsigned OFF_WF = OFF_XF + NR * 32 * 8 * 128;
constexpr unsigned OFF_Q = OFF_WF + 8192;
constexpr unsigned OFF_QT = OFF_Q + 4096;
constexpr unsigned OFF_CNT = OFF_QT + 4096;
constexpr unsigned OFF_TMO = OFF_CNT + GRID * 128;
constexpr unsigned WS_NEED = OFF_TMO + 4096;
constexpr int NQ = 2 * (4 * NAB + 3 * (NGB - NDL));
constexpr int NSTAT = 128 + 16 * NGB;
static_assert(NPW == 64 && NQ <= 256 && MAXT <= 13 && NDL <= NGB, "flag / queue layout");
constexpr unsigned long long DEADLINE_TICKS = 50000000ull;
constexpr int SYS = 17;

struct Params {
    const signed char* x;
    const signed char* w;
    const float* sx;
    const float* sw;
    char* ws[NR];
    char* out;
    int rank;
    int pad;
};

__device__ __forceinline__ unsigned long long rt() { return __builtin_amdgcn_s_memrealtime(); }
__device__ __forceinline__ rsrc_t mk(const void* p, unsigned bytes) {
    return __builtin_amdgcn_make_buffer_rsrc((void*)p, 0, (int)bytes, 0x00020000);
}
__device__ __forceinline__ unsigned rfl(unsigned v) { return __builtin_amdgcn_readfirstlane(v); }

__device__ __forceinline__ void transpose16x4(const unsigned (&r)[16], u32x4 (&o)[4]) {
#pragma unroll
    for (int q = 0; q < 4; ++q) {
        const unsigned r0 = r[4 * q], r1 = r[4 * q + 1], r2 = r[4 * q + 2], r3 = r[4 * q + 3];
        const unsigned lo01 = __builtin_amdgcn_perm(r1, r0, 0x05010400);
        const unsigned hi01 = __builtin_amdgcn_perm(r1, r0, 0x07030602);
        const unsigned lo23 = __builtin_amdgcn_perm(r3, r2, 0x05010400);
        const unsigned hi23 = __builtin_amdgcn_perm(r3, r2, 0x07030602);
        o[0][q] = __builtin_amdgcn_perm(lo23, lo01, 0x05040100);
        o[1][q] = __builtin_amdgcn_perm(lo23, lo01, 0x07060302);
        o[2][q] = __builtin_amdgcn_perm(hi23, hi01, 0x05040100);
        o[3][q] = __builtin_amdgcn_perm(hi23, hi01, 0x07060302);
    }
}

__device__ __forceinline__ unsigned call_tag(unsigned call) { return call % 0x3fffffu + 1u; }

__device__ __forceinline__ bool expired(unsigned& spins, unsigned long long t0, gu32* tmo, unsigned call) {
    spins = rfl(spins + 1u);
    if ((spins & 63u) != 0u) return false;
    const bool late = rfl(__hip_atomic_load(tmo, __ATOMIC_RELAXED, __HIP_MEMORY_SCOPE_AGENT)) == call
                      || rfl(rt() - t0 > DEADLINE_TICKS ? 1u : 0u) != 0u;
    if (late) __hip_atomic_store(tmo, call, __ATOMIC_RELAXED, __HIP_MEMORY_SCOPE_AGENT);
    return late;
}

__global__ void __launch_bounds__(NWAVES * 64, 2) ag_gemm_kernel(Params P) {
    __shared__ __attribute__((aligned(16))) int smem[MAXT * 16 * TSTR + 32];
    const int b = blockIdx.x;
    const int lane = threadIdx.x & 63;
    const int wave = rfl(threadIdx.x >> 6);
    const unsigned rank = (unsigned)P.rank & 7u;
    const unsigned myidx = rank & 3u, mycls = rank & 4u, otcls = mycls ^ 4u;
    char* myws = P.ws[7];
    gu32* cntp = (gu32*)(myws + OFF_CNT + b * 128 + wave * 4);
    gu32* tmo = (gu32*)(myws + OFF_TMO);
    const unsigned cntv = __hip_atomic_load(cntp, __ATOMIC_RELAXED, __HIP_MEMORY_SCOPE_AGENT);
    const rsrc_t xr = mk(P.x, SHARD);
    const rsrc_t wr = mk(P.w, SHARD);
    const int c = lane & 15, kq = lane >> 4;

    if (b < NPUSH) {
        const unsigned tstart = (unsigned)rt();
        const unsigned di = (unsigned)b % 7u, cu = (unsigned)b / 7u;
        const bool cross = di >= 3u;
        const unsigned dj = (di - 3u) & 3u;
        char* pw = P.ws[di];
        const rsrc_t dr = mk(pw, OFF_Y);
        const unsigned xv = (unsigned)(c * KD + kq * 16);
        const unsigned quart = ((myidx + dj) & 3u) * 128u;
        const unsigned wv = (unsigned)(kq * 16 * NS + c * 4) + quart;
        const int npairs = cross ? NSTAT / 2 : 8 * NIN;
        auto until = [&](unsigned t) {
            unsigned now = (unsigned)rt();
            while ((int)(t - now) > 0) {
                if ((int)(t - now) > 8) __builtin_amdgcn_s_sleep(1);
                now = (unsigned)rt();
            }
            return now;
        };
        const int q = wave * 8 + (int)cu;
        unsigned origin = tstart + (cross ? T_FIRST : T_FIRSTI);
        const int tau = cross ? TAU32 : TAUI32;
        LAS unsigned* op = (LAS unsigned*)smem;
        if (wave == NWAVES - 1) op[lane] = 0;
        if (wave >= 2) {
            asm volatile("s_waitcnt lgkmcnt(0)" ::: "memory");
            __builtin_amdgcn_s_barrier();
        }
        unsigned sh = 0;
        auto gate = [&](unsigned t, int need, bool shifted) {
            unsigned now;
            for (;;) {
                const int fin = (int)rfl(__hip_atomic_load(op + 1, __ATOMIC_RELAXED, __HIP_MEMORY_SCOPE_WORKGROUP));
                if (shifted) {
                    const unsigned common = rfl(__hip_atomic_load(op + 4, __ATOMIC_RELAXED, __HIP_MEMORY_SCOPE_WORKGROUP));
                    sh = common ? common - tstart : 0u;
                }
                now = (unsigned)rt();
                const int left = (int)(t + (shifted ? sh : 0u) - now);
                if (left <= 0 && (fin >= need || left < -WMAX)) break;
                if (left > 12) __builtin_amdgcn_s_sleep(1);
            }
            return now;
        };
        auto finished = [&](int n) {
            if (lane == 0) __hip_atomic_fetch_max((LAS int*)op + 1, n, __ATOMIC_RELAXED, __HIP_MEMORY_SCOPE_WORKGROUP);
        };
        char* acc = myws + OFF_QT + 1024 + di * 128u;
        if (wave == NWAVES - 1) {
            const unsigned par = (rfl(cntv) + 1u) & 1u;
            unsigned got = 0;
            unsigned long long keep;
            if (cu == 0u && lane == 0) asm volatile("global_atomic_swap %0, %1, off" :: "v"(acc + (par ^ 1u) * 4u), "v"(0u) : "memory");
            for (unsigned t = TACC; t < TACC + 300u && (got >> 24) != 8u; t += 60u) {
                until(tstart + t);
                asm volatile("s_mov_b64 %1, exec\n\ts_mov_b64 exec, 1\n\tglobal_atomic_add %0, %2, %3, off sc0\n\ts_mov_b64 exec, %1\n\ts_waitcnt vmcnt(0)"
                             : "+v"(got), "=&s"(keep) : "v"(acc + par * 4u), "v"(0u) : "memory");
                got = rfl(got);
            }
            const int d = (int)((((got & 0xffffffu) >> 3) - ACK1 - origin) << 12) >> 12;
            if ((got >> 24) == 8u && d > -100 && d < 300 && lane == 0) op[4] = (tstart + d) | 1u;
        }
        unsigned call = 0;
        for (int g = q; g < npairs; g += NPW) {
            const int sl = 2 * g - (int)cu;
            if (g >= 16) until(origin + sl * tau / 32 - LEAD);
            const bool isw = cross && g < 64;
            const unsigned m = (unsigned)(cross ? g - 64 : (g < 8 * NAB ? g + 8 * NGB : g - 8 * (NAB - NGB + NDL)));
            u32x4 e[2][4];
            if (isw) {
                const unsigned so = (unsigned)(g & 31) * 128u * NS + (unsigned)(g >> 5) * 64u;
#pragma unroll
                for (int u = 0; u < 2; ++u)
#pragma unroll
                    for (int i = 0; i < 16; ++i) e[u][i >> 2][i & 3] = __builtin_amdgcn_raw_buffer_load_b32(wr, wv + i * NS, so + u * 64 * NS, 0);
            } else {
                const unsigned so = (m >> 3) * 16u * KD + (m & 7u) * 512u;
#pragma unroll
                for (int u = 0; u < 2; ++u)
#pragma unroll
                    for (int j = 0; j < 4; ++j) e[u][j] = __builtin_amdgcn_raw_buffer_load_b128(xr, xv + u * 256 + j * 64, so, 0);
            }
            if (g < 16) __builtin_amdgcn_s_barrier();
            if (isw) {
#pragma unroll
                for (int u = 0; u < 2; ++u) {
                    unsigned r[16];
#pragma unroll
                    for (int i = 0; i < 16; ++i) r[i] = e[u][i >> 2][i & 3];
                    transpose16x4(r, e[u]);
                }
            }
            call = rfl(cntv) + 1;
            const unsigned gw = myidx * 64u + (unsigned)g;
            const unsigned so = (call & 1u) * PAR_BYTES + (isw ? OFF_W + gw * 8192u : rank * SHARD + m * 8192u) + lane * 16;
            const unsigned fo = isw ? OFF_WF + (gw >> 2) * 128u + (gw & 3u) * 4u : OFF_XF + (rank * 256u + m) * 128u;
#pragma unroll
            for (int u = 0; u < 2; ++u) asm volatile("" : "+v"(e[u][0]), "+v"(e[u][1]), "+v"(e[u][2]), "+v"(e[u][3]));
            const int j0 = (g >> 3) * 2;
#pragma unroll
            for (int u = 0; u < 2; ++u) {
                const unsigned t = origin + (sl + 8 * u) * tau / 32;
                const unsigned now = gate(t, j0 + u + 1 - WIN, true);
#pragma unroll
                for (int j = 0; j < 4; ++j) __builtin_amdgcn_raw_buffer_store_b128(e[u][j], dr, so + u * 4096 + j * 1024, 0, SYS);
                if (u == 1 && lane == 0) op[2] = now - t + tstart;
            }
            asm volatile("s_waitcnt vmcnt(4)" ::: "memory");
            finished(j0 + 1);
            if (g < 8 && lane == 0)
                asm volatile("global_atomic_add %0, %1, off" :: "v"(acc + (call & 1u) * 4u), "v"(((unsigned)rt() & 0xfffffu) | 0x1000000u) : "memory");
            asm volatile("s_waitcnt vmcnt(0)" ::: "memory");
            if (lane == 0) __hip_atomic_store((gu32*)(pw + fo), call, __ATOMIC_RELAXED, __HIP_MEMORY_SCOPE_SYSTEM);
            finished(j0 + 2);
        }
        call = rfl(cntv) + 1;
        if (lane == 0) __hip_atomic_store(cntp, call, __ATOMIC_RELAXED, __HIP_MEMORY_SCOPE_AGENT);
        if (!cross) return;

        const unsigned ctag = call_tag(call);
        const float scale = P.sx[0] * P.sw[0];
        const rsrc_t yr = mk(pw + OFF_Y + myidx * (512u * 4096u), 512u * 4096u);
        const rsrc_t orr = mk(P.out, OUTB);
        const unsigned obase = quart * 4u + (unsigned)(kq * NS * 4 + c * 16);
        const int p = (q + NPW - NSTAT / 2 % NPW) % NPW;
        const gu32* hp = (const gu32*)(myws + OFF_XF + ((otcls | dj) * 256u + 8 * (NGB - 4)) * 128u);
        u32x4 d[4];
        const gu32* fp = (const gu32*)(pw + OFF_Q + (myidx * 256u + ((unsigned)p & 7u) * 32u + (lane & 31)) * 4u);
        const unsigned long long t0 = rt();
        unsigned spins = 0;
        bool dead = false;
        origin += NSTAT * TAU32 / 32;
        if (p < 8) {
            until(origin - WATCH - 100u);
            unsigned tp = until(origin - WATCH + rfl(op[2]) - tstart);
            unsigned v[8], ti[8];
#pragma unroll
            for (int i = 0; i < 8; ++i) { v[i] = 0; ti[i] = 0; }
            for (bool found = false; !found && !dead;) {
#pragma unroll
                for (int i = 0; i < 8; ++i) {
                    if (!found && rfl(v[i]) >= call) { found = true; origin = ti[i] + HADJ; }
                    unsigned now;
                    do now = (unsigned)rt(); while ((int)(tp + 11u - now) > 0 && !found);
                    tp = ti[i] = now;
                    v[i] = __hip_atomic_load(hp, __ATOMIC_RELAXED, __HIP_MEMORY_SCOPE_SYSTEM);
                    __builtin_amdgcn_sched_barrier(0);
                }
                if (!found && expired(spins, t0, tmo, call)) { dead = true; origin = tp; }
            }
            if (lane == 0) __hip_atomic_store(op, origin | 1u, __ATOMIC_RELAXED, __HIP_MEMORY_SCOPE_WORKGROUP);
        } else {
            while ((origin = rfl(__hip_atomic_load(op, __ATOMIC_RELAXED, __HIP_MEMORY_SCOPE_WORKGROUP))) == 0u)
                __builtin_amdgcn_s_sleep(2);
        }
        for (int k = 0; p + k * NPW < NQ; ++k) {
            const int ei = (p >> 3) + k * 8;
            unsigned ent = rfl(__hip_atomic_load(op + 32 + ei, __ATOMIC_ACQUIRE, __HIP_MEMORY_SCOPE_WORKGROUP));
            if ((ent >> 10) != ctag) {
                until(origin + (p + k * NPW) * TAUP32 / 32 - ELEAD);
                ent = rfl(__hip_atomic_load(op + 32 + ei, __ATOMIC_ACQUIRE, __HIP_MEMORY_SCOPE_WORKGROUP));
            }
            while ((ent >> 10) != ctag && !dead) {
                const unsigned fv = __hip_atomic_load(fp, __ATOMIC_RELAXED, __HIP_MEMORY_SCOPE_SYSTEM);
                if ((fv >> 10) == ctag && lane < 32) __hip_atomic_store(op + 32 + lane, fv, __ATOMIC_RELEASE, __HIP_MEMORY_SCOPE_WORKGROUP);
                ent = __builtin_amdgcn_readlane(fv, ei);
                if ((ent >> 10) != ctag) dead = expired(spins, t0, tmo, call);
            }
            if (dead) break;
            asm volatile("" ::: "memory");
            const unsigned slotT = origin + (p + k * NPW) * TAUP32 / 32;
            const int jp = NSTAT / 8 + (p >> 3) + k * 8;
            const unsigned now = gate(slotT, jp + 1 - WINP, false);
            if ((int)(now - slotT) > CREDIT) origin += now - slotT - CREDIT;
            const unsigned so = (ent & 511u) * 4096u;
            const unsigned oo = (ent & 255u) * (16u * NS * 4u) + (ent & 256u) + obase;
            if (ent & 512u) {
                const u32x4 lo0 = __builtin_amdgcn_raw_buffer_load_b128(yr, lane * 16, so, SYS);
                const u32x4 lo1 = __builtin_amdgcn_raw_buffer_load_b128(yr, lane * 16 + 1024, so, SYS);
                unsigned hi[3];
#pragma unroll
                for (int g = 0; g < 3; ++g) hi[g] = __builtin_amdgcn_raw_buffer_load_b32(yr, lane * 4 + 2048 + g * 256, so, SYS);
                const unsigned h3 = ((hi[0] >> 6) & 0x03030303u) | ((hi[1] >> 4) & 0x0c0c0c0cu) | ((hi[2] >> 2) & 0x30303030u);
#pragma unroll
                for (int g = 0; g < 4; ++g) {
                    const unsigned h = g < 3 ? hi[g < 3 ? g : 0] : h3;
                    const unsigned l0 = (g < 2 ? lo0 : lo1)[(2 * g) & 3], l1 = (g < 2 ? lo0 : lo1)[(2 * g + 1) & 3];
                    f32x4 y;
                    y[0] = (float)(int)__builtin_amdgcn_sbfe(__builtin_amdgcn_perm(h, l0, 0x0c040100), 0, 22) * scale;
                    y[1] = (float)(int)__builtin_amdgcn_sbfe(__builtin_amdgcn_perm(h, l0, 0x0c050302), 0, 22) * scale;
                    y[2] = (float)(int)__builtin_amdgcn_sbfe(__builtin_amdgcn_perm(h, l1, 0x0c060100), 0, 22) * scale;
                    y[3] = (float)(int)__builtin_amdgcn_sbfe(__builtin_amdgcn_perm(h, l1, 0x0c070302), 0, 22) * scale;
                    d[g] = __builtin_bit_cast(u32x4, y);
                }
            } else {
#pragma unroll
                for (int j = 0; j < 4; ++j) d[j] = __builtin_amdgcn_raw_buffer_load_b128(yr, lane * 16 + j * 1024, so, SYS);
                origin += (TAU32 - TAUP32) * NPW / 32;
            }
            asm volatile("" : "+v"(d[0]), "+v"(d[1]), "+v"(d[2]), "+v"(d[3]));
            finished(jp + 1);
#pragma unroll
            for (int g = 0; g < 4; ++g) __builtin_amdgcn_raw_buffer_store_b128(d[g], orr, oo + g * 4 * NS * 4, 0, SYS);
        }
        if (dead && lane == 0)
            __hip_atomic_store((gu32*)(P.out + (size_t)rank * MS * NS * 4), 0x7fc00000u, __ATOMIC_RELAXED, __HIP_MEMORY_SCOPE_SYSTEM);
        return;
    }
    if (b < 64) return;

    const int xcd = b & 7, slot = (b - 64) >> 3;
    const int rs = 3 * (xcd >> 1) + (slot >> 3);
    const bool remote = (slot & 4) != 0;
    const unsigned dj = (unsigned)slot & 3u;
    const unsigned cblk = (unsigned)xcd & 1u;
    const unsigned owner = remote ? (otcls | dj) : rank;
    const unsigned col0 = (cblk * 4u + dj) * 64u;
    LAS int* tcnt = (LAS int*)smem + MAXT * 16 * TSTR;

    u32x4 wf[8][4];
    if (!remote) {
        const unsigned wv = (unsigned)(kq * 16 * NS + c * 4);
#pragma unroll
        for (int s = 0; s < 8; ++s) {
            unsigned r[16];
            const unsigned so = (unsigned)(wave * 8 + s) * 64u * NS + col0;
#pragma unroll
            for (int i = 0; i < 16; ++i) r[i] = __builtin_amdgcn_raw_buffer_load_b32(wr, wv + i * NS, so, 0);
            transpose16x4(r, wf[s]);
        }
    }
    for (int i = threadIdx.x; i < MAXT * 16 * TSTR + 32; i += NWAVES * 64) smem[i] = 0;
    __syncthreads();
    const unsigned call = rfl(cntv) + 1;
    if (lane == 0) __hip_atomic_store(cntp, call, __ATOMIC_RELAXED, __HIP_MEMORY_SCOPE_AGENT);
    const float scale = P.sx[0] * P.sw[0];
    const unsigned pbase = (call & 1u) * PAR_BYTES;
    const rsrc_t sr = mk(myws + pbase, PAR_BYTES);
    const rsrc_t orr = remote ? mk(myws + OFF_Y + dj * (512u * 4096u), 512u * 4096u) : mk(P.out, OUTB);
    const unsigned long long t0 = rt();
    unsigned spins = 0;
    bool dead = false;

    unsigned tsrc, trb;
    bool tvalid;
    {
        const unsigned idx = (unsigned)rs + RS * (unsigned)lane;
        constexpr unsigned E1 = 3 * NIN, E2 = E1 + NAB, E3 = E2 + 4 * NGB;
        if (idx < E1) {
            const unsigned u = idx / 3u;
            tsrc = mycls | ((myidx + 1u + idx % 3u) & 3u); trb = u < NAB ? NGB + u : u - (NAB - NGB + NDL); tvalid = !remote || u < NAB;
        } else if (idx < E2) {
            tsrc = rank; trb = NGB + idx - E1; tvalid = true;
        } else if (idx < E3) {
            const unsigned u = idx - E2;
            tsrc = otcls | (u & 3u); trb = u >> 2; tvalid = !remote || (tsrc != owner && trb < NGB - NDL);
        } else {
            tsrc = rank; trb = (idx - E3) & 31u; tvalid = !remote && idx < NLIST;
        }
    }
    const unsigned tdesc = tsrc * 32u + trb;
    const unsigned long long validmask = __ballot(tvalid);
    const unsigned long long ownmask = __ballot(tvalid && tsrc == rank);
    const gu32* flagp = (tvalid && tsrc != rank) ? (const gu32*)(myws + OFF_XF + (tdesc * 8u + (unsigned)wave) * 128u) : tmo;
    const unsigned vown = (unsigned)(c * KD + kq * 16), vrem = (unsigned)(lane * 16);

    unsigned pend = (unsigned)validmask;
    auto refill = [&](unsigned ready, u32x4 (&a)[8]) {
        const unsigned todo = ready & pend;
        const int t = todo ? __builtin_ctz(todo) : -1;
        pend &= ~(todo & (0u - todo));
        const unsigned desc = __builtin_amdgcn_readlane(tdesc, t & 63);
        const bool own = (desc >> 5) == rank;
        const rsrc_t r = mk(own ? (const char*)P.x : myws + pbase, todo ? (own ? SHARD : OFF_W) : 0u);
        const unsigned so = own ? (desc & 31u) * 65536u + (unsigned)wave * 512u : desc * 65536u + (unsigned)wave * 8192u;
        const unsigned step = own ? 64u : 1024u;
        const unsigned vo = own ? vown : vrem;
#pragma unroll
        for (int s = 0; s < 8; ++s) a[s] = __builtin_amdgcn_raw_buffer_load_b128(r, vo, so + s * step, SYS);
        return t;
    };
    auto arrive = [&](int t, bool take) {
        asm volatile("s_waitcnt lgkmcnt(0)" ::: "memory");
        int old = 0;
        if (lane == 0) old = __hip_atomic_fetch_add(tcnt + t, 1, __ATOMIC_RELAXED, __HIP_MEMORY_SCOPE_WORKGROUP);
        old = rfl(old);
        if (old == NWAVES - 1 && lane == 0) __hip_atomic_fetch_or(tcnt + (take ? 18 : 16), 1 << t, __ATOMIC_RELAXED, __HIP_MEMORY_SCOPE_WORKGROUP);
        asm volatile("" ::: "memory");
        return old;
    };
    auto store_tile = [&](int t) {
        unsigned ticket = 0;
        if (remote) {
            unsigned long long keep;
            asm volatile("s_mov_b64 %1, exec\n\ts_mov_b64 exec, 1\n\tglobal_atomic_add %0, %2, %3, off sc0\n\ts_mov_b64 exec, %1"
                         : "+v"(ticket), "=&s"(keep) : "v"(myws + OFF_QT + dj * 128u), "v"(1u) : "memory");
        }
        const unsigned desc = __builtin_amdgcn_readlane(tdesc, t);
        const LAS int* tp = (const LAS int*)smem + t * 16 * TSTR + kq * TSTR + c * 4;
        unsigned v[16];
#pragma unroll
        for (int i = 0; i < 16; ++i) v[i] = (unsigned)tp[(i >> 2) * 4 * TSTR + (i & 3)];
        const bool bad = __hip_atomic_load(tcnt + 17, __ATOMIC_RELAXED, __HIP_MEMORY_SCOPE_WORKGROUP) != 0;
        const float sc = bad ? __builtin_nanf("") : scale;
        unsigned span = 0;
#pragma unroll
        for (int i = 0; i < 16; ++i) span |= v[i] + 0x200000u;
        const bool pack = remote && __all((span >> 22) == 0u && !bad);
        const unsigned id = cblk * 256u + desc;
        if (pack) {
            unsigned h[4];
            u32x4 lo[2];
#pragma unroll
            for (int g = 0; g < 4; ++g) {
                h[g] = __builtin_amdgcn_perm(v[4 * g + 1], v[4 * g], 0x0c0c0602) | __builtin_amdgcn_perm(v[4 * g + 3], v[4 * g + 2], 0x06020c0c);
                lo[g >> 1][(2 * g) & 3] = __builtin_amdgcn_perm(v[4 * g + 1], v[4 * g], 0x05040100);
                lo[g >> 1][(2 * g + 1) & 3] = __builtin_amdgcn_perm(v[4 * g + 3], v[4 * g + 2], 0x05040100);
            }
            __builtin_amdgcn_raw_buffer_store_b128(lo[0], orr, id * 4096u + lane * 16, 0, SYS);
            __builtin_amdgcn_raw_buffer_store_b128(lo[1], orr, id * 4096u + 1024 + lane * 16, 0, SYS);
#pragma unroll
            for (int g = 0; g < 3; ++g)
                __builtin_amdgcn_raw_buffer_store_b32((h[g] & 0x3f3f3f3fu) | ((h[3] << (6 - 2 * g)) & 0xc0c0c0c0u), orr,
                                                      id * 4096u + 2048 + g * 256 + lane * 4, 0, SYS);
        } else {
            const unsigned oo = remote ? id * 4096u + lane * 16 : (desc * 16u * NS + col0) * 4u + (unsigned)(kq * NS * 4 + c * 16);
            const unsigned st = remote ? 1024u : 4u * NS * 4u;
#pragma unroll
            for (int i = 0; i < 4; ++i) {
                f32x4 y;
#pragma unroll
                for (int e = 0; e < 4; ++e) y[e] = (float)(int)v[4 * i + e] * sc;
                __builtin_amdgcn_raw_buffer_store_b128(__builtin_bit_cast(u32x4, y), orr, oo + i * st, 0, SYS);
            }
        }
        if (remote) {
            asm volatile("s_waitcnt vmcnt(0)" : "+v"(ticket) :: "memory");
            const unsigned slot = ticket - (call - 1u) * NQ;
            if (lane == 0)
                __hip_atomic_store((gu32*)(myws + OFF_Q + dj * 1024u) + ((slot & 7u) * 32u + ((slot >> 3) & 31u)),
                                   (call_tag(call) << 10) | (pack ? 512u : 0u) | id, __ATOMIC_RELAXED, __HIP_MEMORY_SCOPE_SYSTEM);
        }
    };
    u32x4 a0[8], a1[8];
    int t0i = refill((unsigned)ownmask, a0);
    int t1i = refill((unsigned)ownmask, a1);
    if (remote) {
        const unsigned u0 = (dj * 2u + cblk) * 64u + (unsigned)wave * 8u;
        const gu32* fp = (const gu32*)(myws + OFF_WF + u0 * 16u + ((lane >> 1) & 3) * 4u);
        unsigned have = 0;
#pragma unroll
        for (int s = 0; s < 8; ++s) {
            while (((have >> s) & 1u) == 0u && !dead) {
                have = (unsigned)__ballot(__hip_atomic_load(fp, __ATOMIC_RELAXED, __HIP_MEMORY_SCOPE_SYSTEM) >= call);
                if (((have >> s) & 1u) == 0u) dead = expired(spins, t0, tmo, call);
            }
            asm volatile("" ::: "memory");
#pragma unroll
            for (int e = 0; e < 4; ++e)
                wf[s][e] = __builtin_amdgcn_raw_buffer_load_b128(sr, lane * 16 + e * 1024, OFF_W + (u0 + s) * 4096u, SYS);
        }
        if (dead && lane == 0) __hip_atomic_store(tcnt + 17, 1, __ATOMIC_RELAXED, __HIP_MEMORY_SCOPE_WORKGROUP);
#pragma unroll
        for (int s = 0; s < 8; s += 2)
            asm volatile("" : "+v"(wf[s][0]), "+v"(wf[s][1]), "+v"(wf[s][2]), "+v"(wf[s][3]),
                              "+v"(wf[s + 1][0]), "+v"(wf[s + 1][1]), "+v"(wf[s + 1][2]), "+v"(wf[s + 1][3]));
    }

    unsigned fv = __hip_atomic_load(flagp, __ATOMIC_RELAXED, __HIP_MEMORY_SCOPE_SYSTEM);
    asm volatile("" : "+v"(fv));
    auto claim = [&]() {
        const unsigned fin = rfl(__hip_atomic_load(tcnt + 16, __ATOMIC_RELAXED, __HIP_MEMORY_SCOPE_WORKGROUP));
        const unsigned clm = rfl(__hip_atomic_load(tcnt + 18, __ATOMIC_RELAXED, __HIP_MEMORY_SCOPE_WORKGROUP));
        if ((fin & ~clm) == 0u) return clm == (unsigned)validmask ? -2 : -1;
        const int ts = __builtin_ctz(fin & ~clm);
        int old = 0;
        if (lane == 0) old = __hip_atomic_fetch_or(tcnt + 18, 1 << ts, __ATOMIC_RELAXED, __HIP_MEMORY_SCOPE_WORKGROUP);
        asm volatile("" ::: "memory");
        return ((rfl(old) >> ts) & 1u) != 0u ? -1 : ts;
    };
    auto mma = [&](const u32x4 (&a)[8], i32x4 (&acc)[4]) {
#pragma unroll
        for (int e = 0; e < 4; ++e) acc[e] = (i32x4){0, 0, 0, 0};
#pragma unroll
        for (int s = 0; s < 8; ++s)
#pragma unroll
            for (int e = 0; e < 4; ++e)
                acc[e] = __builtin_amdgcn_mfma_i32_16x16x64_i8(__builtin_bit_cast(i32x4, wf[s][e]), __builtin_bit_cast(i32x4, a[s]), acc[e], 0, 0, 0);
    };
    auto add_tile = [&](int t, const i32x4 (&acc)[4], bool take) {
        LAS int* tp = (LAS int*)smem + t * 16 * TSTR + c * TSTR + kq * 16;
#pragma unroll
        for (int r = 0; r < 4; ++r)
#pragma unroll
            for (int e = 0; e < 4; ++e)
                __hip_atomic_fetch_add(tp + r * 4 + e, acc[e][r], __ATOMIC_RELAXED, __HIP_MEMORY_SCOPE_WORKGROUP);
        const int pos = arrive(t, take);
        return pos;
    };
    auto ready = [&]() {
        if (__builtin_amdgcn_readlane(fv, 63) == call) dead = true;
        const unsigned r = (unsigned)__ballot(fv >= call) | (unsigned)ownmask;
        asm volatile("" ::: "memory");
        return dead ? 0u : r;
    };
    auto half = [&](u32x4 (&a)[8], int& t, int other) {
        i32x4 acc[4];
        const int tc = t;
        if (tc >= 0) mma(a, acc);
        const unsigned rdy = ready();
        fv = __hip_atomic_load(flagp, __ATOMIC_RELAXED, __HIP_MEMORY_SCOPE_SYSTEM);
        t = refill(rdy, a);
        __builtin_amdgcn_sched_barrier(0);
        if (tc >= 0) {
            const bool busy = t >= 0 || other >= 0;
            const int pos = add_tile(tc, acc, !busy);
            if (busy ? pos < AHEAD : pos == NWAVES - 1) {
                const int ts = busy ? claim() : tc;
                if (ts >= 0) store_tile(ts);
            }
        }
    };
    for (;;) {
        do {
            half(a0, t0i, t1i);
            half(a1, t1i, t0i);
        } while (t0i >= 0 || t1i >= 0);
        unsigned rdy;
        for (;;) {
            rdy = ready() & pend;
            if ((rdy & (rdy - 1u)) != 0u) break;
            int ts;
            if (rdy != 0u) {
                const int t = refill(rdy, a0);
                fv = __hip_atomic_load(flagp, __ATOMIC_RELAXED, __HIP_MEMORY_SCOPE_SYSTEM);
                __builtin_amdgcn_sched_barrier(0);
                i32x4 acc[4];
                mma(a0, acc);
                ts = add_tile(t, acc, true) == NWAVES - 1 ? t : -1;
            } else {
                if (dead && pend != 0u) {
                    if (lane == 0) __hip_atomic_store(tcnt + 17, 1, __ATOMIC_RELAXED, __HIP_MEMORY_SCOPE_WORKGROUP);
                    for (; pend != 0u; pend &= pend - 1u) arrive(__builtin_ctz(pend), false);
                }
                if (pend != 0u) {
                    if (expired(spins, t0, tmo, call)) dead = true;
                    fv = __hip_atomic_load(flagp, __ATOMIC_RELAXED, __HIP_MEMORY_SCOPE_SYSTEM);
                } else {
                    __builtin_amdgcn_s_sleep(1);
                }
                ts = claim();
                if (ts == -2) {
                    asm volatile("s_endpgm");
                }
            }
            asm volatile("" : "+v"(fv));
            ts = (int)rfl((unsigned)ts);
            if (ts >= 0) store_tile(ts);
        }
        t0i = refill(rdy, a0);
        t1i = refill(rdy, a1);
    }
}

extern "C" void kernel_launch(int rank, int nranks, ncclComm_t comm,
                              hipStream_t stream,
                              void* const* d_in, const int* in_sizes, int n_in,
                              void* d_out, int out_size,
                              const int* in_dtypes, int out_dtype,
                              void* const* d_out_all,
                              void* const* d_ws_all, size_t ws_bytes) {
    if (nranks != NR || (unsigned)rank >= (unsigned)NR || n_in != 4 || in_sizes[0] != MS * KD || in_sizes[1] != KD * NS
        || out_size != NR * MS * NS || ws_bytes < WS_NEED) {
        fprintf(stderr, "kernel_launch: unexpected shapes\n");
        return;
    }
    Params P;
    P.x = (const signed char*)d_in[0];
    P.w = (const signed char*)d_in[1];
    P.sx = (const float*)d_in[2];
    P.sw = (const float*)d_in[3];
    for (int i = 0; i < NR; ++i) {
        const int p = i < 3 ? ((rank & 4) | ((rank + 1 + i) & 3)) : (i < 7 ? (((rank & 4) ^ 4) | (i - 3)) : rank);
        P.ws[i] = (char*)d_ws_all[p];
    }
    P.out = (char*)d_out;
    P.rank = rank;
    P.pad = 0;
    ag_gemm_kernel<<<GRID, NWAVES * 64, 0, stream>>>(P);
    HIP_CHECK(hipGetLastError());
}
```
